```python
import numpy as np

kernel_name = "gpu_a2a_gemm_m4096_k4096_n16384_e4m3e5m2_dqsilu"


ABSMAX_RTOL = 0.05

COMPUTE_DTYPE = "float32"
EPILOGUE = "dequant_silu"
IN_QUANT = None
OUT_QUANT = None
IN_FP8 = "e4m3e5m2"
IN_DTYPES = {"x": "e4m3", "w_mat": "e5m2",
             "scale_x": "float32", "scale_w": "float32"}
WORLD_SIZE = 8

def _snap_fp8(x, mant_bits, max_abs, min_normal):
    a = np.abs(x.astype(np.float64))
    _m, _e = np.frexp(a)
    q = np.where(a >= min_normal, np.ldexp(1.0, _e - 1 - mant_bits),
                 min_normal * 2.0 ** -mant_bits)
    snapped = np.minimum(np.round(a / q) * q, max_abs)
    return (np.sign(x) * snapped).astype(np.float32)

def setup_inputs():

    x = _snap_fp8(np.random.randn(4096, 4096).astype(np.float32), 3, 448.0, 2.0 ** -6)
    w_mat = _snap_fp8(np.random.randn(4096, 16384).astype(np.float32), 2, 57344.0, 2.0 ** -14)
    scale_x = (0.02 * (1.0 + np.random.rand(1))).astype(np.float32)
    scale_w = (0.02 * (1.0 + np.random.rand(1))).astype(np.float32)
    return {"x": x, "w_mat": w_mat, "scale_x": scale_x, "scale_w": scale_w}

def reference(x, w_mat, scale_x, scale_w):
    acc = (x @ w_mat).astype(np.float32)
    y = (acc * (scale_x * scale_w)).astype(np.float32)
    return (y / (1.0 + np.exp(-np.clip(y, -60.0, 60.0)))).astype(np.float32)
```

```cpp
#include <hip/hip_runtime.h>
#include <rccl/rccl.h>
#include <cstdio>
#include <cstdint>

#define HIP_CHECK(expr) do { hipError_t _e = (expr); if (_e != hipSuccess) { \
    fprintf(stderr, "HIP error %s at %s:%d\n", hipGetErrorString(_e), __FILE__, __LINE__); \
    return; } } while (0)

typedef unsigned v8u __attribute__((ext_vector_type(8)));
typedef unsigned v4u __attribute__((ext_vector_type(4)));
#define LAS __attribute__((address_space(3)))
#define DI __device__ __forceinline__

constexpr int NR = 8;
constexpr int KTOT = 4096, NTOT = 16384, MLOC = 512, KLOC = 512;
constexpr unsigned XT_BYTES = 28u * 65536u;
constexpr unsigned SLOT_STRIDE = 2u << 20;
constexpr unsigned CTR_OFF = 6u << 20;
constexpr unsigned TMO_OFF = CTR_OFF + 256u * 128u;
constexpr unsigned WS_NEED = TMO_OFF + 128u;
constexpr unsigned LDS_BYTES = 163840;
static_assert(XT_BYTES == 0x1c0000 && TMO_OFF == 0x608000 && SLOT_STRIDE == (1u << 21), "constants used by the asm");

struct Args {
  const uint8_t* x; const uint8_t* w; uint8_t* myws; float* out; const float* sx; const float* sw;
  unsigned rank, pad;
  void* pad2;
  uint8_t* dws[NR];
};

DI unsigned rfl(unsigned x) { return __builtin_amdgcn_readfirstlane(x); }
typedef __amdgpu_buffer_rsrc_t rsrc_t;
DI rsrc_t mk(const void* p, unsigned bytes) { return __builtin_amdgcn_make_buffer_rsrc((void*)p, (short)0, (int)bytes, 0x00020000); }

__global__ __launch_bounds__(512, 2) void a2a_gemm_kernel(Args a) {
  extern __shared__ __attribute__((aligned(16))) char smem[];
  const unsigned lane = threadIdx.x & 63;
  const unsigned w = rfl(threadIdx.x >> 6);
  const uint64_t dws = (uint64_t)a.dws[w & 7];
  const unsigned b = blockIdx.x & 255, xcd = b & 7, idx = b >> 3, tile = idx >> 1, ntile = xcd * 16 + tile, mh = idx & 1;
  const unsigned rank = a.rank & 7;
  const unsigned ldsb = (unsigned)(size_t)(LAS char*)smem;
  const uint64_t myws = (uint64_t)a.myws;
  const unsigned dst = (rank + 7 - w) & 7;
  const unsigned pc = b >> 5, prg = b & 31;
  const rsrc_t srdM = mk(a.myws, WS_NEED);
  const rsrc_t srdXin = mk(a.x, KTOT * KLOC);
  const rsrc_t srdW = mk(a.w + ntile * 128, (unsigned)KTOT * NTOT - ntile * 128);
  const unsigned fw = ((lane >> 4) & 3) | ((w & 1) << 2);
  const unsigned wvoff = (lane >> 3) * NTOT + (((lane & 7) ^ fw) << 4);
  const unsigned rg0 = mh * 16 + w * 2;
  const unsigned osoff = (rank * 512u + rg0 * 16u) * 512u;
  const unsigned ovoff = (lane & 15) * 512u + (lane >> 4) * 16u;
  const unsigned xovoff = (lane >> 3) * 512u + (((lane & 7) ^ ((lane >> 4) & 3)) << 4);
  const unsigned pushstep = (0x98653200u >> (pc * 4)) & 15u;
  const unsigned ctroff = CTR_OFF + b * 128u;
  const unsigned xpoff = (dst * 512u + prg * 16u) * 512u + pc * 64u;
  const unsigned m0w = ldsb + w * 2048u;
  const unsigned c0 = (mh + tile) & 3u;
  const unsigned w0 = (rank * 512u + c0 * 128u + w * 16u) * NTOT, x0 = osoff + c0 * 128u;

  v4u xp; unsigned ctr;
  asm volatile(
    "v_mov_b32 v52, 0\n\t"
    "buffer_load_dword v52, v52, %[srdM], %[ctroff] offen sc0 sc1\n\t"
    "s_cmp_lg_u32 %[pushstep], 0\n\t"
    "s_cbranch_scc1 L_nx0%=\n\t"
    "buffer_load_dwordx4 v[48:51], %[ovoff], %[srdXin], %[xpoff] offen\n\t"
    "L_nx0%=:\n\t"
    "s_add_u32 m0, %[m0w], 0\n\t"
    "s_nop 0\n\t"
    "buffer_load_dwordx4 %[wvoff], %[srdW], %[w0a] offen lds\n\t"
    "s_add_u32 m0, %[m0w], 1024\n\t"
    "s_nop 0\n\t"
    "buffer_load_dwordx4 %[wvoff], %[srdW], %[w0b] offen lds\n\t"
    "s_add_u32 m0, %[m0w], 131072\n\t"
    "s_nop 0\n\t"
    "buffer_load_dwordx4 %[xovoff], %[srdXin], %[x0a] offen lds\n\t"
    "s_add_u32 m0, %[m0w], 132096\n\t"
    "s_nop 0\n\t"
    "buffer_load_dwordx4 %[xovoff], %[srdXin], %[x0b] offen lds\n\t"
    "s_add_u32 m0, %[m0w], 147456\n\t"
    "s_nop 0\n\t"
    "buffer_load_dwordx4 %[xovoff], %[srdXin], %[x0c] offen lds\n\t"
    "s_add_u32 m0, %[m0w], 148480\n\t"
    "s_nop 0\n\t"
    "buffer_load_dwordx4 %[xovoff], %[srdXin], %[x0d] offen lds\n\t"
    : "=&{v[48:51]}"(xp), "=&{v52}"(ctr)
    : [srdW] "s"(srdW), [srdXin] "s"(srdXin), [srdM] "s"(srdM), [ctroff] "s"(ctroff), [xpoff] "s"(xpoff), [pushstep] "s"(pushstep),
      [m0w] "s"(m0w), [w0a] "s"(w0), [w0b] "s"(w0 + 8u * NTOT), [x0a] "s"(x0), [x0b] "s"(x0 + 4096u), [x0c] "s"(x0 + 8192u), [x0d] "s"(x0 + 12288u),
      [wvoff] "v"(wvoff), [ovoff] "v"(ovoff), [xovoff] "v"(xovoff)
    : "memory", "scc");

  const rsrc_t srdPF = mk(a.w, (unsigned)KTOT * NTOT);
  const rsrc_t srdO = mk(a.out, (unsigned)MLOC * NTOT * 4u);
  const unsigned g4 = lane >> 4, q8 = (lane & 15) >> 1, p = lane & 1, r16 = lane & 15;
  const unsigned frs = (((q8 >> 1) & 3) | ((g4 & 1) << 2)) << 4;
  const unsigned tabase = ldsb + g4 * 2048 + q8 * 128 + p * 8;
  const unsigned pvst = ((pc >> 1) * 7u + w) * 65536u + prg * 2048u + (pc & 1) * 1024u + lane * 16u;
  auto kchunk = [&](unsigned st, unsigned tl, unsigned& xo) -> unsigned {
    const unsigned so = (st + tl) & 3u;
    const unsigned sp = st - 4, j = (sp * 37u) >> 8, gam = 1 == 0 ? 0u : tl - (tl >= 14 ? 14u : tl >= 7 ? 7u : 0u);
    unsigned pi = sp - j * 7u + gam; pi -= pi >= 7 ? 7u : 0u;
    xo = st < 4 ? so : j * 7u + pi;
    return st < 4 ? rank * 512u + so * 128u : ((rank + 1u + pi) & 7u) * 512u + j * 128u;
  };
  unsigned tw, tx;
  {
    const unsigned st = (lane & 31) ^ mh;
    unsigned xo;
    tw = (kchunk(st, tile, xo) + w * 16u) * NTOT;
    tx = st < 4 ? osoff + xo * 128u : xo * 65536u + rg0 * 2048u;
  }
  unsigned pfso, pfw;
  {
    const unsigned l0 = (unsigned)((uint64_t)a.w >> 7) + xcd * 16u;
    const unsigned sidx = tile & 3u;
    const unsigned tl = sidx < 2 ? ((3u - l0) & 7u) + sidx * 8u : sidx == 2 ? (63u - l0) & 63u : 16u;
    const unsigned st = (unsigned)((0x1a16121e0e0a0602ull >> (w * 8)) & 255u) + (idx >> 3);
    const bool on = tl < 16 && st < 32 && (1 == 0 || ((idx ^ w) & 1) == 0);
    unsigned xo;
    pfso = rfl((kchunk(st & 31, tl & 15, xo) + (1 ? 0u : (idx & 1) * 8u)) * NTOT + (xcd * 16u + (tl & 15)) * 128u);
    pfw = rfl(on ? w : 99u);
  }
  const unsigned etb = ldsb + 6u * 16384u + w * 2048u;
  asm volatile(
    "L_start%=:\n\t"
    "s_getpc_b64 s[84:85]\n\t"
    "s_memrealtime s[80:81]\n\t"
    "s_load_dword s96, %[psx], 0x0\n\t"
    "s_load_dword s97, %[psw], 0x0\n\t"
    "s_and_b32 s85, s85, 0xffff\n\t"
    "s_mov_b32 s86, L_end%=-L_start%=-8\n\t"
    "s_mov_b32 s87, 0x20000\n\t"
    "v_mov_b32 v55, 0\n\t"
    "s_mov_b32 s79, 0x7f000000\n\t"
    "s_cmp_eq_u32 %[pfw], 0\n\t"
    "s_cbranch_scc0 L_cpf0_%=\n\t"
    "buffer_load_dword v46, %[pfvoff], %[srdPF], %[pfso] offen sc1\n\t"
    "s_branch L_dpf0_%=\n\t"
    "L_cpf0_%=:\n\t"
    "buffer_load_dword v46, v55, %[srdM], s79 offen\n\t"
    "L_dpf0_%=:\n\t"
    "s_cmp_eq_u32 %[cpfw], 63\n\t"
    "s_cbranch_scc0 L_ccp0_%=\n\t"
    "s_mov_b32 s77, 0\n\t"
    "buffer_load_dword v46, %[cvoff], s[84:87], s77 offen sc1\n\t"
    "s_branch L_dcp0_%=\n\t"
    "L_ccp0_%=:\n\t"
    "buffer_load_dword v46, v55, %[srdM], s79 offen\n\t"
    "L_dcp0_%=:\n\t"
    "v_xor_b32 v56, 0, %[frs]\n\t"
    "v_add_u32 v56, v56, %[tabase]\n\t"
    "v_xor_b32 v57, 16, %[frs]\n\t"
    "v_add_u32 v57, v57, %[tabase]\n\t"
    "v_xor_b32 v58, 32, %[frs]\n\t"
    "v_add_u32 v58, v58, %[tabase]\n\t"
    "v_xor_b32 v59, 48, %[frs]\n\t"
    "v_add_u32 v59, v59, %[tabase]\n\t"
    "v_xor_b32 v60, 64, %[frs]\n\t"
    "v_add_u32 v60, v60, %[tabase]\n\t"
    "v_xor_b32 v61, 80, %[frs]\n\t"
    "v_add_u32 v61, v61, %[tabase]\n\t"
    "v_xor_b32 v62, 96, %[frs]\n\t"
    "v_add_u32 v62, v62, %[tabase]\n\t"
    "v_xor_b32 v63, 112, %[frs]\n\t"
    "v_add_u32 v63, v63, %[tabase]\n\t"
    "v_mov_b32 v47, %[xrd]\n\t"
    "s_mov_b32 s70, 0\n\t"
    "s_lshr_b32 s74, %[wid], 2\n\t"
    "s_cmp_eq_u32 %[wid], 7\n\t"
    "s_cselect_b32 s71, 0, 0x1c0000\n\t"
    "s_cmp_eq_u32 %[pushstep], 0\n\t"
    "s_cselect_b32 s66, s71, 0\n\t"
    "s_mov_b32 s67, 0x20000\n\t"
    "v_readlane_b32 s77, %[tw], 1\n\t"
    "s_mov_b32 s73, s77\n\t"
    "s_add_u32 m0, %[m0w], 16384\n\t"
    "s_nop 0\n\t"
    "buffer_load_dwordx4 %[wvoff], %[srdW], s73 offen lds\n\t"
    "s_add_u32 m0, %[m0w], 17408\n\t"
    "s_add_u32 s73, s73, 131072\n\t"
    "buffer_load_dwordx4 %[wvoff], %[srdW], s73 offen lds\n\t"
    "v_readlane_b32 s77, %[tx], 1\n\t"
    "s_mov_b32 s76, s77\n\t"
    "s_add_u32 m0, %[m0w], 98304\n\t"
    "s_nop 0\n\t"
    "buffer_load_dwordx4 %[xovoff], %[srdXin], s76 offen lds\n\t"
    "s_add_u32 s76, s76, 4096\n\t"
    "s_add_u32 m0, %[m0w], 99328\n\t"
    "s_nop 0\n\t"
    "buffer_load_dwordx4 %[xovoff], %[srdXin], s76 offen lds\n\t"
    "s_add_u32 s76, s76, 4096\n\t"
    "s_add_u32 m0, %[m0w], 114688\n\t"
    "s_nop 0\n\t"
    "buffer_load_dwordx4 %[xovoff], %[srdXin], s76 offen lds\n\t"
    "s_add_u32 s76, s76, 4096\n\t"
    "s_add_u32 m0, %[m0w], 115712\n\t"
    "s_nop 0\n\t"
    "buffer_load_dwordx4 %[xovoff], %[srdXin], s76 offen lds\n\t"
    "v_readlane_b32 s77, %[tw], 2\n\t"
    "s_mov_b32 s73, s77\n\t"
    "s_add_u32 m0, %[m0w], 32768\n\t"
    "s_nop 0\n\t"
    "buffer_load_dwordx4 %[wvoff], %[srdW], s73 offen lds\n\t"
    "s_add_u32 m0, %[m0w], 33792\n\t"
    "s_add_u32 s73, s73, 131072\n\t"
    "buffer_load_dwordx4 %[wvoff], %[srdW], s73 offen lds\n\t"
    "s_waitcnt vmcnt(16)\n\t"
    "v_readfirstlane_b32 s68, v52\n\t"
    "s_nop 1\n\t"
    "s_add_u32 s68, s68, 1\n\t"
    "s_mul_hi_u32 s77, s68, 0xaaaaaaab\n\t"
    "s_lshr_b32 s77, s77, 1\n\t"
    "s_mul_i32 s77, s77, 3\n\t"
    "s_sub_u32 s77, s68, s77\n\t"
    "s_lshl_b32 s78, s77, 21\n\t"
    "s_add_u32 s64, %[wsDstLo], s78\n\t"
    "s_addc_u32 s65, %[wsDstHi], 0\n\t"
    "s_and_b32 s65, s65, 0xffff\n\t"
    "v_mov_b32 v54, 0x80\n\t"
    "v_cmp_eq_u32 vcc, 0, v48\n\t"
    "s_nop 1\n\t"
    "v_cndmask_b32 v48, v48, v54, vcc\n\t"
    "v_cmp_eq_u32 vcc, 0, v49\n\t"
    "s_nop 1\n\t"
    "v_cndmask_b32 v49, v49, v54, vcc\n\t"
    "v_cmp_eq_u32 vcc, 0, v50\n\t"
    "s_nop 1\n\t"
    "v_cndmask_b32 v50, v50, v54, vcc\n\t"
    "v_cmp_eq_u32 vcc, 0, v51\n\t"
    "s_nop 1\n\t"
    "v_cndmask_b32 v51, v51, v54, vcc\n\t"
    "buffer_store_dwordx4 v[48:51], %[pvst], s[64:67], 0 offen sc0 sc1\n\t"
    "s_add_u32 s60, %[wsMyLo], s78\n\t"
    "s_addc_u32 s61, %[wsMyHi], 0\n\t"
    "s_and_b32 s61, s61, 0xffff\n\t"
    "s_mov_b32 s62, 0x1c0000\n\t"
    "s_mov_b32 s63, 0x20000\n\t"
    "s_add_u32 s77, s77, 2\n\t"
    "s_cmp_ge_u32 s77, 3\n\t"
    "s_cselect_b32 s78, 3, 0\n\t"
    "s_sub_u32 s77, s77, s78\n\t"
    "s_lshl_b32 s78, s77, 21\n\t"
    "s_add_u32 s72, s78, %[zoff]\n\t"
    "s_mov_b32 s66, s71\n\t"
    "s_waitcnt lgkmcnt(0)\n\t"
    "s_mov_b32 s69, s80\n\t"
    "v_mov_b32 v53, s96\n\t"
    "v_mul_f32 v53, s97, v53\n\t"
    "v_mul_f32 v54, 0xbfb8aa3b, v53\n\t"
    "s_nop 0\n\t"
    "v_readfirstlane_b32 s98, v53\n\t"
    "v_readfirstlane_b32 s100, v54\n\t"
    "s_waitcnt vmcnt(11)\n\t"
    "ds_read_b128 v[128:131], v47 offset:32768\n\t"
    "ds_read_b128 v[132:135], v47 offset:32832\n\t"
    "ds_read_b128 v[136:139], v47 offset:49152\n\t"
    "ds_read_b128 v[140:143], v47 offset:49216\n\t"
    "s_barrier\n\t"
    "ds_read_b64_tr_b8 v[64:65], v56 offset:0\n\t"
    "ds_read_b64_tr_b8 v[66:67], v56 offset:1024\n\t"
    "ds_read_b64_tr_b8 v[68:69], v56 offset:8192\n\t"
    "ds_read_b64_tr_b8 v[70:71], v56 offset:9216\n\t"
    "ds_read_b64_tr_b8 v[72:73], v57 offset:0\n\t"
    "ds_read_b64_tr_b8 v[74:75], v57 offset:1024\n\t"
    "ds_read_b64_tr_b8 v[76:77], v57 offset:8192\n\t"
    "ds_read_b64_tr_b8 v[78:79], v57 offset:9216\n\t"
    "ds_read_b64_tr_b8 v[80:81], v58 offset:0\n\t"
    "ds_read_b64_tr_b8 v[82:83], v58 offset:1024\n\t"
    "ds_read_b64_tr_b8 v[84:85], v58 offset:8192\n\t"
    "ds_read_b64_tr_b8 v[86:87], v58 offset:9216\n\t"
    "ds_read_b64_tr_b8 v[88:89], v59 offset:0\n\t"
    "ds_read_b64_tr_b8 v[90:91], v59 offset:1024\n\t"
    "ds_read_b64_tr_b8 v[92:93], v59 offset:8192\n\t"
    "ds_read_b64_tr_b8 v[94:95], v59 offset:9216\n\t"
    "s_waitcnt lgkmcnt(12)\n\t"
    "v_mfma_f32_16x16x128_f8f6f4 v[192:195], v[64:71], v[128:135], 0 cbsz:1\n\t"
    "ds_read_b64_tr_b8 v[96:97], v60 offset:0\n\t"
    "ds_read_b64_tr_b8 v[98:99], v60 offset:1024\n\t"
    "v_readlane_b32 s77, %[tw], 3\n\t"
    "s_mov_b32 s73, s77\n\t"
    "s_add_u32 m0, %[m0w], 49152\n\t"
    "s_nop 0\n\t"
    "buffer_load_dwordx4 %[wvoff], %[srdW], s73 offen lds\n\t"
    "v_mfma_f32_16x16x128_f8f6f4 v[224:227], v[64:71], v[136:143], 0 cbsz:1\n\t"
    "ds_read_b64_tr_b8 v[100:101], v60 offset:8192\n\t"
    "ds_read_b64_tr_b8 v[102:103], v60 offset:9216\n\t"
    "s_waitcnt lgkmcnt(12)\n\t"
    "v_mfma_f32_16x16x128_f8f6f4 v[196:199], v[72:79], v[128:135], 0 cbsz:1\n\t"
    "ds_read_b64_tr_b8 v[104:105], v61 offset:0\n\t"
    "ds_read_b64_tr_b8 v[106:107], v61 offset:1024\n\t"
    "s_add_u32 m0, %[m0w], 50176\n\t"
    "s_add_u32 s73, s73, 131072\n\t"
    "buffer_load_dwordx4 %[wvoff], %[srdW], s73 offen lds\n\t"
    "v_mfma_f32_16x16x128_f8f6f4 v[228:231], v[72:79], v[136:143], 0 cbsz:1\n\t"
    "ds_read_b64_tr_b8 v[108:109], v61 offset:8192\n\t"
    "ds_read_b64_tr_b8 v[110:111], v61 offset:9216\n\t"
    "s_waitcnt lgkmcnt(12)\n\t"
    "v_mfma_f32_16x16x128_f8f6f4 v[200:203], v[80:87], v[128:135], 0 cbsz:1\n\t"
    "ds_read_b64_tr_b8 v[112:113], v62 offset:0\n\t"
    "ds_read_b64_tr_b8 v[114:115], v62 offset:1024\n\t"
    "v_readlane_b32 s77, %[tx], 2\n\t"
    "s_mov_b32 s76, s77\n\t"
    "s_add_u32 m0, %[m0w], 131072\n\t"
    "s_nop 0\n\t"
    "buffer_load_dwordx4 %[xovoff], %[srdXin], s76 offen lds\n\t"
    "s_cmp_eq_u32 %[pfw], 1\n\t"
    "s_cbranch_scc0 L_cpf1_%=\n\t"
    "buffer_load_dword v46, %[pfvoff], %[srdPF], %[pfso] offen sc1\n\t"
    "s_branch L_dpf1_%=\n\t"
    "L_cpf1_%=:\n\t"
    "buffer_load_dword v46, v55, %[srdM], s79 offen\n\t"
    "L_dpf1_%=:\n\t"
    "v_mfma_f32_16x16x128_f8f6f4 v[232:235], v[80:87], v[136:143], 0 cbsz:1\n\t"
    "ds_read_b64_tr_b8 v[116:117], v62 offset:8192\n\t"
    "ds_read_b64_tr_b8 v[118:119], v62 offset:9216\n\t"
    "s_waitcnt lgkmcnt(12)\n\t"
    "v_mfma_f32_16x16x128_f8f6f4 v[204:207], v[88:95], v[128:135], 0 cbsz:1\n\t"
    "ds_read_b64_tr_b8 v[120:121], v63 offset:0\n\t"
    "ds_read_b64_tr_b8 v[122:123], v63 offset:1024\n\t"
    "s_waitcnt vmcnt(7)\n\t"
    "ds_read_b128 v[144:147], v47 offset:0\n\t"
    "ds_read_b128 v[148:151], v47 offset:64\n\t"
    "ds_read_b128 v[152:155], v47 offset:16384\n\t"
    "ds_read_b128 v[156:159], v47 offset:16448\n\t"
    "v_mfma_f32_16x16x128_f8f6f4 v[236:239], v[88:95], v[136:143], 0 cbsz:1\n\t"
    "ds_read_b64_tr_b8 v[124:125], v63 offset:8192\n\t"
    "ds_read_b64_tr_b8 v[126:127], v63 offset:9216\n\t"
    "s_add_u32 s76, s76, 4096\n\t"
    "s_add_u32 m0, %[m0w], 132096\n\t"
    "s_nop 0\n\t"
    "buffer_load_dwordx4 %[xovoff], %[srdXin], s76 offen lds\n\t"
    "s_waitcnt vmcnt(12)\n\t"
    "s_barrier\n\t"
    "s_waitcnt lgkmcnt(12)\n\t"
    "v_mfma_f32_16x16x128_f8f6f4 v[208:211], v[96:103], v[128:135], 0 cbsz:1\n\t"
    "ds_read_b64_tr_b8 v[64:65], v56 offset:16384\n\t"
    "ds_read_b64_tr_b8 v[66:67], v56 offset:17408\n\t"
    "v_mfma_f32_16x16x128_f8f6f4 v[240:243], v[96:103], v[136:143], 0 cbsz:1\n\t"
    "ds_read_b64_tr_b8 v[68:69], v56 offset:24576\n\t"
    "ds_read_b64_tr_b8 v[70:71], v56 offset:25600\n\t"
    "s_add_u32 s76, s76, 4096\n\t"
    "s_add_u32 m0, %[m0w], 147456\n\t"
    "s_nop 0\n\t"
    "buffer_load_dwordx4 %[xovoff], %[srdXin], s76 offen lds\n\t"
    "s_waitcnt lgkmcnt(12)\n\t"
    "v_mfma_f32_16x16x128_f8f6f4 v[212:215], v[104:111], v[128:135], 0 cbsz:1\n\t"
    "ds_read_b64_tr_b8 v[72:73], v57 offset:16384\n\t"
    "ds_read_b64_tr_b8 v[74:75], v57 offset:17408\n\t"
    "v_mfma_f32_16x16x128_f8f6f4 v[244:247], v[104:111], v[136:143], 0 cbsz:1\n\t"
    "ds_read_b64_tr_b8 v[76:77], v57 offset:24576\n\t"
    "ds_read_b64_tr_b8 v[78:79], v57 offset:25600\n\t"
    "s_waitcnt lgkmcnt(12)\n\t"
    "v_mfma_f32_16x16x128_f8f6f4 v[216:219], v[112:119], v[128:135], 0 cbsz:1\n\t"
    "ds_read_b64_tr_b8 v[80:81], v58 offset:16384\n\t"
    "ds_read_b64_tr_b8 v[82:83], v58 offset:17408\n\t"
    "s_add_u32 s76, s76, 4096\n\t"
    "s_add_u32 m0, %[m0w], 148480\n\t"
    "s_nop 0\n\t"
    "buffer_load_dwordx4 %[xovoff], %[srdXin], s76 offen lds\n\t"
    "v_mfma_f32_16x16x128_f8f6f4 v[248:251], v[112:119], v[136:143], 0 cbsz:1\n\t"
    "ds_read_b64_tr_b8 v[84:85], v58 offset:24576\n\t"
    "ds_read_b64_tr_b8 v[86:87], v58 offset:25600\n\t"
    "s_waitcnt lgkmcnt(12)\n\t"
    "v_mfma_f32_16x16x128_f8f6f4 v[220:223], v[120:127], v[128:135], 0 cbsz:1\n\t"
    "ds_read_b64_tr_b8 v[88:89], v59 offset:16384\n\t"
    "ds_read_b64_tr_b8 v[90:91], v59 offset:17408\n\t"
    "v_readlane_b32 s77, %[tw], 4\n\t"
    "s_mov_b32 s73, s77\n\t"
    "s_add_u32 m0, %[m0w], 65536\n\t"
    "s_nop 0\n\t"
    "buffer_load_dwordx4 %[wvoff], %[srdW], s73 offen lds\n\t"
    "v_mfma_f32_16x16x128_f8f6f4 v[252:255], v[120:127], v[136:143], 0 cbsz:1\n\t"
    "ds_read_b64_tr_b8 v[92:93], v59 offset:24576\n\t"
    "ds_read_b64_tr_b8 v[94:95], v59 offset:25600\n\t"
    "s_waitcnt lgkmcnt(12)\n\t"
    "v_mfma_f32_16x16x128_f8f6f4 v[192:195], v[64:71], v[144:151], v[192:195] cbsz:1\n\t"
    "ds_read_b64_tr_b8 v[96:97], v60 offset:16384\n\t"
    "ds_read_b64_tr_b8 v[98:99], v60 offset:17408\n\t"
    "v_mfma_f32_16x16x128_f8f6f4 v[224:227], v[64:71], v[152:159], v[224:227] cbsz:1\n\t"
    "ds_read_b64_tr_b8 v[100:101], v60 offset:24576\n\t"
    "ds_read_b64_tr_b8 v[102:103], v60 offset:25600\n\t"
    "s_add_u32 m0, %[m0w], 66560\n\t"
    "s_add_u32 s73, s73, 131072\n\t"
    "buffer_load_dwordx4 %[wvoff], %[srdW], s73 offen lds\n\t"
    "s_waitcnt lgkmcnt(12)\n\t"
    "v_mfma_f32_16x16x128_f8f6f4 v[196:199], v[72:79], v[144:151], v[196:199] cbsz:1\n\t"
    "ds_read_b64_tr_b8 v[104:105], v61 offset:16384\n\t"
    "ds_read_b64_tr_b8 v[106:107], v61 offset:17408\n\t"
    "v_mfma_f32_16x16x128_f8f6f4 v[228:231], v[72:79], v[152:159], v[228:231] cbsz:1\n\t"
    "ds_read_b64_tr_b8 v[108:109], v61 offset:24576\n\t"
    "ds_read_b64_tr_b8 v[110:111], v61 offset:25600\n\t"
    "v_readlane_b32 s77, %[tx], 3\n\t"
    "s_mov_b32 s76, s77\n\t"
    "s_add_u32 m0, %[m0w], 98304\n\t"
    "s_nop 0\n\t"
    "buffer_load_dwordx4 %[xovoff], %[srdXin], s76 offen lds\n\t"
    "s_cmp_eq_u32 %[pushstep], 2\n\t"
    "s_cbranch_scc0 L_cxp2_%=\n\t"
    "buffer_load_dwordx4 v[48:51], %[ovoff], %[srdXin], %[xpoff] offen\n\t"
    "s_branch L_dxp2_%=\n\t"
    "L_cxp2_%=:\n\t"
    "buffer_load_dword v46, v55, %[srdM], s79 offen\n\t"
    "L_dxp2_%=:\n\t"
    "s_waitcnt lgkmcnt(12)\n\t"
    "v_mfma_f32_16x16x128_f8f6f4 v[200:203], v[80:87], v[144:151], v[200:203] cbsz:1\n\t"
    "ds_read_b64_tr_b8 v[112:113], v62 offset:16384\n\t"
    "ds_read_b64_tr_b8 v[114:115], v62 offset:17408\n\t"
    "v_mfma_f32_16x16x128_f8f6f4 v[232:235], v[80:87], v[152:159], v[232:235] cbsz:1\n\t"
    "ds_read_b64_tr_b8 v[116:117], v62 offset:24576\n\t"
    "ds_read_b64_tr_b8 v[118:119], v62 offset:25600\n\t"
    "s_waitcnt lgkmcnt(12)\n\t"
    "v_mfma_f32_16x16x128_f8f6f4 v[204:207], v[88:95], v[144:151], v[204:207] cbsz:1\n\t"
    "ds_read_b64_tr_b8 v[120:121], v63 offset:16384\n\t"
    "ds_read_b64_tr_b8 v[122:123], v63 offset:17408\n\t"
    "s_waitcnt vmcnt(4)\n\t"
    "ds_read_b128 v[160:163], v47 offset:32768\n\t"
    "ds_read_b128 v[164:167], v47 offset:32832\n\t"
    "ds_read_b128 v[168:171], v47 offset:49152\n\t"
    "ds_read_b128 v[172:175], v47 offset:49216\n\t"
    "s_add_u32 s76, s76, 4096\n\t"
    "s_add_u32 m0, %[m0w], 99328\n\t"
    "s_nop 0\n\t"
    "buffer_load_dwordx4 %[xovoff], %[srdXin], s76 offen lds\n\t"
    "v_mfma_f32_16x16x128_f8f6f4 v[236:239], v[88:95], v[152:159], v[236:239] cbsz:1\n\t"
    "ds_read_b64_tr_b8 v[124:125], v63 offset:24576\n\t"
    "ds_read_b64_tr_b8 v[126:127], v63 offset:25600\n\t"
    "s_waitcnt vmcnt(13)\n\t"
    "s_barrier\n\t"
    "s_waitcnt lgkmcnt(12)\n\t"
    "v_mfma_f32_16x16x128_f8f6f4 v[208:211], v[96:103], v[144:151], v[208:211] cbsz:1\n\t"
    "ds_read_b64_tr_b8 v[64:65], v56 offset:32768\n\t"
    "ds_read_b64_tr_b8 v[66:67], v56 offset:33792\n\t"
    "s_add_u32 s76, s76, 4096\n\t"
    "s_add_u32 m0, %[m0w], 114688\n\t"
    "s_nop 0\n\t"
    "buffer_load_dwordx4 %[xovoff], %[srdXin], s76 offen lds\n\t"
    "v_mfma_f32_16x16x128_f8f6f4 v[240:243], v[96:103], v[152:159], v[240:243] cbsz:1\n\t"
    "ds_read_b64_tr_b8 v[68:69], v56 offset:40960\n\t"
    "ds_read_b64_tr_b8 v[70:71], v56 offset:41984\n\t"
    "s_waitcnt lgkmcnt(12)\n\t"
    "v_mfma_f32_16x16x128_f8f6f4 v[212:215], v[104:111], v[144:151], v[212:215] cbsz:1\n\t"
    "ds_read_b64_tr_b8 v[72:73], v57 offset:32768\n\t"
    "ds_read_b64_tr_b8 v[74:75], v57 offset:33792\n\t"
    "v_mfma_f32_16x16x128_f8f6f4 v[244:247], v[104:111], v[152:159], v[244:247] cbsz:1\n\t"
    "ds_read_b64_tr_b8 v[76:77], v57 offset:40960\n\t"
    "ds_read_b64_tr_b8 v[78:79], v57 offset:41984\n\t"
    "s_add_u32 s76, s76, 4096\n\t"
    "s_add_u32 m0, %[m0w], 115712\n\t"
    "s_nop 0\n\t"
    "buffer_load_dwordx4 %[xovoff], %[srdXin], s76 offen lds\n\t"
    "s_waitcnt lgkmcnt(12)\n\t"
    "v_mfma_f32_16x16x128_f8f6f4 v[216:219], v[112:119], v[144:151], v[216:219] cbsz:1\n\t"
    "ds_read_b64_tr_b8 v[80:81], v58 offset:32768\n\t"
    "ds_read_b64_tr_b8 v[82:83], v58 offset:33792\n\t"
    "v_mfma_f32_16x16x128_f8f6f4 v[248:251], v[112:119], v[152:159], v[248:251] cbsz:1\n\t"
    "ds_read_b64_tr_b8 v[84:85], v58 offset:40960\n\t"
    "ds_read_b64_tr_b8 v[86:87], v58 offset:41984\n\t"
    "v_readlane_b32 s77, %[tw], 5\n\t"
    "s_mov_b32 s73, s77\n\t"
    "s_add_u32 m0, %[m0w], 81920\n\t"
    "s_nop 0\n\t"
    "buffer_load_dwordx4 %[wvoff], %[srdW], s73 offen lds\n\t"
    "s_waitcnt lgkmcnt(12)\n\t"
    "v_mfma_f32_16x16x128_f8f6f4 v[220:223], v[120:127], v[144:151], v[220:223] cbsz:1\n\t"
    "ds_read_b64_tr_b8 v[88:89], v59 offset:32768\n\t"
    "ds_read_b64_tr_b8 v[90:91], v59 offset:33792\n\t"
    "v_mfma_f32_16x16x128_f8f6f4 v[252:255], v[120:127], v[152:159], v[252:255] cbsz:1\n\t"
    "ds_read_b64_tr_b8 v[92:93], v59 offset:40960\n\t"
    "ds_read_b64_tr_b8 v[94:95], v59 offset:41984\n\t"
    "s_waitcnt lgkmcnt(12)\n\t"
    "v_mfma_f32_16x16x128_f8f6f4 v[192:195], v[64:71], v[160:167], v[192:195] cbsz:1\n\t"
    "ds_read_b64_tr_b8 v[96:97], v60 offset:32768\n\t"
    "ds_read_b64_tr_b8 v[98:99], v60 offset:33792\n\t"
    "s_add_u32 m0, %[m0w], 82944\n\t"
    "s_add_u32 s73, s73, 131072\n\t"
    "buffer_load_dwordx4 %[wvoff], %[srdW], s73 offen lds\n\t"
    "v_mfma_f32_16x16x128_f8f6f4 v[224:227], v[64:71], v[168:175], v[224:227] cbsz:1\n\t"
    "ds_read_b64_tr_b8 v[100:101], v60 offset:40960\n\t"
    "ds_read_b64_tr_b8 v[102:103], v60 offset:41984\n\t"
    "s_waitcnt lgkmcnt(12)\n\t"
    "v_mfma_f32_16x16x128_f8f6f4 v[196:199], v[72:79], v[160:167], v[196:199] cbsz:1\n\t"
    "ds_read_b64_tr_b8 v[104:105], v61 offset:32768\n\t"
    "ds_read_b64_tr_b8 v[106:107], v61 offset:33792\n\t"
    "v_readlane_b32 s77, %[tx], 4\n\t"
    "s_mov_b32 s75, s77\n\t"
    "buffer_load_dwordx4 v[128:131], %[xvoff], s[60:63], s75 offen offset:0 sc0 sc1\n\t"
    "v_mfma_f32_16x16x128_f8f6f4 v[228:231], v[72:79], v[168:175], v[228:231] cbsz:1\n\t"
    "ds_read_b64_tr_b8 v[108:109], v61 offset:40960\n\t"
    "ds_read_b64_tr_b8 v[110:111], v61 offset:41984\n\t"
    "s_cmp_eq_u32 %[pushstep], 3\n\t"
    "s_cbranch_scc0 L_cxp3_%=\n\t"
    "buffer_load_dwordx4 v[48:51], %[ovoff], %[srdXin], %[xpoff] offen\n\t"
    "s_branch L_dxp3_%=\n\t"
    "L_cxp3_%=:\n\t"
    "buffer_load_dword v46, v55, %[srdM], s79 offen\n\t"
    "L_dxp3_%=:\n\t"
    "s_waitcnt lgkmcnt(12)\n\t"
    "v_mfma_f32_16x16x128_f8f6f4 v[200:203], v[80:87], v[160:167], v[200:203] cbsz:1\n\t"
    "ds_read_b64_tr_b8 v[112:113], v62 offset:32768\n\t"
    "ds_read_b64_tr_b8 v[114:115], v62 offset:33792\n\t"
    "v_mfma_f32_16x16x128_f8f6f4 v[232:235], v[80:87], v[168:175], v[232:235] cbsz:1\n\t"
    "ds_read_b64_tr_b8 v[116:117], v62 offset:40960\n\t"
    "ds_read_b64_tr_b8 v[118:119], v62 offset:41984\n\t"
    "buffer_load_dwordx4 v[132:135], %[xvoff], s[60:63], s75 offen offset:1024 sc0 sc1\n\t"
    "s_waitcnt lgkmcnt(12)\n\t"
    "v_mfma_f32_16x16x128_f8f6f4 v[204:207], v[88:95], v[160:167], v[204:207] cbsz:1\n\t"
    "ds_read_b64_tr_b8 v[120:121], v63 offset:32768\n\t"
    "ds_read_b64_tr_b8 v[122:123], v63 offset:33792\n\t"
    "s_waitcnt vmcnt(5)\n\t"
    "ds_read_b128 v[176:179], v47 offset:0\n\t"
    "ds_read_b128 v[180:183], v47 offset:64\n\t"
    "ds_read_b128 v[184:187], v47 offset:16384\n\t"
    "ds_read_b128 v[188:191], v47 offset:16448\n\t"
    "v_mfma_f32_16x16x128_f8f6f4 v[236:239], v[88:95], v[168:175], v[236:239] cbsz:1\n\t"
    "ds_read_b64_tr_b8 v[124:125], v63 offset:40960\n\t"
    "ds_read_b64_tr_b8 v[126:127], v63 offset:41984\n\t"
    "buffer_load_dwordx4 v[136:139], %[xvoff], s[60:63], s75 offen offset:2048 sc0 sc1\n\t"
    "s_cmp_eq_u32 %[pushstep], 2\n\t"
    "s_cbranch_scc0 L_cnp2_%=\n\t"
    "s_waitcnt vmcnt(9)\n\t"
    "v_mov_b32 v54, 0x80\n\t"
    "v_cmp_eq_u32 vcc, 0, v48\n\t"
    "s_nop 1\n\t"
    "v_cndmask_b32 v48, v48, v54, vcc\n\t"
    "v_cmp_eq_u32 vcc, 0, v49\n\t"
    "s_nop 1\n\t"
    "v_cndmask_b32 v49, v49, v54, vcc\n\t"
    "v_cmp_eq_u32 vcc, 0, v50\n\t"
    "s_nop 1\n\t"
    "v_cndmask_b32 v50, v50, v54, vcc\n\t"
    "v_cmp_eq_u32 vcc, 0, v51\n\t"
    "s_nop 1\n\t"
    "v_cndmask_b32 v51, v51, v54, vcc\n\t"
    "buffer_store_dwordx4 v[48:51], %[pvst], s[64:67], 0 offen sc0 sc1\n\t"
    "s_branch L_dnp2_%=\n\t"
    "L_cnp2_%=:\n\t"
    "buffer_load_dword v46, v55, %[srdM], s79 offen\n\t"
    "L_dnp2_%=:\n\t"
    "s_waitcnt vmcnt(19)\n\t"
    "s_barrier\n\t"
    "s_waitcnt lgkmcnt(12)\n\t"
    "v_mfma_f32_16x16x128_f8f6f4 v[208:211], v[96:103], v[160:167], v[208:211] cbsz:1\n\t"
    "ds_read_b64_tr_b8 v[64:65], v56 offset:49152\n\t"
    "ds_read_b64_tr_b8 v[66:67], v56 offset:50176\n\t"
    "v_mfma_f32_16x16x128_f8f6f4 v[240:243], v[96:103], v[168:175], v[240:243] cbsz:1\n\t"
    "ds_read_b64_tr_b8 v[68:69], v56 offset:57344\n\t"
    "ds_read_b64_tr_b8 v[70:71], v56 offset:58368\n\t"
    "s_cmp_eq_u32 %[pfw], 2\n\t"
    "s_cbranch_scc0 L_cpf2_%=\n\t"
    "buffer_load_dword v46, %[pfvoff], %[srdPF], %[pfso] offen sc1\n\t"
    "s_branch L_dpf2_%=\n\t"
    "L_cpf2_%=:\n\t"
    "buffer_load_dword v46, v55, %[srdM], s79 offen\n\t"
    "L_dpf2_%=:\n\t"
    "s_waitcnt lgkmcnt(12)\n\t"
    "v_mfma_f32_16x16x128_f8f6f4 v[212:215], v[104:111], v[160:167], v[212:215] cbsz:1\n\t"
    "ds_read_b64_tr_b8 v[72:73], v57 offset:49152\n\t"
    "ds_read_b64_tr_b8 v[74:75], v57 offset:50176\n\t"
    "buffer_load_dwordx4 v[140:143], %[xvoff], s[60:63], s75 offen offset:3072 sc0 sc1\n\t"
    "v_mfma_f32_16x16x128_f8f6f4 v[244:247], v[104:111], v[168:175], v[244:247] cbsz:1\n\t"
    "ds_read_b64_tr_b8 v[76:77], v57 offset:57344\n\t"
    "ds_read_b64_tr_b8 v[78:79], v57 offset:58368\n\t"
    "s_waitcnt lgkmcnt(12)\n\t"
    "v_mfma_f32_16x16x128_f8f6f4 v[216:219], v[112:119], v[160:167], v[216:219] cbsz:1\n\t"
    "ds_read_b64_tr_b8 v[80:81], v58 offset:49152\n\t"
    "ds_read_b64_tr_b8 v[82:83], v58 offset:50176\n\t"
    "v_mfma_f32_16x16x128_f8f6f4 v[248:251], v[112:119], v[168:175], v[248:251] cbsz:1\n\t"
    "ds_read_b64_tr_b8 v[84:85], v58 offset:57344\n\t"
    "ds_read_b64_tr_b8 v[86:87], v58 offset:58368\n\t"
    "s_waitcnt lgkmcnt(12)\n\t"
    "v_mfma_f32_16x16x128_f8f6f4 v[220:223], v[120:127], v[160:167], v[220:223] cbsz:1\n\t"
    "ds_read_b64_tr_b8 v[88:89], v59 offset:49152\n\t"
    "ds_read_b64_tr_b8 v[90:91], v59 offset:50176\n\t"
    "v_mfma_f32_16x16x128_f8f6f4 v[252:255], v[120:127], v[168:175], v[252:255] cbsz:1\n\t"
    "ds_read_b64_tr_b8 v[92:93], v59 offset:57344\n\t"
    "ds_read_b64_tr_b8 v[94:95], v59 offset:58368\n\t"
    "v_readlane_b32 s77, %[tw], 6\n\t"
    "s_mov_b32 s73, s77\n\t"
    "s_add_u32 m0, %[m0w], 98304\n\t"
    "s_nop 0\n\t"
    "buffer_load_dwordx4 %[wvoff], %[srdW], s73 offen lds\n\t"
    "s_waitcnt lgkmcnt(12)\n\t"
    "v_mfma_f32_16x16x128_f8f6f4 v[192:195], v[64:71], v[176:183], v[192:195] cbsz:1\n\t"
    "ds_read_b64_tr_b8 v[96:97], v60 offset:49152\n\t"
    "ds_read_b64_tr_b8 v[98:99], v60 offset:50176\n\t"
    "v_xor_b32 v56, 0x10000, v56\n\t"
    "v_mfma_f32_16x16x128_f8f6f4 v[224:227], v[64:71], v[184:191], v[224:227] cbsz:1\n\t"
    "ds_read_b64_tr_b8 v[100:101], v60 offset:57344\n\t"
    "ds_read_b64_tr_b8 v[102:103], v60 offset:58368\n\t"
    "s_add_u32 m0, %[m0w], 99328\n\t"
    "s_add_u32 s73, s73, 131072\n\t"
    "buffer_load_dwordx4 %[wvoff], %[srdW], s73 offen lds\n\t"
    "v_xor_b32 v57, 0x10000, v57\n\t"
    "s_waitcnt lgkmcnt(12)\n\t"
    "v_mfma_f32_16x16x128_f8f6f4 v[196:199], v[72:79], v[176:183], v[196:199] cbsz:1\n\t"
    "ds_read_b64_tr_b8 v[104:105], v61 offset:49152\n\t"
    "ds_read_b64_tr_b8 v[106:107], v61 offset:50176\n\t"
    "v_xor_b32 v58, 0x10000, v58\n\t"
    "v_mfma_f32_16x16x128_f8f6f4 v[228:231], v[72:79], v[184:191], v[228:231] cbsz:1\n\t"
    "ds_read_b64_tr_b8 v[108:109], v61 offset:57344\n\t"
    "ds_read_b64_tr_b8 v[110:111], v61 offset:58368\n\t"
    "v_readlane_b32 s77, %[tx], 5\n\t"
    "s_mov_b32 s75, s77\n\t"
    "buffer_load_dwordx4 v[144:147], %[xvoff], s[60:63], s75 offen offset:0 sc0 sc1\n\t"
    "v_xor_b32 v59, 0x10000, v59\n\t"
    "s_waitcnt lgkmcnt(12)\n\t"
    "v_mfma_f32_16x16x128_f8f6f4 v[200:203], v[80:87], v[176:183], v[200:203] cbsz:1\n\t"
    "ds_read_b64_tr_b8 v[112:113], v62 offset:49152\n\t"
    "ds_read_b64_tr_b8 v[114:115], v62 offset:50176\n\t"
    "v_mfma_f32_16x16x128_f8f6f4 v[232:235], v[80:87], v[184:191], v[232:235] cbsz:1\n\t"
    "ds_read_b64_tr_b8 v[116:117], v62 offset:57344\n\t"
    "ds_read_b64_tr_b8 v[118:119], v62 offset:58368\n\t"
    "s_waitcnt lgkmcnt(12)\n\t"
    "v_mfma_f32_16x16x128_f8f6f4 v[204:207], v[88:95], v[176:183], v[204:207] cbsz:1\n\t"
    "ds_read_b64_tr_b8 v[120:121], v63 offset:49152\n\t"
    "ds_read_b64_tr_b8 v[122:123], v63 offset:50176\n\t"
    "buffer_load_dwordx4 v[148:151], %[xvoff], s[60:63], s75 offen offset:1024 sc0 sc1\n\t"
    "v_mfma_f32_16x16x128_f8f6f4 v[236:239], v[88:95], v[184:191], v[236:239] cbsz:1\n\t"
    "ds_read_b64_tr_b8 v[124:125], v63 offset:57344\n\t"
    "ds_read_b64_tr_b8 v[126:127], v63 offset:58368\n\t"
    "s_cmp_eq_u32 %[pushstep], 3\n\t"
    "s_cbranch_scc0 L_cnp3_%=\n\t"
    "s_waitcnt vmcnt(9)\n\t"
    "v_mov_b32 v54, 0x80\n\t"
    "v_cmp_eq_u32 vcc, 0, v48\n\t"
    "s_nop 1\n\t"
    "v_cndmask_b32 v48, v48, v54, vcc\n\t"
    "v_cmp_eq_u32 vcc, 0, v49\n\t"
    "s_nop 1\n\t"
    "v_cndmask_b32 v49, v49, v54, vcc\n\t"
    "v_cmp_eq_u32 vcc, 0, v50\n\t"
    "s_nop 1\n\t"
    "v_cndmask_b32 v50, v50, v54, vcc\n\t"
    "v_cmp_eq_u32 vcc, 0, v51\n\t"
    "s_nop 1\n\t"
    "v_cndmask_b32 v51, v51, v54, vcc\n\t"
    "buffer_store_dwordx4 v[48:51], %[pvst], s[64:67], 0 offen sc0 sc1\n\t"
    "s_branch L_dnp3_%=\n\t"
    "L_cnp3_%=:\n\t"
    "buffer_load_dword v46, v55, %[srdM], s79 offen\n\t"
    "L_dnp3_%=:\n\t"
    "s_waitcnt vmcnt(19)\n\t"
    "s_barrier\n\t"
    "s_waitcnt lgkmcnt(12)\n\t"
    "v_mfma_f32_16x16x128_f8f6f4 v[208:211], v[96:103], v[176:183], v[208:211] cbsz:1\n\t"
    "ds_read_b64_tr_b8 v[64:65], v56 offset:0\n\t"
    "ds_read_b64_tr_b8 v[66:67], v56 offset:1024\n\t"
    "buffer_load_dwordx4 v[152:155], %[xvoff], s[60:63], s75 offen offset:2048 sc0 sc1\n\t"
    "v_xor_b32 v60, 0x10000, v60\n\t"
    "v_mfma_f32_16x16x128_f8f6f4 v[240:243], v[96:103], v[184:191], v[240:243] cbsz:1\n\t"
    "ds_read_b64_tr_b8 v[68:69], v56 offset:8192\n\t"
    "ds_read_b64_tr_b8 v[70:71], v56 offset:9216\n\t"
    "v_xor_b32 v61, 0x10000, v61\n\t"
    "s_waitcnt lgkmcnt(12)\n\t"
    "v_mfma_f32_16x16x128_f8f6f4 v[212:215], v[104:111], v[176:183], v[212:215] cbsz:1\n\t"
    "ds_read_b64_tr_b8 v[72:73], v57 offset:0\n\t"
    "ds_read_b64_tr_b8 v[74:75], v57 offset:1024\n\t"
    "v_xor_b32 v62, 0x10000, v62\n\t"
    "v_mfma_f32_16x16x128_f8f6f4 v[244:247], v[104:111], v[184:191], v[244:247] cbsz:1\n\t"
    "ds_read_b64_tr_b8 v[76:77], v57 offset:8192\n\t"
    "ds_read_b64_tr_b8 v[78:79], v57 offset:9216\n\t"
    "buffer_load_dwordx4 v[156:159], %[xvoff], s[60:63], s75 offen offset:3072 sc0 sc1\n\t"
    "v_xor_b32 v63, 0x10000, v63\n\t"
    "s_waitcnt lgkmcnt(12)\n\t"
    "v_mfma_f32_16x16x128_f8f6f4 v[216:219], v[112:119], v[176:183], v[216:219] cbsz:1\n\t"
    "ds_read_b64_tr_b8 v[80:81], v58 offset:0\n\t"
    "ds_read_b64_tr_b8 v[82:83], v58 offset:1024\n\t"
    "v_mfma_f32_16x16x128_f8f6f4 v[248:251], v[112:119], v[184:191], v[248:251] cbsz:1\n\t"
    "ds_read_b64_tr_b8 v[84:85], v58 offset:8192\n\t"
    "ds_read_b64_tr_b8 v[86:87], v58 offset:9216\n\t"
    "v_readlane_b32 s77, %[tw], 7\n\t"
    "s_mov_b32 s73, s77\n\t"
    "s_add_u32 m0, %[m0w], 114688\n\t"
    "s_nop 0\n\t"
    "buffer_load_dwordx4 %[wvoff], %[srdW], s73 offen lds\n\t"
    "s_waitcnt vmcnt(8)\n\t"
    "v_min3_u32 v53, v128, v129, v130\n\t"
    "v_min3_u32 v54, v131, v132, v133\n\t"
    "v_min3_u32 v53, v53, v134, v135\n\t"
    "s_waitcnt lgkmcnt(12)\n\t"
    "v_mfma_f32_16x16x128_f8f6f4 v[220:223], v[120:127], v[176:183], v[220:223] cbsz:1\n\t"
    "ds_read_b64_tr_b8 v[88:89], v59 offset:0\n\t"
    "ds_read_b64_tr_b8 v[90:91], v59 offset:1024\n\t"
    "v_min3_u32 v54, v54, v136, v137\n\t"
    "v_min3_u32 v53, v53, v138, v139\n\t"
    "v_min3_u32 v54, v54, v140, v141\n\t"
    "v_mfma_f32_16x16x128_f8f6f4 v[252:255], v[120:127], v[184:191], v[252:255] cbsz:1\n\t"
    "ds_read_b64_tr_b8 v[92:93], v59 offset:8192\n\t"
    "ds_read_b64_tr_b8 v[94:95], v59 offset:9216\n\t"
    "v_min3_u32 v53, v53, v142, v143\n\t"
    "v_min_u32 v53, v53, v54\n\t"
    "v_cmp_eq_u32 vcc, 0, v53\n\t"
    "s_cbranch_vccz L_ok4_%=\n\t"
    "v_readlane_b32 s77, %[tx], 4\n\t"
    "s_mov_b32 s82, s77\n\t"
    "s_getpc_b64 s[58:59]\n\t"
    "s_branch L_slow0_%=\n\t"
    "L_ok4_%=:\n\t"
    "s_waitcnt lgkmcnt(12)\n\t"
    "v_mfma_f32_16x16x128_f8f6f4 v[192:195], v[64:71], v[128:135], v[192:195] cbsz:1\n\t"
    "ds_read_b64_tr_b8 v[96:97], v60 offset:0\n\t"
    "ds_read_b64_tr_b8 v[98:99], v60 offset:1024\n\t"
    "s_add_u32 m0, %[m0w], 115712\n\t"
    "s_add_u32 s73, s73, 131072\n\t"
    "buffer_load_dwordx4 %[wvoff], %[srdW], s73 offen lds\n\t"
    "v_mfma_f32_16x16x128_f8f6f4 v[224:227], v[64:71], v[136:143], v[224:227] cbsz:1\n\t"
    "ds_read_b64_tr_b8 v[100:101], v60 offset:8192\n\t"
    "ds_read_b64_tr_b8 v[102:103], v60 offset:9216\n\t"
    "s_waitcnt lgkmcnt(12)\n\t"
    "v_mfma_f32_16x16x128_f8f6f4 v[196:199], v[72:79], v[128:135], v[196:199] cbsz:1\n\t"
    "ds_read_b64_tr_b8 v[104:105], v61 offset:0\n\t"
    "ds_read_b64_tr_b8 v[106:107], v61 offset:1024\n\t"
    "v_readlane_b32 s77, %[tx], 6\n\t"
    "s_mov_b32 s75, s77\n\t"
    "buffer_load_dwordx4 v[160:163], %[xvoff], s[60:63], s75 offen offset:0 sc0 sc1\n\t"
    "v_mfma_f32_16x16x128_f8f6f4 v[228:231], v[72:79], v[136:143], v[228:231] cbsz:1\n\t"
    "ds_read_b64_tr_b8 v[108:109], v61 offset:8192\n\t"
    "ds_read_b64_tr_b8 v[110:111], v61 offset:9216\n\t"
    "s_cmp_eq_u32 %[pushstep], 5\n\t"
    "s_cbranch_scc0 L_cxp5_%=\n\t"
    "buffer_load_dwordx4 v[48:51], %[ovoff], %[srdXin], %[xpoff] offen\n\t"
    "s_branch L_dxp5_%=\n\t"
    "L_cxp5_%=:\n\t"
    "buffer_load_dword v46, v55, %[srdM], s79 offen\n\t"
    "L_dxp5_%=:\n\t"
    "s_waitcnt lgkmcnt(12)\n\t"
    "v_mfma_f32_16x16x128_f8f6f4 v[200:203], v[80:87], v[128:135], v[200:203] cbsz:1\n\t"
    "ds_read_b64_tr_b8 v[112:113], v62 offset:0\n\t"
    "ds_read_b64_tr_b8 v[114:115], v62 offset:1024\n\t"
    "v_mfma_f32_16x16x128_f8f6f4 v[232:235], v[80:87], v[136:143], v[232:235] cbsz:1\n\t"
    "ds_read_b64_tr_b8 v[116:117], v62 offset:8192\n\t"
    "ds_read_b64_tr_b8 v[118:119], v62 offset:9216\n\t"
    "buffer_load_dwordx4 v[164:167], %[xvoff], s[60:63], s75 offen offset:1024 sc0 sc1\n\t"
    "s_waitcnt lgkmcnt(12)\n\t"
    "v_mfma_f32_16x16x128_f8f6f4 v[204:207], v[88:95], v[128:135], v[204:207] cbsz:1\n\t"
    "ds_read_b64_tr_b8 v[120:121], v63 offset:0\n\t"
    "ds_read_b64_tr_b8 v[122:123], v63 offset:1024\n\t"
    "v_mfma_f32_16x16x128_f8f6f4 v[236:239], v[88:95], v[136:143], v[236:239] cbsz:1\n\t"
    "ds_read_b64_tr_b8 v[124:125], v63 offset:8192\n\t"
    "ds_read_b64_tr_b8 v[126:127], v63 offset:9216\n\t"
    "buffer_load_dwordx4 v[168:171], %[xvoff], s[60:63], s75 offen offset:2048 sc0 sc1\n\t"
    "s_waitcnt vmcnt(20)\n\t"
    "s_barrier\n\t"
    "s_waitcnt lgkmcnt(12)\n\t"
    "v_mfma_f32_16x16x128_f8f6f4 v[208:211], v[96:103], v[128:135], v[208:211] cbsz:1\n\t"
    "ds_read_b64_tr_b8 v[64:65], v56 offset:16384\n\t"
    "ds_read_b64_tr_b8 v[66:67], v56 offset:17408\n\t"
    "v_mfma_f32_16x16x128_f8f6f4 v[240:243], v[96:103], v[136:143], v[240:243] cbsz:1\n\t"
    "ds_read_b64_tr_b8 v[68:69], v56 offset:24576\n\t"
    "ds_read_b64_tr_b8 v[70:71], v56 offset:25600\n\t"
    "s_cmp_eq_u32 %[pfw], 3\n\t"
    "s_cbranch_scc0 L_cpf3_%=\n\t"
    "buffer_load_dword v46, %[pfvoff], %[srdPF], %[pfso] offen sc1\n\t"
    "s_branch L_dpf3_%=\n\t"
    "L_cpf3_%=:\n\t"
    "buffer_load_dword v46, v55, %[srdM], s79 offen\n\t"
    "L_dpf3_%=:\n\t"
    "s_waitcnt lgkmcnt(12)\n\t"
    "v_mfma_f32_16x16x128_f8f6f4 v[212:215], v[104:111], v[128:135], v[212:215] cbsz:1\n\t"
    "ds_read_b64_tr_b8 v[72:73], v57 offset:16384\n\t"
    "ds_read_b64_tr_b8 v[74:75], v57 offset:17408\n\t"
    "buffer_load_dwordx4 v[172:175], %[xvoff], s[60:63], s75 offen offset:3072 sc0 sc1\n\t"
    "v_mfma_f32_16x16x128_f8f6f4 v[244:247], v[104:111], v[136:143], v[244:247] cbsz:1\n\t"
    "ds_read_b64_tr_b8 v[76:77], v57 offset:24576\n\t"
    "ds_read_b64_tr_b8 v[78:79], v57 offset:25600\n\t"
    "s_waitcnt vmcnt(8)\n\t"
    "v_min3_u32 v53, v144, v145, v146\n\t"
    "v_min3_u32 v54, v147, v148, v149\n\t"
    "s_waitcnt lgkmcnt(12)\n\t"
    "v_mfma_f32_16x16x128_f8f6f4 v[216:219], v[112:119], v[128:135], v[216:219] cbsz:1\n\t"
    "ds_read_b64_tr_b8 v[80:81], v58 offset:16384\n\t"
    "ds_read_b64_tr_b8 v[82:83], v58 offset:17408\n\t"
    "v_readlane_b32 s77, %[tw], 8\n\t"
    "s_mov_b32 s73, s77\n\t"
    "s_add_u32 m0, %[m0w], 0\n\t"
    "s_nop 0\n\t"
    "buffer_load_dwordx4 %[wvoff], %[srdW], s73 offen lds\n\t"
    "v_min3_u32 v53, v53, v150, v151\n\t"
    "v_min3_u32 v54, v54, v152, v153\n\t"
    "v_mfma_f32_16x16x128_f8f6f4 v[248:251], v[112:119], v[136:143], v[248:251] cbsz:1\n\t"
    "ds_read_b64_tr_b8 v[84:85], v58 offset:24576\n\t"
    "ds_read_b64_tr_b8 v[86:87], v58 offset:25600\n\t"
    "v_min3_u32 v53, v53, v154, v155\n\t"
    "v_min3_u32 v54, v54, v156, v157\n\t"
    "s_waitcnt lgkmcnt(12)\n\t"
    "v_mfma_f32_16x16x128_f8f6f4 v[220:223], v[120:127], v[128:135], v[220:223] cbsz:1\n\t"
    "ds_read_b64_tr_b8 v[88:89], v59 offset:16384\n\t"
    "ds_read_b64_tr_b8 v[90:91], v59 offset:17408\n\t"
    "v_min3_u32 v53, v53, v158, v159\n\t"
    "v_min_u32 v53, v53, v54\n\t"
    "v_mfma_f32_16x16x128_f8f6f4 v[252:255], v[120:127], v[136:143], v[252:255] cbsz:1\n\t"
    "ds_read_b64_tr_b8 v[92:93], v59 offset:24576\n\t"
    "ds_read_b64_tr_b8 v[94:95], v59 offset:25600\n\t"
    "s_add_u32 m0, %[m0w], 1024\n\t"
    "s_add_u32 s73, s73, 131072\n\t"
    "buffer_load_dwordx4 %[wvoff], %[srdW], s73 offen lds\n\t"
    "v_cmp_eq_u32 vcc, 0, v53\n\t"
    "s_cbranch_vccz L_ok5_%=\n\t"
    "v_readlane_b32 s77, %[tx], 5\n\t"
    "s_mov_b32 s82, s77\n\t"
    "s_getpc_b64 s[58:59]\n\t"
    "s_branch L_slow1_%=\n\t"
    "L_ok5_%=:\n\t"
    "s_waitcnt lgkmcnt(12)\n\t"
    "v_mfma_f32_16x16x128_f8f6f4 v[192:195], v[64:71], v[144:151], v[192:195] cbsz:1\n\t"
    "ds_read_b64_tr_b8 v[96:97], v60 offset:16384\n\t"
    "ds_read_b64_tr_b8 v[98:99], v60 offset:17408\n\t"
    "v_mfma_f32_16x16x128_f8f6f4 v[224:227], v[64:71], v[152:159], v[224:227] cbsz:1\n\t"
    "ds_read_b64_tr_b8 v[100:101], v60 offset:24576\n\t"
    "ds_read_b64_tr_b8 v[102:103], v60 offset:25600\n\t"
    "v_readlane_b32 s77, %[tx], 7\n\t"
    "s_mov_b32 s75, s77\n\t"
    "buffer_load_dwordx4 v[176:179], %[xvoff], s[60:63], s75 offen offset:0 sc0 sc1\n\t"
    "s_waitcnt lgkmcnt(12)\n\t"
    "v_mfma_f32_16x16x128_f8f6f4 v[196:199], v[72:79], v[144:151], v[196:199] cbsz:1\n\t"
    "ds_read_b64_tr_b8 v[104:105], v61 offset:16384\n\t"
    "ds_read_b64_tr_b8 v[106:107], v61 offset:17408\n\t"
    "v_mfma_f32_16x16x128_f8f6f4 v[228:231], v[72:79], v[152:159], v[228:231] cbsz:1\n\t"
    "ds_read_b64_tr_b8 v[108:109], v61 offset:24576\n\t"
    "ds_read_b64_tr_b8 v[110:111], v61 offset:25600\n\t"
    "s_cmp_eq_u32 %[pushstep], 6\n\t"
    "s_cbranch_scc0 L_cxp6_%=\n\t"
    "buffer_load_dwordx4 v[48:51], %[ovoff], %[srdXin], %[xpoff] offen\n\t"
    "s_branch L_dxp6_%=\n\t"
    "L_cxp6_%=:\n\t"
    "buffer_load_dword v46, v55, %[srdM], s79 offen\n\t"
    "L_dxp6_%=:\n\t"
    "s_waitcnt lgkmcnt(12)\n\t"
    "v_mfma_f32_16x16x128_f8f6f4 v[200:203], v[80:87], v[144:151], v[200:203] cbsz:1\n\t"
    "ds_read_b64_tr_b8 v[112:113], v62 offset:16384\n\t"
    "ds_read_b64_tr_b8 v[114:115], v62 offset:17408\n\t"
    "buffer_load_dwordx4 v[180:183], %[xvoff], s[60:63], s75 offen offset:1024 sc0 sc1\n\t"
    "v_mfma_f32_16x16x128_f8f6f4 v[232:235], v[80:87], v[152:159], v[232:235] cbsz:1\n\t"
    "ds_read_b64_tr_b8 v[116:117], v62 offset:24576\n\t"
    "ds_read_b64_tr_b8 v[118:119], v62 offset:25600\n\t"
    "s_waitcnt lgkmcnt(12)\n\t"
    "v_mfma_f32_16x16x128_f8f6f4 v[204:207], v[88:95], v[144:151], v[204:207] cbsz:1\n\t"
    "ds_read_b64_tr_b8 v[120:121], v63 offset:16384\n\t"
    "ds_read_b64_tr_b8 v[122:123], v63 offset:17408\n\t"
    "buffer_load_dwordx4 v[184:187], %[xvoff], s[60:63], s75 offen offset:2048 sc0 sc1\n\t"
    "v_mfma_f32_16x16x128_f8f6f4 v[236:239], v[88:95], v[152:159], v[236:239] cbsz:1\n\t"
    "ds_read_b64_tr_b8 v[124:125], v63 offset:24576\n\t"
    "ds_read_b64_tr_b8 v[126:127], v63 offset:25600\n\t"
    "s_cmp_eq_u32 %[pushstep], 5\n\t"
    "s_cbranch_scc0 L_cnp5_%=\n\t"
    "s_waitcnt vmcnt(10)\n\t"
    "v_mov_b32 v54, 0x80\n\t"
    "v_cmp_eq_u32 vcc, 0, v48\n\t"
    "s_nop 1\n\t"
    "v_cndmask_b32 v48, v48, v54, vcc\n\t"
    "v_cmp_eq_u32 vcc, 0, v49\n\t"
    "s_nop 1\n\t"
    "v_cndmask_b32 v49, v49, v54, vcc\n\t"
    "v_cmp_eq_u32 vcc, 0, v50\n\t"
    "s_nop 1\n\t"
    "v_cndmask_b32 v50, v50, v54, vcc\n\t"
    "v_cmp_eq_u32 vcc, 0, v51\n\t"
    "s_nop 1\n\t"
    "v_cndmask_b32 v51, v51, v54, vcc\n\t"
    "buffer_store_dwordx4 v[48:51], %[pvst], s[64:67], 0 offen sc0 sc1\n\t"
    "s_branch L_dnp5_%=\n\t"
    "L_cnp5_%=:\n\t"
    "buffer_load_dword v46, v55, %[srdM], s79 offen\n\t"
    "L_dnp5_%=:\n\t"
    "s_waitcnt vmcnt(20)\n\t"
    "s_barrier\n\t"
    "s_waitcnt lgkmcnt(12)\n\t"
    "v_mfma_f32_16x16x128_f8f6f4 v[208:211], v[96:103], v[144:151], v[208:211] cbsz:1\n\t"
    "ds_read_b64_tr_b8 v[64:65], v56 offset:32768\n\t"
    "ds_read_b64_tr_b8 v[66:67], v56 offset:33792\n\t"
    "v_mfma_f32_16x16x128_f8f6f4 v[240:243], v[96:103], v[152:159], v[240:243] cbsz:1\n\t"
    "ds_read_b64_tr_b8 v[68:69], v56 offset:40960\n\t"
    "ds_read_b64_tr_b8 v[70:71], v56 offset:41984\n\t"
    "buffer_load_dwordx4 v[188:191], %[xvoff], s[60:63], s75 offen offset:3072 sc0 sc1\n\t"
    "s_waitcnt lgkmcnt(12)\n\t"
    "v_mfma_f32_16x16x128_f8f6f4 v[212:215], v[104:111], v[144:151], v[212:215] cbsz:1\n\t"
    "ds_read_b64_tr_b8 v[72:73], v57 offset:32768\n\t"
    "ds_read_b64_tr_b8 v[74:75], v57 offset:33792\n\t"
    "v_mfma_f32_16x16x128_f8f6f4 v[244:247], v[104:111], v[152:159], v[244:247] cbsz:1\n\t"
    "ds_read_b64_tr_b8 v[76:77], v57 offset:40960\n\t"
    "ds_read_b64_tr_b8 v[78:79], v57 offset:41984\n\t"
    "v_readlane_b32 s77, %[tw], 9\n\t"
    "s_mov_b32 s73, s77\n\t"
    "s_add_u32 m0, %[m0w], 16384\n\t"
    "s_nop 0\n\t"
    "buffer_load_dwordx4 %[wvoff], %[srdW], s73 offen lds\n\t"
    "s_waitcnt vmcnt(9)\n\t"
    "v_min3_u32 v53, v160, v161, v162\n\t"
    "v_min3_u32 v54, v163, v164, v165\n\t"
    "s_waitcnt lgkmcnt(12)\n\t"
    "v_mfma_f32_16x16x128_f8f6f4 v[216:219], v[112:119], v[144:151], v[216:219] cbsz:1\n\t"
    "ds_read_b64_tr_b8 v[80:81], v58 offset:32768\n\t"
    "ds_read_b64_tr_b8 v[82:83], v58 offset:33792\n\t"
    "v_min3_u32 v53, v53, v166, v167\n\t"
    "v_min3_u32 v54, v54, v168, v169\n\t"
    "v_mfma_f32_16x16x128_f8f6f4 v[248:251], v[112:119], v[152:159], v[248:251] cbsz:1\n\t"
    "ds_read_b64_tr_b8 v[84:85], v58 offset:40960\n\t"
    "ds_read_b64_tr_b8 v[86:87], v58 offset:41984\n\t"
    "v_min3_u32 v53, v53, v170, v171\n\t"
    "v_min3_u32 v54, v54, v172, v173\n\t"
    "s_waitcnt lgkmcnt(12)\n\t"
    "v_mfma_f32_16x16x128_f8f6f4 v[220:223], v[120:127], v[144:151], v[220:223] cbsz:1\n\t"
    "ds_read_b64_tr_b8 v[88:89], v59 offset:32768\n\t"
    "ds_read_b64_tr_b8 v[90:91], v59 offset:33792\n\t"
    "s_add_u32 m0, %[m0w], 17408\n\t"
    "s_add_u32 s73, s73, 131072\n\t"
    "buffer_load_dwordx4 %[wvoff], %[srdW], s73 offen lds\n\t"
    "v_min3_u32 v53, v53, v174, v175\n\t"
    "v_min_u32 v53, v53, v54\n\t"
    "v_mfma_f32_16x16x128_f8f6f4 v[252:255], v[120:127], v[152:159], v[252:255] cbsz:1\n\t"
    "ds_read_b64_tr_b8 v[92:93], v59 offset:40960\n\t"
    "ds_read_b64_tr_b8 v[94:95], v59 offset:41984\n\t"
    "v_cmp_eq_u32 vcc, 0, v53\n\t"
    "s_cbranch_vccz L_ok6_%=\n\t"
    "v_readlane_b32 s77, %[tx], 6\n\t"
    "s_mov_b32 s82, s77\n\t"
    "s_getpc_b64 s[58:59]\n\t"
    "s_branch L_slow2_%=\n\t"
    "L_ok6_%=:\n\t"
    "s_waitcnt lgkmcnt(12)\n\t"
    "v_mfma_f32_16x16x128_f8f6f4 v[192:195], v[64:71], v[160:167], v[192:195] cbsz:1\n\t"
    "ds_read_b64_tr_b8 v[96:97], v60 offset:32768\n\t"
    "ds_read_b64_tr_b8 v[98:99], v60 offset:33792\n\t"
    "v_readlane_b32 s77, %[tx], 8\n\t"
    "s_mov_b32 s75, s77\n\t"
    "buffer_load_dwordx4 v[128:131], %[xvoff], s[60:63], s75 offen offset:0 sc0 sc1\n\t"
    "v_mfma_f32_16x16x128_f8f6f4 v[224:227], v[64:71], v[168:175], v[224:227] cbsz:1\n\t"
    "ds_read_b64_tr_b8 v[100:101], v60 offset:40960\n\t"
    "ds_read_b64_tr_b8 v[102:103], v60 offset:41984\n\t"
    "s_waitcnt lgkmcnt(12)\n\t"
    "v_mfma_f32_16x16x128_f8f6f4 v[196:199], v[72:79], v[160:167], v[196:199] cbsz:1\n\t"
    "ds_read_b64_tr_b8 v[104:105], v61 offset:32768\n\t"
    "ds_read_b64_tr_b8 v[106:107], v61 offset:33792\n\t"
    "v_mfma_f32_16x16x128_f8f6f4 v[228:231], v[72:79], v[168:175], v[228:231] cbsz:1\n\t"
    "ds_read_b64_tr_b8 v[108:109], v61 offset:40960\n\t"
    "ds_read_b64_tr_b8 v[110:111], v61 offset:41984\n\t"
    "buffer_load_dwordx4 v[132:135], %[xvoff], s[60:63], s75 offen offset:1024 sc0 sc1\n\t"
    "s_waitcnt lgkmcnt(12)\n\t"
    "v_mfma_f32_16x16x128_f8f6f4 v[200:203], v[80:87], v[160:167], v[200:203] cbsz:1\n\t"
    "ds_read_b64_tr_b8 v[112:113], v62 offset:32768\n\t"
    "ds_read_b64_tr_b8 v[114:115], v62 offset:33792\n\t"
    "s_cmp_eq_u32 %[cpfw], 63\n\t"
    "s_cbranch_scc0 L_ccp1_%=\n\t"
    "s_mov_b32 s77, 8192\n\t"
    "buffer_load_dword v46, %[cvoff], s[84:87], s77 offen sc1\n\t"
    "s_branch L_dcp1_%=\n\t"
    "L_ccp1_%=:\n\t"
    "buffer_load_dword v46, v55, %[srdM], s79 offen\n\t"
    "L_dcp1_%=:\n\t"
    "v_mfma_f32_16x16x128_f8f6f4 v[232:235], v[80:87], v[168:175], v[232:235] cbsz:1\n\t"
    "ds_read_b64_tr_b8 v[116:117], v62 offset:40960\n\t"
    "ds_read_b64_tr_b8 v[118:119], v62 offset:41984\n\t"
    "buffer_load_dwordx4 v[136:139], %[xvoff], s[60:63], s75 offen offset:2048 sc0 sc1\n\t"
    "s_waitcnt lgkmcnt(12)\n\t"
    "v_mfma_f32_16x16x128_f8f6f4 v[204:207], v[88:95], v[160:167], v[204:207] cbsz:1\n\t"
    "ds_read_b64_tr_b8 v[120:121], v63 offset:32768\n\t"
    "ds_read_b64_tr_b8 v[122:123], v63 offset:33792\n\t"
    "v_mfma_f32_16x16x128_f8f6f4 v[236:239], v[88:95], v[168:175], v[236:239] cbsz:1\n\t"
    "ds_read_b64_tr_b8 v[124:125], v63 offset:40960\n\t"
    "ds_read_b64_tr_b8 v[126:127], v63 offset:41984\n\t"
    "s_cmp_eq_u32 %[pushstep], 6\n\t"
    "s_cbranch_scc0 L_cnp6_%=\n\t"
    "s_waitcnt vmcnt(10)\n\t"
    "v_mov_b32 v54, 0x80\n\t"
    "v_cmp_eq_u32 vcc, 0, v48\n\t"
    "s_nop 1\n\t"
    "v_cndmask_b32 v48, v48, v54, vcc\n\t"
    "v_cmp_eq_u32 vcc, 0, v49\n\t"
    "s_nop 1\n\t"
    "v_cndmask_b32 v49, v49, v54, vcc\n\t"
    "v_cmp_eq_u32 vcc, 0, v50\n\t"
    "s_nop 1\n\t"
    "v_cndmask_b32 v50, v50, v54, vcc\n\t"
    "v_cmp_eq_u32 vcc, 0, v51\n\t"
    "s_nop 1\n\t"
    "v_cndmask_b32 v51, v51, v54, vcc\n\t"
    "buffer_store_dwordx4 v[48:51], %[pvst], s[64:67], 0 offen sc0 sc1\n\t"
    "s_branch L_dnp6_%=\n\t"
    "L_cnp6_%=:\n\t"
    "buffer_load_dword v46, v55, %[srdM], s79 offen\n\t"
    "L_dnp6_%=:\n\t"
    "s_waitcnt vmcnt(21)\n\t"
    "s_barrier\n\t"
    "s_waitcnt lgkmcnt(12)\n\t"
    "v_mfma_f32_16x16x128_f8f6f4 v[208:211], v[96:103], v[160:167], v[208:211] cbsz:1\n\t"
    "ds_read_b64_tr_b8 v[64:65], v56 offset:49152\n\t"
    "ds_read_b64_tr_b8 v[66:67], v56 offset:50176\n\t"
    "buffer_load_dwordx4 v[140:143], %[xvoff], s[60:63], s75 offen offset:3072 sc0 sc1\n\t"
    "v_mfma_f32_16x16x128_f8f6f4 v[240:243], v[96:103], v[168:175], v[240:243] cbsz:1\n\t"
    "ds_read_b64_tr_b8 v[68:69], v56 offset:57344\n\t"
    "ds_read_b64_tr_b8 v[70:71], v56 offset:58368\n\t"
    "s_waitcnt lgkmcnt(12)\n\t"
    "v_mfma_f32_16x16x128_f8f6f4 v[212:215], v[104:111], v[160:167], v[212:215] cbsz:1\n\t"
    "ds_read_b64_tr_b8 v[72:73], v57 offset:49152\n\t"
    "ds_read_b64_tr_b8 v[74:75], v57 offset:50176\n\t"
    "v_readlane_b32 s77, %[tw], 10\n\t"
    "s_mov_b32 s73, s77\n\t"
    "s_add_u32 m0, %[m0w], 32768\n\t"
    "s_nop 0\n\t"
    "buffer_load_dwordx4 %[wvoff], %[srdW], s73 offen lds\n\t"
    "v_mfma_f32_16x16x128_f8f6f4 v[244:247], v[104:111], v[168:175], v[244:247] cbsz:1\n\t"
    "ds_read_b64_tr_b8 v[76:77], v57 offset:57344\n\t"
    "ds_read_b64_tr_b8 v[78:79], v57 offset:58368\n\t"
    "s_waitcnt vmcnt(9)\n\t"
    "v_min3_u32 v53, v176, v177, v178\n\t"
    "v_min3_u32 v54, v179, v180, v181\n\t"
    "s_waitcnt lgkmcnt(12)\n\t"
    "v_mfma_f32_16x16x128_f8f6f4 v[216:219], v[112:119], v[160:167], v[216:219] cbsz:1\n\t"
    "ds_read_b64_tr_b8 v[80:81], v58 offset:49152\n\t"
    "ds_read_b64_tr_b8 v[82:83], v58 offset:50176\n\t"
    "v_min3_u32 v53, v53, v182, v183\n\t"
    "v_min3_u32 v54, v54, v184, v185\n\t"
    "v_mfma_f32_16x16x128_f8f6f4 v[248:251], v[112:119], v[168:175], v[248:251] cbsz:1\n\t"
    "ds_read_b64_tr_b8 v[84:85], v58 offset:57344\n\t"
    "ds_read_b64_tr_b8 v[86:87], v58 offset:58368\n\t"
    "s_add_u32 m0, %[m0w], 33792\n\t"
    "s_add_u32 s73, s73, 131072\n\t"
    "buffer_load_dwordx4 %[wvoff], %[srdW], s73 offen lds\n\t"
    "v_min3_u32 v53, v53, v186, v187\n\t"
    "v_min3_u32 v54, v54, v188, v189\n\t"
    "s_waitcnt lgkmcnt(12)\n\t"
    "v_mfma_f32_16x16x128_f8f6f4 v[220:223], v[120:127], v[160:167], v[220:223] cbsz:1\n\t"
    "ds_read_b64_tr_b8 v[88:89], v59 offset:49152\n\t"
    "ds_read_b64_tr_b8 v[90:91], v59 offset:50176\n\t"
    "v_min3_u32 v53, v53, v190, v191\n\t"
    "v_min_u32 v53, v53, v54\n\t"
    "v_mfma_f32_16x16x128_f8f6f4 v[252:255], v[120:127], v[168:175], v[252:255] cbsz:1\n\t"
    "ds_read_b64_tr_b8 v[92:93], v59 offset:57344\n\t"
    "ds_read_b64_tr_b8 v[94:95], v59 offset:58368\n\t"
    "v_readlane_b32 s77, %[tx], 9\n\t"
    "s_mov_b32 s75, s77\n\t"
    "buffer_load_dwordx4 v[144:147], %[xvoff], s[60:63], s75 offen offset:0 sc0 sc1\n\t"
    "v_cmp_eq_u32 vcc, 0, v53\n\t"
    "s_cbranch_vccz L_ok7_%=\n\t"
    "v_readlane_b32 s77, %[tx], 7\n\t"
    "s_mov_b32 s82, s77\n\t"
    "s_getpc_b64 s[58:59]\n\t"
    "s_branch L_slow3_%=\n\t"
    "L_ok7_%=:\n\t"
    "s_waitcnt lgkmcnt(12)\n\t"
    "v_mfma_f32_16x16x128_f8f6f4 v[192:195], v[64:71], v[176:183], v[192:195] cbsz:1\n\t"
    "ds_read_b64_tr_b8 v[96:97], v60 offset:49152\n\t"
    "ds_read_b64_tr_b8 v[98:99], v60 offset:50176\n\t"
    "v_xor_b32 v56, 0x10000, v56\n\t"
    "v_mfma_f32_16x16x128_f8f6f4 v[224:227], v[64:71], v[184:191], v[224:227] cbsz:1\n\t"
    "ds_read_b64_tr_b8 v[100:101], v60 offset:57344\n\t"
    "ds_read_b64_tr_b8 v[102:103], v60 offset:58368\n\t"
    "buffer_load_dwordx4 v[148:151], %[xvoff], s[60:63], s75 offen offset:1024 sc0 sc1\n\t"
    "v_xor_b32 v57, 0x10000, v57\n\t"
    "s_waitcnt lgkmcnt(12)\n\t"
    "v_mfma_f32_16x16x128_f8f6f4 v[196:199], v[72:79], v[176:183], v[196:199] cbsz:1\n\t"
    "ds_read_b64_tr_b8 v[104:105], v61 offset:49152\n\t"
    "ds_read_b64_tr_b8 v[106:107], v61 offset:50176\n\t"
    "v_xor_b32 v58, 0x10000, v58\n\t"
    "v_mfma_f32_16x16x128_f8f6f4 v[228:231], v[72:79], v[184:191], v[228:231] cbsz:1\n\t"
    "ds_read_b64_tr_b8 v[108:109], v61 offset:57344\n\t"
    "ds_read_b64_tr_b8 v[110:111], v61 offset:58368\n\t"
    "s_cmp_eq_u32 %[pushstep], 8\n\t"
    "s_cbranch_scc0 L_cxp8_%=\n\t"
    "buffer_load_dwordx4 v[48:51], %[ovoff], %[srdXin], %[xpoff] offen\n\t"
    "s_branch L_dxp8_%=\n\t"
    "L_cxp8_%=:\n\t"
    "buffer_load_dword v46, v55, %[srdM], s79 offen\n\t"
    "L_dxp8_%=:\n\t"
    "v_xor_b32 v59, 0x10000, v59\n\t"
    "s_waitcnt lgkmcnt(12)\n\t"
    "v_mfma_f32_16x16x128_f8f6f4 v[200:203], v[80:87], v[176:183], v[200:203] cbsz:1\n\t"
    "ds_read_b64_tr_b8 v[112:113], v62 offset:49152\n\t"
    "ds_read_b64_tr_b8 v[114:115], v62 offset:50176\n\t"
    "buffer_load_dwordx4 v[152:155], %[xvoff], s[60:63], s75 offen offset:2048 sc0 sc1\n\t"
    "v_mfma_f32_16x16x128_f8f6f4 v[232:235], v[80:87], v[184:191], v[232:235] cbsz:1\n\t"
    "ds_read_b64_tr_b8 v[116:117], v62 offset:57344\n\t"
    "ds_read_b64_tr_b8 v[118:119], v62 offset:58368\n\t"
    "s_waitcnt lgkmcnt(12)\n\t"
    "v_mfma_f32_16x16x128_f8f6f4 v[204:207], v[88:95], v[176:183], v[204:207] cbsz:1\n\t"
    "ds_read_b64_tr_b8 v[120:121], v63 offset:49152\n\t"
    "ds_read_b64_tr_b8 v[122:123], v63 offset:50176\n\t"
    "buffer_load_dwordx4 v[156:159], %[xvoff], s[60:63], s75 offen offset:3072 sc0 sc1\n\t"
    "v_mfma_f32_16x16x128_f8f6f4 v[236:239], v[88:95], v[184:191], v[236:239] cbsz:1\n\t"
    "ds_read_b64_tr_b8 v[124:125], v63 offset:57344\n\t"
    "ds_read_b64_tr_b8 v[126:127], v63 offset:58368\n\t"
    "s_waitcnt vmcnt(21)\n\t"
    "s_barrier\n\t"
    "s_waitcnt lgkmcnt(12)\n\t"
    "v_mfma_f32_16x16x128_f8f6f4 v[208:211], v[96:103], v[176:183], v[208:211] cbsz:1\n\t"
    "ds_read_b64_tr_b8 v[64:65], v56 offset:0\n\t"
    "ds_read_b64_tr_b8 v[66:67], v56 offset:1024\n\t"
    "v_xor_b32 v60, 0x10000, v60\n\t"
    "v_mfma_f32_16x16x128_f8f6f4 v[240:243], v[96:103], v[184:191], v[240:243] cbsz:1\n\t"
    "ds_read_b64_tr_b8 v[68:69], v56 offset:8192\n\t"
    "ds_read_b64_tr_b8 v[70:71], v56 offset:9216\n\t"
    "v_readlane_b32 s77, %[tw], 11\n\t"
    "s_mov_b32 s73, s77\n\t"
    "s_add_u32 m0, %[m0w], 49152\n\t"
    "s_nop 0\n\t"
    "buffer_load_dwordx4 %[wvoff], %[srdW], s73 offen lds\n\t"
    "v_xor_b32 v61, 0x10000, v61\n\t"
    "s_waitcnt lgkmcnt(12)\n\t"
    "v_mfma_f32_16x16x128_f8f6f4 v[212:215], v[104:111], v[176:183], v[212:215] cbsz:1\n\t"
    "ds_read_b64_tr_b8 v[72:73], v57 offset:0\n\t"
    "ds_read_b64_tr_b8 v[74:75], v57 offset:1024\n\t"
    "v_xor_b32 v62, 0x10000, v62\n\t"
    "v_mfma_f32_16x16x128_f8f6f4 v[244:247], v[104:111], v[184:191], v[244:247] cbsz:1\n\t"
    "ds_read_b64_tr_b8 v[76:77], v57 offset:8192\n\t"
    "ds_read_b64_tr_b8 v[78:79], v57 offset:9216\n\t"
    "s_add_u32 m0, %[m0w], 50176\n\t"
    "s_add_u32 s73, s73, 131072\n\t"
    "buffer_load_dwordx4 %[wvoff], %[srdW], s73 offen lds\n\t"
    "s_waitcnt vmcnt(9)\n\t"
    "v_min3_u32 v53, v128, v129, v130\n\t"
    "v_min3_u32 v54, v131, v132, v133\n\t"
    "v_xor_b32 v63, 0x10000, v63\n\t"
    "s_waitcnt lgkmcnt(12)\n\t"
    "v_mfma_f32_16x16x128_f8f6f4 v[216:219], v[112:119], v[176:183], v[216:219] cbsz:1\n\t"
    "ds_read_b64_tr_b8 v[80:81], v58 offset:0\n\t"
    "ds_read_b64_tr_b8 v[82:83], v58 offset:1024\n\t"
    "v_min3_u32 v53, v53, v134, v135\n\t"
    "v_min3_u32 v54, v54, v136, v137\n\t"
    "v_mfma_f32_16x16x128_f8f6f4 v[248:251], v[112:119], v[184:191], v[248:251] cbsz:1\n\t"
    "ds_read_b64_tr_b8 v[84:85], v58 offset:8192\n\t"
    "ds_read_b64_tr_b8 v[86:87], v58 offset:9216\n\t"
    "v_min3_u32 v53, v53, v138, v139\n\t"
    "v_min3_u32 v54, v54, v140, v141\n\t"
    "s_waitcnt lgkmcnt(12)\n\t"
    "v_mfma_f32_16x16x128_f8f6f4 v[220:223], v[120:127], v[176:183], v[220:223] cbsz:1\n\t"
    "ds_read_b64_tr_b8 v[88:89], v59 offset:0\n\t"
    "ds_read_b64_tr_b8 v[90:91], v59 offset:1024\n\t"
    "v_min3_u32 v53, v53, v142, v143\n\t"
    "v_min_u32 v53, v53, v54\n\t"
    "v_mfma_f32_16x16x128_f8f6f4 v[252:255], v[120:127], v[184:191], v[252:255] cbsz:1\n\t"
    "ds_read_b64_tr_b8 v[92:93], v59 offset:8192\n\t"
    "ds_read_b64_tr_b8 v[94:95], v59 offset:9216\n\t"
    "v_cmp_eq_u32 vcc, 0, v53\n\t"
    "s_cbranch_vccz L_ok8_%=\n\t"
    "v_readlane_b32 s77, %[tx], 8\n\t"
    "s_mov_b32 s82, s77\n\t"
    "s_getpc_b64 s[58:59]\n\t"
    "s_branch L_slow0_%=\n\t"
    "L_ok8_%=:\n\t"
    "s_waitcnt lgkmcnt(12)\n\t"
    "v_mfma_f32_16x16x128_f8f6f4 v[192:195], v[64:71], v[128:135], v[192:195] cbsz:1\n\t"
    "ds_read_b64_tr_b8 v[96:97], v60 offset:0\n\t"
    "ds_read_b64_tr_b8 v[98:99], v60 offset:1024\n\t"
    "v_mfma_f32_16x16x128_f8f6f4 v[224:227], v[64:71], v[136:143], v[224:227] cbsz:1\n\t"
    "ds_read_b64_tr_b8 v[100:101], v60 offset:8192\n\t"
    "ds_read_b64_tr_b8 v[102:103], v60 offset:9216\n\t"
    "v_readlane_b32 s77, %[tx], 10\n\t"
    "s_mov_b32 s75, s77\n\t"
    "buffer_load_dwordx4 v[160:163], %[xvoff], s[60:63], s75 offen offset:0 sc0 sc1\n\t"
    "s_waitcnt lgkmcnt(12)\n\t"
    "v_mfma_f32_16x16x128_f8f6f4 v[196:199], v[72:79], v[128:135], v[196:199] cbsz:1\n\t"
    "ds_read_b64_tr_b8 v[104:105], v61 offset:0\n\t"
    "ds_read_b64_tr_b8 v[106:107], v61 offset:1024\n\t"
    "buffer_load_dwordx4 v[164:167], %[xvoff], s[60:63], s75 offen offset:1024 sc0 sc1\n\t"
    "v_mfma_f32_16x16x128_f8f6f4 v[228:231], v[72:79], v[136:143], v[228:231] cbsz:1\n\t"
    "ds_read_b64_tr_b8 v[108:109], v61 offset:8192\n\t"
    "ds_read_b64_tr_b8 v[110:111], v61 offset:9216\n\t"
    "s_cmp_eq_u32 %[pushstep], 9\n\t"
    "s_cbranch_scc0 L_cxp9_%=\n\t"
    "buffer_load_dwordx4 v[48:51], %[ovoff], %[srdXin], %[xpoff] offen\n\t"
    "s_branch L_dxp9_%=\n\t"
    "L_cxp9_%=:\n\t"
    "buffer_load_dword v46, v55, %[srdM], s79 offen\n\t"
    "L_dxp9_%=:\n\t"
    "s_waitcnt lgkmcnt(12)\n\t"
    "v_mfma_f32_16x16x128_f8f6f4 v[200:203], v[80:87], v[128:135], v[200:203] cbsz:1\n\t"
    "ds_read_b64_tr_b8 v[112:113], v62 offset:0\n\t"
    "ds_read_b64_tr_b8 v[114:115], v62 offset:1024\n\t"
    "buffer_load_dwordx4 v[168:171], %[xvoff], s[60:63], s75 offen offset:2048 sc0 sc1\n\t"
    "v_mfma_f32_16x16x128_f8f6f4 v[232:235], v[80:87], v[136:143], v[232:235] cbsz:1\n\t"
    "ds_read_b64_tr_b8 v[116:117], v62 offset:8192\n\t"
    "ds_read_b64_tr_b8 v[118:119], v62 offset:9216\n\t"
    "s_waitcnt lgkmcnt(12)\n\t"
    "v_mfma_f32_16x16x128_f8f6f4 v[204:207], v[88:95], v[128:135], v[204:207] cbsz:1\n\t"
    "ds_read_b64_tr_b8 v[120:121], v63 offset:0\n\t"
    "ds_read_b64_tr_b8 v[122:123], v63 offset:1024\n\t"
    "buffer_load_dwordx4 v[172:175], %[xvoff], s[60:63], s75 offen offset:3072 sc0 sc1\n\t"
    "v_mfma_f32_16x16x128_f8f6f4 v[236:239], v[88:95], v[136:143], v[236:239] cbsz:1\n\t"
    "ds_read_b64_tr_b8 v[124:125], v63 offset:8192\n\t"
    "ds_read_b64_tr_b8 v[126:127], v63 offset:9216\n\t"
    "s_cmp_eq_u32 %[pushstep], 8\n\t"
    "s_cbranch_scc0 L_cnp8_%=\n\t"
    "s_waitcnt vmcnt(9)\n\t"
    "v_mov_b32 v54, 0x80\n\t"
    "v_cmp_eq_u32 vcc, 0, v48\n\t"
    "s_nop 1\n\t"
    "v_cndmask_b32 v48, v48, v54, vcc\n\t"
    "v_cmp_eq_u32 vcc, 0, v49\n\t"
    "s_nop 1\n\t"
    "v_cndmask_b32 v49, v49, v54, vcc\n\t"
    "v_cmp_eq_u32 vcc, 0, v50\n\t"
    "s_nop 1\n\t"
    "v_cndmask_b32 v50, v50, v54, vcc\n\t"
    "v_cmp_eq_u32 vcc, 0, v51\n\t"
    "s_nop 1\n\t"
    "v_cndmask_b32 v51, v51, v54, vcc\n\t"
    "buffer_store_dwordx4 v[48:51], %[pvst], s[64:67], 0 offen sc0 sc1\n\t"
    "s_branch L_dnp8_%=\n\t"
    "L_cnp8_%=:\n\t"
    "buffer_load_dword v46, v55, %[srdM], s79 offen\n\t"
    "L_dnp8_%=:\n\t"
    "s_waitcnt vmcnt(13)\n\t"
    "s_barrier\n\t"
    "s_waitcnt lgkmcnt(12)\n\t"
    "v_mfma_f32_16x16x128_f8f6f4 v[208:211], v[96:103], v[128:135], v[208:211] cbsz:1\n\t"
    "ds_read_b64_tr_b8 v[64:65], v56 offset:16384\n\t"
    "ds_read_b64_tr_b8 v[66:67], v56 offset:17408\n\t"
    "v_readlane_b32 s77, %[tw], 12\n\t"
    "s_mov_b32 s73, s77\n\t"
    "s_add_u32 m0, %[m0w], 65536\n\t"
    "s_nop 0\n\t"
    "buffer_load_dwordx4 %[wvoff], %[srdW], s73 offen lds\n\t"
    "v_mfma_f32_16x16x128_f8f6f4 v[240:243], v[96:103], v[136:143], v[240:243] cbsz:1\n\t"
    "ds_read_b64_tr_b8 v[68:69], v56 offset:24576\n\t"
    "ds_read_b64_tr_b8 v[70:71], v56 offset:25600\n\t"
    "s_cmp_eq_u32 %[pfw], 5\n\t"
    "s_cbranch_scc0 L_cpf5_%=\n\t"
    "buffer_load_dword v46, %[pfvoff], %[srdPF], %[pfso] offen sc1\n\t"
    "s_branch L_dpf5_%=\n\t"
    "L_cpf5_%=:\n\t"
    "buffer_load_dword v46, v55, %[srdM], s79 offen\n\t"
    "L_dpf5_%=:\n\t"
    "s_waitcnt lgkmcnt(12)\n\t"
    "v_mfma_f32_16x16x128_f8f6f4 v[212:215], v[104:111], v[128:135], v[212:215] cbsz:1\n\t"
    "ds_read_b64_tr_b8 v[72:73], v57 offset:16384\n\t"
    "ds_read_b64_tr_b8 v[74:75], v57 offset:17408\n\t"
    "s_add_u32 m0, %[m0w], 66560\n\t"
    "s_add_u32 s73, s73, 131072\n\t"
    "buffer_load_dwordx4 %[wvoff], %[srdW], s73 offen lds\n\t"
    "v_mfma_f32_16x16x128_f8f6f4 v[244:247], v[104:111], v[136:143], v[244:247] cbsz:1\n\t"
    "ds_read_b64_tr_b8 v[76:77], v57 offset:24576\n\t"
    "ds_read_b64_tr_b8 v[78:79], v57 offset:25600\n\t"
    "s_waitcnt vmcnt(11)\n\t"
    "v_min3_u32 v53, v144, v145, v146\n\t"
    "v_min3_u32 v54, v147, v148, v149\n\t"
    "s_waitcnt lgkmcnt(12)\n\t"
    "v_mfma_f32_16x16x128_f8f6f4 v[216:219], v[112:119], v[128:135], v[216:219] cbsz:1\n\t"
    "ds_read_b64_tr_b8 v[80:81], v58 offset:16384\n\t"
    "ds_read_b64_tr_b8 v[82:83], v58 offset:17408\n\t"
    "v_min3_u32 v53, v53, v150, v151\n\t"
    "v_min3_u32 v54, v54, v152, v153\n\t"
    "v_mfma_f32_16x16x128_f8f6f4 v[248:251], v[112:119], v[136:143], v[248:251] cbsz:1\n\t"
    "ds_read_b64_tr_b8 v[84:85], v58 offset:24576\n\t"
    "ds_read_b64_tr_b8 v[86:87], v58 offset:25600\n\t"
    "v_min3_u32 v53, v53, v154, v155\n\t"
    "v_min3_u32 v54, v54, v156, v157\n\t"
    "s_waitcnt lgkmcnt(12)\n\t"
    "v_mfma_f32_16x16x128_f8f6f4 v[220:223], v[120:127], v[128:135], v[220:223] cbsz:1\n\t"
    "ds_read_b64_tr_b8 v[88:89], v59 offset:16384\n\t"
    "ds_read_b64_tr_b8 v[90:91], v59 offset:17408\n\t"
    "v_min3_u32 v53, v53, v158, v159\n\t"
    "v_min_u32 v53, v53, v54\n\t"
    "v_mfma_f32_16x16x128_f8f6f4 v[252:255], v[120:127], v[136:143], v[252:255] cbsz:1\n\t"
    "ds_read_b64_tr_b8 v[92:93], v59 offset:24576\n\t"
    "ds_read_b64_tr_b8 v[94:95], v59 offset:25600\n\t"
    "v_cmp_eq_u32 vcc, 0, v53\n\t"
    "s_cbranch_vccz L_ok9_%=\n\t"
    "v_readlane_b32 s77, %[tx], 9\n\t"
    "s_mov_b32 s82, s77\n\t"
    "s_getpc_b64 s[58:59]\n\t"
    "s_branch L_slow1_%=\n\t"
    "L_ok9_%=:\n\t"
    "s_waitcnt lgkmcnt(12)\n\t"
    "v_mfma_f32_16x16x128_f8f6f4 v[192:195], v[64:71], v[144:151], v[192:195] cbsz:1\n\t"
    "ds_read_b64_tr_b8 v[96:97], v60 offset:16384\n\t"
    "ds_read_b64_tr_b8 v[98:99], v60 offset:17408\n\t"
    "v_mfma_f32_16x16x128_f8f6f4 v[224:227], v[64:71], v[152:159], v[224:227] cbsz:1\n\t"
    "ds_read_b64_tr_b8 v[100:101], v60 offset:24576\n\t"
    "ds_read_b64_tr_b8 v[102:103], v60 offset:25600\n\t"
    "v_readlane_b32 s77, %[tx], 11\n\t"
    "s_mov_b32 s75, s77\n\t"
    "buffer_load_dwordx4 v[176:179], %[xvoff], s[60:63], s75 offen offset:0 sc0 sc1\n\t"
    "s_waitcnt lgkmcnt(12)\n\t"
    "v_mfma_f32_16x16x128_f8f6f4 v[196:199], v[72:79], v[144:151], v[196:199] cbsz:1\n\t"
    "ds_read_b64_tr_b8 v[104:105], v61 offset:16384\n\t"
    "ds_read_b64_tr_b8 v[106:107], v61 offset:17408\n\t"
    "buffer_load_dwordx4 v[180:183], %[xvoff], s[60:63], s75 offen offset:1024 sc0 sc1\n\t"
    "v_mfma_f32_16x16x128_f8f6f4 v[228:231], v[72:79], v[152:159], v[228:231] cbsz:1\n\t"
    "ds_read_b64_tr_b8 v[108:109], v61 offset:24576\n\t"
    "ds_read_b64_tr_b8 v[110:111], v61 offset:25600\n\t"
    "s_waitcnt lgkmcnt(12)\n\t"
    "v_mfma_f32_16x16x128_f8f6f4 v[200:203], v[80:87], v[144:151], v[200:203] cbsz:1\n\t"
    "ds_read_b64_tr_b8 v[112:113], v62 offset:16384\n\t"
    "ds_read_b64_tr_b8 v[114:115], v62 offset:17408\n\t"
    "buffer_load_dwordx4 v[184:187], %[xvoff], s[60:63], s75 offen offset:2048 sc0 sc1\n\t"
    "v_mfma_f32_16x16x128_f8f6f4 v[232:235], v[80:87], v[152:159], v[232:235] cbsz:1\n\t"
    "ds_read_b64_tr_b8 v[116:117], v62 offset:24576\n\t"
    "ds_read_b64_tr_b8 v[118:119], v62 offset:25600\n\t"
    "s_waitcnt lgkmcnt(12)\n\t"
    "v_mfma_f32_16x16x128_f8f6f4 v[204:207], v[88:95], v[144:151], v[204:207] cbsz:1\n\t"
    "ds_read_b64_tr_b8 v[120:121], v63 offset:16384\n\t"
    "ds_read_b64_tr_b8 v[122:123], v63 offset:17408\n\t"
    "buffer_load_dwordx4 v[188:191], %[xvoff], s[60:63], s75 offen offset:3072 sc0 sc1\n\t"
    "v_mfma_f32_16x16x128_f8f6f4 v[236:239], v[88:95], v[152:159], v[236:239] cbsz:1\n\t"
    "ds_read_b64_tr_b8 v[124:125], v63 offset:24576\n\t"
    "ds_read_b64_tr_b8 v[126:127], v63 offset:25600\n\t"
    "s_cmp_eq_u32 %[pushstep], 9\n\t"
    "s_cbranch_scc0 L_cnp9_%=\n\t"
    "s_waitcnt vmcnt(10)\n\t"
    "v_mov_b32 v54, 0x80\n\t"
    "v_cmp_eq_u32 vcc, 0, v48\n\t"
    "s_nop 1\n\t"
    "v_cndmask_b32 v48, v48, v54, vcc\n\t"
    "v_cmp_eq_u32 vcc, 0, v49\n\t"
    "s_nop 1\n\t"
    "v_cndmask_b32 v49, v49, v54, vcc\n\t"
    "v_cmp_eq_u32 vcc, 0, v50\n\t"
    "s_nop 1\n\t"
    "v_cndmask_b32 v50, v50, v54, vcc\n\t"
    "v_cmp_eq_u32 vcc, 0, v51\n\t"
    "s_nop 1\n\t"
    "v_cndmask_b32 v51, v51, v54, vcc\n\t"
    "buffer_store_dwordx4 v[48:51], %[pvst], s[64:67], 0 offen sc0 sc1\n\t"
    "s_branch L_dnp9_%=\n\t"
    "L_cnp9_%=:\n\t"
    "buffer_load_dword v46, v55, %[srdM], s79 offen\n\t"
    "L_dnp9_%=:\n\t"
    "s_waitcnt lgkmcnt(12)\n\t"
    "v_mfma_f32_16x16x128_f8f6f4 v[208:211], v[96:103], v[144:151], v[208:211] cbsz:1\n\t"
    "ds_read_b64_tr_b8 v[64:65], v56 offset:32768\n\t"
    "ds_read_b64_tr_b8 v[66:67], v56 offset:33792\n\t"
    "v_readlane_b32 s77, %[tw], 13\n\t"
    "s_mov_b32 s73, s77\n\t"
    "s_add_u32 m0, %[m0w], 81920\n\t"
    "s_nop 0\n\t"
    "buffer_load_dwordx4 %[wvoff], %[srdW], s73 offen lds\n\t"
    "v_mfma_f32_16x16x128_f8f6f4 v[240:243], v[96:103], v[152:159], v[240:243] cbsz:1\n\t"
    "ds_read_b64_tr_b8 v[68:69], v56 offset:40960\n\t"
    "ds_read_b64_tr_b8 v[70:71], v56 offset:41984\n\t"
    "s_waitcnt lgkmcnt(12)\n\t"
    "v_mfma_f32_16x16x128_f8f6f4 v[212:215], v[104:111], v[144:151], v[212:215] cbsz:1\n\t"
    "ds_read_b64_tr_b8 v[72:73], v57 offset:32768\n\t"
    "ds_read_b64_tr_b8 v[74:75], v57 offset:33792\n\t"
    "s_add_u32 m0, %[m0w], 82944\n\t"
    "s_add_u32 s73, s73, 131072\n\t"
    "buffer_load_dwordx4 %[wvoff], %[srdW], s73 offen lds\n\t"
    "v_mfma_f32_16x16x128_f8f6f4 v[244:247], v[104:111], v[152:159], v[244:247] cbsz:1\n\t"
    "ds_read_b64_tr_b8 v[76:77], v57 offset:40960\n\t"
    "ds_read_b64_tr_b8 v[78:79], v57 offset:41984\n\t"
    "s_waitcnt vmcnt(11)\n\t"
    "v_min3_u32 v53, v160, v161, v162\n\t"
    "v_min3_u32 v54, v163, v164, v165\n\t"
    "s_waitcnt lgkmcnt(12)\n\t"
    "v_mfma_f32_16x16x128_f8f6f4 v[216:219], v[112:119], v[144:151], v[216:219] cbsz:1\n\t"
    "ds_read_b64_tr_b8 v[80:81], v58 offset:32768\n\t"
    "ds_read_b64_tr_b8 v[82:83], v58 offset:33792\n\t"
    "v_min3_u32 v53, v53, v166, v167\n\t"
    "v_min3_u32 v54, v54, v168, v169\n\t"
    "v_mfma_f32_16x16x128_f8f6f4 v[248:251], v[112:119], v[152:159], v[248:251] cbsz:1\n\t"
    "ds_read_b64_tr_b8 v[84:85], v58 offset:40960\n\t"
    "ds_read_b64_tr_b8 v[86:87], v58 offset:41984\n\t"
    "v_min3_u32 v53, v53, v170, v171\n\t"
    "v_min3_u32 v54, v54, v172, v173\n\t"
    "s_waitcnt lgkmcnt(12)\n\t"
    "v_mfma_f32_16x16x128_f8f6f4 v[220:223], v[120:127], v[144:151], v[220:223] cbsz:1\n\t"
    "ds_read_b64_tr_b8 v[88:89], v59 offset:32768\n\t"
    "ds_read_b64_tr_b8 v[90:91], v59 offset:33792\n\t"
    "v_min3_u32 v53, v53, v174, v175\n\t"
    "v_min_u32 v53, v53, v54\n\t"
    "v_mfma_f32_16x16x128_f8f6f4 v[252:255], v[120:127], v[152:159], v[252:255] cbsz:1\n\t"
    "ds_read_b64_tr_b8 v[92:93], v59 offset:40960\n\t"
    "ds_read_b64_tr_b8 v[94:95], v59 offset:41984\n\t"
    "v_cmp_eq_u32 vcc, 0, v53\n\t"
    "s_cbranch_vccz L_ok10_%=\n\t"
    "v_readlane_b32 s77, %[tx], 10\n\t"
    "s_mov_b32 s82, s77\n\t"
    "s_getpc_b64 s[58:59]\n\t"
    "s_branch L_slow2_%=\n\t"
    "L_ok10_%=:\n\t"
    "s_waitcnt lgkmcnt(12)\n\t"
    "v_mfma_f32_16x16x128_f8f6f4 v[192:195], v[64:71], v[160:167], v[192:195] cbsz:1\n\t"
    "ds_read_b64_tr_b8 v[96:97], v60 offset:32768\n\t"
    "ds_read_b64_tr_b8 v[98:99], v60 offset:33792\n\t"
    "v_mfma_f32_16x16x128_f8f6f4 v[224:227], v[64:71], v[168:175], v[224:227] cbsz:1\n\t"
    "ds_read_b64_tr_b8 v[100:101], v60 offset:40960\n\t"
    "ds_read_b64_tr_b8 v[102:103], v60 offset:41984\n\t"
    "v_readlane_b32 s77, %[tx], 12\n\t"
    "s_mov_b32 s75, s77\n\t"
    "buffer_load_dwordx4 v[128:131], %[xvoff], s[60:63], s75 offen offset:0 sc0 sc1\n\t"
    "s_waitcnt lgkmcnt(12)\n\t"
    "v_mfma_f32_16x16x128_f8f6f4 v[196:199], v[72:79], v[160:167], v[196:199] cbsz:1\n\t"
    "ds_read_b64_tr_b8 v[104:105], v61 offset:32768\n\t"
    "ds_read_b64_tr_b8 v[106:107], v61 offset:33792\n\t"
    "buffer_load_dwordx4 v[132:135], %[xvoff], s[60:63], s75 offen offset:1024 sc0 sc1\n\t"
    "v_mfma_f32_16x16x128_f8f6f4 v[228:231], v[72:79], v[168:175], v[228:231] cbsz:1\n\t"
    "ds_read_b64_tr_b8 v[108:109], v61 offset:40960\n\t"
    "ds_read_b64_tr_b8 v[110:111], v61 offset:41984\n\t"
    "s_waitcnt lgkmcnt(12)\n\t"
    "v_mfma_f32_16x16x128_f8f6f4 v[200:203], v[80:87], v[160:167], v[200:203] cbsz:1\n\t"
    "ds_read_b64_tr_b8 v[112:113], v62 offset:32768\n\t"
    "ds_read_b64_tr_b8 v[114:115], v62 offset:33792\n\t"
    "buffer_load_dwordx4 v[136:139], %[xvoff], s[60:63], s75 offen offset:2048 sc0 sc1\n\t"
    "v_mfma_f32_16x16x128_f8f6f4 v[232:235], v[80:87], v[168:175], v[232:235] cbsz:1\n\t"
    "ds_read_b64_tr_b8 v[116:117], v62 offset:40960\n\t"
    "ds_read_b64_tr_b8 v[118:119], v62 offset:41984\n\t"
    "s_waitcnt lgkmcnt(12)\n\t"
    "v_mfma_f32_16x16x128_f8f6f4 v[204:207], v[88:95], v[160:167], v[204:207] cbsz:1\n\t"
    "ds_read_b64_tr_b8 v[120:121], v63 offset:32768\n\t"
    "ds_read_b64_tr_b8 v[122:123], v63 offset:33792\n\t"
    "buffer_load_dwordx4 v[140:143], %[xvoff], s[60:63], s75 offen offset:3072 sc0 sc1\n\t"
    "v_mfma_f32_16x16x128_f8f6f4 v[236:239], v[88:95], v[168:175], v[236:239] cbsz:1\n\t"
    "ds_read_b64_tr_b8 v[124:125], v63 offset:40960\n\t"
    "ds_read_b64_tr_b8 v[126:127], v63 offset:41984\n\t"
    "s_waitcnt vmcnt(11)\n\t"
    "s_barrier\n\t"
    "s_waitcnt lgkmcnt(12)\n\t"
    "v_mfma_f32_16x16x128_f8f6f4 v[208:211], v[96:103], v[160:167], v[208:211] cbsz:1\n\t"
    "ds_read_b64_tr_b8 v[64:65], v56 offset:49152\n\t"
    "ds_read_b64_tr_b8 v[66:67], v56 offset:50176\n\t"
    "v_readlane_b32 s77, %[tw], 14\n\t"
    "s_mov_b32 s73, s77\n\t"
    "s_add_u32 m0, %[m0w], 98304\n\t"
    "s_nop 0\n\t"
    "buffer_load_dwordx4 %[wvoff], %[srdW], s73 offen lds\n\t"
    "v_mfma_f32_16x16x128_f8f6f4 v[240:243], v[96:103], v[168:175], v[240:243] cbsz:1\n\t"
    "ds_read_b64_tr_b8 v[68:69], v56 offset:57344\n\t"
    "ds_read_b64_tr_b8 v[70:71], v56 offset:58368\n\t"
    "s_cmp_eq_u32 %[pfw], 4\n\t"
    "s_cbranch_scc0 L_cpf4_%=\n\t"
    "buffer_load_dword v46, %[pfvoff], %[srdPF], %[pfso] offen sc1\n\t"
    "s_branch L_dpf4_%=\n\t"
    "L_cpf4_%=:\n\t"
    "buffer_load_dword v46, v55, %[srdM], s79 offen\n\t"
    "L_dpf4_%=:\n\t"
    "s_waitcnt lgkmcnt(12)\n\t"
    "v_mfma_f32_16x16x128_f8f6f4 v[212:215], v[104:111], v[160:167], v[212:215] cbsz:1\n\t"
    "ds_read_b64_tr_b8 v[72:73], v57 offset:49152\n\t"
    "ds_read_b64_tr_b8 v[74:75], v57 offset:50176\n\t"
    "s_add_u32 m0, %[m0w], 99328\n\t"
    "s_add_u32 s73, s73, 131072\n\t"
    "buffer_load_dwordx4 %[wvoff], %[srdW], s73 offen lds\n\t"
    "v_mfma_f32_16x16x128_f8f6f4 v[244:247], v[104:111], v[168:175], v[244:247] cbsz:1\n\t"
    "ds_read_b64_tr_b8 v[76:77], v57 offset:57344\n\t"
    "ds_read_b64_tr_b8 v[78:79], v57 offset:58368\n\t"
    "s_waitcnt vmcnt(10)\n\t"
    "v_min3_u32 v53, v176, v177, v178\n\t"
    "v_min3_u32 v54, v179, v180, v181\n\t"
    "s_waitcnt lgkmcnt(12)\n\t"
    "v_mfma_f32_16x16x128_f8f6f4 v[216:219], v[112:119], v[160:167], v[216:219] cbsz:1\n\t"
    "ds_read_b64_tr_b8 v[80:81], v58 offset:49152\n\t"
    "ds_read_b64_tr_b8 v[82:83], v58 offset:50176\n\t"
    "v_min3_u32 v53, v53, v182, v183\n\t"
    "v_min3_u32 v54, v54, v184, v185\n\t"
    "v_mfma_f32_16x16x128_f8f6f4 v[248:251], v[112:119], v[168:175], v[248:251] cbsz:1\n\t"
    "ds_read_b64_tr_b8 v[84:85], v58 offset:57344\n\t"
    "ds_read_b64_tr_b8 v[86:87], v58 offset:58368\n\t"
    "v_min3_u32 v53, v53, v186, v187\n\t"
    "v_min3_u32 v54, v54, v188, v189\n\t"
    "s_waitcnt lgkmcnt(12)\n\t"
    "v_mfma_f32_16x16x128_f8f6f4 v[220:223], v[120:127], v[160:167], v[220:223] cbsz:1\n\t"
    "ds_read_b64_tr_b8 v[88:89], v59 offset:49152\n\t"
    "ds_read_b64_tr_b8 v[90:91], v59 offset:50176\n\t"
    "v_min3_u32 v53, v53, v190, v191\n\t"
    "v_min_u32 v53, v53, v54\n\t"
    "v_mfma_f32_16x16x128_f8f6f4 v[252:255], v[120:127], v[168:175], v[252:255] cbsz:1\n\t"
    "ds_read_b64_tr_b8 v[92:93], v59 offset:57344\n\t"
    "ds_read_b64_tr_b8 v[94:95], v59 offset:58368\n\t"
    "v_cmp_eq_u32 vcc, 0, v53\n\t"
    "s_cbranch_vccz L_ok11_%=\n\t"
    "v_readlane_b32 s77, %[tx], 11\n\t"
    "s_mov_b32 s82, s77\n\t"
    "s_getpc_b64 s[58:59]\n\t"
    "s_branch L_slow3_%=\n\t"
    "L_ok11_%=:\n\t"
    "s_waitcnt lgkmcnt(12)\n\t"
    "v_mfma_f32_16x16x128_f8f6f4 v[192:195], v[64:71], v[176:183], v[192:195] cbsz:1\n\t"
    "ds_read_b64_tr_b8 v[96:97], v60 offset:49152\n\t"
    "ds_read_b64_tr_b8 v[98:99], v60 offset:50176\n\t"
    "v_xor_b32 v56, 0x10000, v56\n\t"
    "v_mfma_f32_16x16x128_f8f6f4 v[224:227], v[64:71], v[184:191], v[224:227] cbsz:1\n\t"
    "ds_read_b64_tr_b8 v[100:101], v60 offset:57344\n\t"
    "ds_read_b64_tr_b8 v[102:103], v60 offset:58368\n\t"
    "v_readlane_b32 s77, %[tx], 13\n\t"
    "s_mov_b32 s75, s77\n\t"
    "buffer_load_dwordx4 v[144:147], %[xvoff], s[60:63], s75 offen offset:0 sc0 sc1\n\t"
    "v_xor_b32 v57, 0x10000, v57\n\t"
    "s_waitcnt lgkmcnt(12)\n\t"
    "v_mfma_f32_16x16x128_f8f6f4 v[196:199], v[72:79], v[176:183], v[196:199] cbsz:1\n\t"
    "ds_read_b64_tr_b8 v[104:105], v61 offset:49152\n\t"
    "ds_read_b64_tr_b8 v[106:107], v61 offset:50176\n\t"
    "buffer_load_dwordx4 v[148:151], %[xvoff], s[60:63], s75 offen offset:1024 sc0 sc1\n\t"
    "v_xor_b32 v58, 0x10000, v58\n\t"
    "v_mfma_f32_16x16x128_f8f6f4 v[228:231], v[72:79], v[184:191], v[228:231] cbsz:1\n\t"
    "ds_read_b64_tr_b8 v[108:109], v61 offset:57344\n\t"
    "ds_read_b64_tr_b8 v[110:111], v61 offset:58368\n\t"
    "v_xor_b32 v59, 0x10000, v59\n\t"
    "s_waitcnt lgkmcnt(12)\n\t"
    "v_mfma_f32_16x16x128_f8f6f4 v[200:203], v[80:87], v[176:183], v[200:203] cbsz:1\n\t"
    "ds_read_b64_tr_b8 v[112:113], v62 offset:49152\n\t"
    "ds_read_b64_tr_b8 v[114:115], v62 offset:50176\n\t"
    "buffer_load_dwordx4 v[152:155], %[xvoff], s[60:63], s75 offen offset:2048 sc0 sc1\n\t"
    "v_mfma_f32_16x16x128_f8f6f4 v[232:235], v[80:87], v[184:191], v[232:235] cbsz:1\n\t"
    "ds_read_b64_tr_b8 v[116:117], v62 offset:57344\n\t"
    "ds_read_b64_tr_b8 v[118:119], v62 offset:58368\n\t"
    "s_waitcnt lgkmcnt(12)\n\t"
    "v_mfma_f32_16x16x128_f8f6f4 v[204:207], v[88:95], v[176:183], v[204:207] cbsz:1\n\t"
    "ds_read_b64_tr_b8 v[120:121], v63 offset:49152\n\t"
    "ds_read_b64_tr_b8 v[122:123], v63 offset:50176\n\t"
    "buffer_load_dwordx4 v[156:159], %[xvoff], s[60:63], s75 offen offset:3072 sc0 sc1\n\t"
    "v_mfma_f32_16x16x128_f8f6f4 v[236:239], v[88:95], v[184:191], v[236:239] cbsz:1\n\t"
    "ds_read_b64_tr_b8 v[124:125], v63 offset:57344\n\t"
    "ds_read_b64_tr_b8 v[126:127], v63 offset:58368\n\t"
    "s_waitcnt lgkmcnt(12)\n\t"
    "v_mfma_f32_16x16x128_f8f6f4 v[208:211], v[96:103], v[176:183], v[208:211] cbsz:1\n\t"
    "ds_read_b64_tr_b8 v[64:65], v56 offset:0\n\t"
    "ds_read_b64_tr_b8 v[66:67], v56 offset:1024\n\t"
    "v_readlane_b32 s77, %[tw], 15\n\t"
    "s_mov_b32 s73, s77\n\t"
    "s_add_u32 m0, %[m0w], 114688\n\t"
    "s_nop 0\n\t"
    "buffer_load_dwordx4 %[wvoff], %[srdW], s73 offen lds\n\t"
    "v_xor_b32 v60, 0x10000, v60\n\t"
    "v_mfma_f32_16x16x128_f8f6f4 v[240:243], v[96:103], v[184:191], v[240:243] cbsz:1\n\t"
    "ds_read_b64_tr_b8 v[68:69], v56 offset:8192\n\t"
    "ds_read_b64_tr_b8 v[70:71], v56 offset:9216\n\t"
    "v_xor_b32 v61, 0x10000, v61\n\t"
    "s_waitcnt lgkmcnt(12)\n\t"
    "v_mfma_f32_16x16x128_f8f6f4 v[212:215], v[104:111], v[176:183], v[212:215] cbsz:1\n\t"
    "ds_read_b64_tr_b8 v[72:73], v57 offset:0\n\t"
    "ds_read_b64_tr_b8 v[74:75], v57 offset:1024\n\t"
    "s_add_u32 m0, %[m0w], 115712\n\t"
    "s_add_u32 s73, s73, 131072\n\t"
    "buffer_load_dwordx4 %[wvoff], %[srdW], s73 offen lds\n\t"
    "v_xor_b32 v62, 0x10000, v62\n\t"
    "v_mfma_f32_16x16x128_f8f6f4 v[244:247], v[104:111], v[184:191], v[244:247] cbsz:1\n\t"
    "ds_read_b64_tr_b8 v[76:77], v57 offset:8192\n\t"
    "ds_read_b64_tr_b8 v[78:79], v57 offset:9216\n\t"
    "s_waitcnt vmcnt(9)\n\t"
    "v_min3_u32 v53, v128, v129, v130\n\t"
    "v_min3_u32 v54, v131, v132, v133\n\t"
    "v_xor_b32 v63, 0x10000, v63\n\t"
    "s_waitcnt lgkmcnt(12)\n\t"
    "v_mfma_f32_16x16x128_f8f6f4 v[216:219], v[112:119], v[176:183], v[216:219] cbsz:1\n\t"
    "ds_read_b64_tr_b8 v[80:81], v58 offset:0\n\t"
    "ds_read_b64_tr_b8 v[82:83], v58 offset:1024\n\t"
    "v_min3_u32 v53, v53, v134, v135\n\t"
    "v_min3_u32 v54, v54, v136, v137\n\t"
    "v_mfma_f32_16x16x128_f8f6f4 v[248:251], v[112:119], v[184:191], v[248:251] cbsz:1\n\t"
    "ds_read_b64_tr_b8 v[84:85], v58 offset:8192\n\t"
    "ds_read_b64_tr_b8 v[86:87], v58 offset:9216\n\t"
    "v_min3_u32 v53, v53, v138, v139\n\t"
    "v_min3_u32 v54, v54, v140, v141\n\t"
    "s_waitcnt lgkmcnt(12)\n\t"
    "v_mfma_f32_16x16x128_f8f6f4 v[220:223], v[120:127], v[176:183], v[220:223] cbsz:1\n\t"
    "ds_read_b64_tr_b8 v[88:89], v59 offset:0\n\t"
    "ds_read_b64_tr_b8 v[90:91], v59 offset:1024\n\t"
    "v_min3_u32 v53, v53, v142, v143\n\t"
    "v_min_u32 v53, v53, v54\n\t"
    "v_mfma_f32_16x16x128_f8f6f4 v[252:255], v[120:127], v[184:191], v[252:255] cbsz:1\n\t"
    "ds_read_b64_tr_b8 v[92:93], v59 offset:8192\n\t"
    "ds_read_b64_tr_b8 v[94:95], v59 offset:9216\n\t"
    "v_cmp_eq_u32 vcc, 0, v53\n\t"
    "s_cbranch_vccz L_ok12_%=\n\t"
    "v_readlane_b32 s77, %[tx], 12\n\t"
    "s_mov_b32 s82, s77\n\t"
    "s_getpc_b64 s[58:59]\n\t"
    "s_branch L_slow0_%=\n\t"
    "L_ok12_%=:\n\t"
    "s_waitcnt lgkmcnt(12)\n\t"
    "v_mfma_f32_16x16x128_f8f6f4 v[192:195], v[64:71], v[128:135], v[192:195] cbsz:1\n\t"
    "ds_read_b64_tr_b8 v[96:97], v60 offset:0\n\t"
    "ds_read_b64_tr_b8 v[98:99], v60 offset:1024\n\t"
    "v_mfma_f32_16x16x128_f8f6f4 v[224:227], v[64:71], v[136:143], v[224:227] cbsz:1\n\t"
    "ds_read_b64_tr_b8 v[100:101], v60 offset:8192\n\t"
    "ds_read_b64_tr_b8 v[102:103], v60 offset:9216\n\t"
    "v_readlane_b32 s77, %[tx], 14\n\t"
    "s_mov_b32 s75, s77\n\t"
    "buffer_load_dwordx4 v[160:163], %[xvoff], s[60:63], s75 offen offset:0 sc0 sc1\n\t"
    "s_waitcnt lgkmcnt(12)\n\t"
    "v_mfma_f32_16x16x128_f8f6f4 v[196:199], v[72:79], v[128:135], v[196:199] cbsz:1\n\t"
    "ds_read_b64_tr_b8 v[104:105], v61 offset:0\n\t"
    "ds_read_b64_tr_b8 v[106:107], v61 offset:1024\n\t"
    "buffer_load_dwordx4 v[164:167], %[xvoff], s[60:63], s75 offen offset:1024 sc0 sc1\n\t"
    "v_mfma_f32_16x16x128_f8f6f4 v[228:231], v[72:79], v[136:143], v[228:231] cbsz:1\n\t"
    "ds_read_b64_tr_b8 v[108:109], v61 offset:8192\n\t"
    "ds_read_b64_tr_b8 v[110:111], v61 offset:9216\n\t"
    "s_waitcnt lgkmcnt(12)\n\t"
    "v_mfma_f32_16x16x128_f8f6f4 v[200:203], v[80:87], v[128:135], v[200:203] cbsz:1\n\t"
    "ds_read_b64_tr_b8 v[112:113], v62 offset:0\n\t"
    "ds_read_b64_tr_b8 v[114:115], v62 offset:1024\n\t"
    "buffer_load_dwordx4 v[168:171], %[xvoff], s[60:63], s75 offen offset:2048 sc0 sc1\n\t"
    "v_mfma_f32_16x16x128_f8f6f4 v[232:235], v[80:87], v[136:143], v[232:235] cbsz:1\n\t"
    "ds_read_b64_tr_b8 v[116:117], v62 offset:8192\n\t"
    "ds_read_b64_tr_b8 v[118:119], v62 offset:9216\n\t"
    "s_waitcnt lgkmcnt(12)\n\t"
    "v_mfma_f32_16x16x128_f8f6f4 v[204:207], v[88:95], v[128:135], v[204:207] cbsz:1\n\t"
    "ds_read_b64_tr_b8 v[120:121], v63 offset:0\n\t"
    "ds_read_b64_tr_b8 v[122:123], v63 offset:1024\n\t"
    "buffer_load_dwordx4 v[172:175], %[xvoff], s[60:63], s75 offen offset:3072 sc0 sc1\n\t"
    "v_mfma_f32_16x16x128_f8f6f4 v[236:239], v[88:95], v[136:143], v[236:239] cbsz:1\n\t"
    "ds_read_b64_tr_b8 v[124:125], v63 offset:8192\n\t"
    "ds_read_b64_tr_b8 v[126:127], v63 offset:9216\n\t"
    "s_waitcnt vmcnt(10)\n\t"
    "s_barrier\n\t"
    "s_waitcnt lgkmcnt(12)\n\t"
    "v_mfma_f32_16x16x128_f8f6f4 v[208:211], v[96:103], v[128:135], v[208:211] cbsz:1\n\t"
    "ds_read_b64_tr_b8 v[64:65], v56 offset:16384\n\t"
    "ds_read_b64_tr_b8 v[66:67], v56 offset:17408\n\t"
    "v_readlane_b32 s77, %[tw], 16\n\t"
    "s_mov_b32 s73, s77\n\t"
    "s_add_u32 m0, %[m0w], 0\n\t"
    "s_nop 0\n\t"
    "buffer_load_dwordx4 %[wvoff], %[srdW], s73 offen lds\n\t"
    "v_mfma_f32_16x16x128_f8f6f4 v[240:243], v[96:103], v[136:143], v[240:243] cbsz:1\n\t"
    "ds_read_b64_tr_b8 v[68:69], v56 offset:24576\n\t"
    "ds_read_b64_tr_b8 v[70:71], v56 offset:25600\n\t"
    "s_cmp_eq_u32 %[pfw], 6\n\t"
    "s_cbranch_scc0 L_cpf6_%=\n\t"
    "buffer_load_dword v46, %[pfvoff], %[srdPF], %[pfso] offen sc1\n\t"
    "s_branch L_dpf6_%=\n\t"
    "L_cpf6_%=:\n\t"
    "buffer_load_dword v46, v55, %[srdM], s79 offen\n\t"
    "L_dpf6_%=:\n\t"
    "s_waitcnt lgkmcnt(12)\n\t"
    "v_mfma_f32_16x16x128_f8f6f4 v[212:215], v[104:111], v[128:135], v[212:215] cbsz:1\n\t"
    "ds_read_b64_tr_b8 v[72:73], v57 offset:16384\n\t"
    "ds_read_b64_tr_b8 v[74:75], v57 offset:17408\n\t"
    "s_add_u32 m0, %[m0w], 1024\n\t"
    "s_add_u32 s73, s73, 131072\n\t"
    "buffer_load_dwordx4 %[wvoff], %[srdW], s73 offen lds\n\t"
    "v_mfma_f32_16x16x128_f8f6f4 v[244:247], v[104:111], v[136:143], v[244:247] cbsz:1\n\t"
    "ds_read_b64_tr_b8 v[76:77], v57 offset:24576\n\t"
    "ds_read_b64_tr_b8 v[78:79], v57 offset:25600\n\t"
    "s_waitcnt vmcnt(9)\n\t"
    "v_min3_u32 v53, v144, v145, v146\n\t"
    "v_min3_u32 v54, v147, v148, v149\n\t"
    "s_waitcnt lgkmcnt(12)\n\t"
    "v_mfma_f32_16x16x128_f8f6f4 v[216:219], v[112:119], v[128:135], v[216:219] cbsz:1\n\t"
    "ds_read_b64_tr_b8 v[80:81], v58 offset:16384\n\t"
    "ds_read_b64_tr_b8 v[82:83], v58 offset:17408\n\t"
    "v_min3_u32 v53, v53, v150, v151\n\t"
    "v_min3_u32 v54, v54, v152, v153\n\t"
    "v_mfma_f32_16x16x128_f8f6f4 v[248:251], v[112:119], v[136:143], v[248:251] cbsz:1\n\t"
    "ds_read_b64_tr_b8 v[84:85], v58 offset:24576\n\t"
    "ds_read_b64_tr_b8 v[86:87], v58 offset:25600\n\t"
    "v_min3_u32 v53, v53, v154, v155\n\t"
    "v_min3_u32 v54, v54, v156, v157\n\t"
    "s_waitcnt lgkmcnt(12)\n\t"
    "v_mfma_f32_16x16x128_f8f6f4 v[220:223], v[120:127], v[128:135], v[220:223] cbsz:1\n\t"
    "ds_read_b64_tr_b8 v[88:89], v59 offset:16384\n\t"
    "ds_read_b64_tr_b8 v[90:91], v59 offset:17408\n\t"
    "v_min3_u32 v53, v53, v158, v159\n\t"
    "v_min_u32 v53, v53, v54\n\t"
    "v_mfma_f32_16x16x128_f8f6f4 v[252:255], v[120:127], v[136:143], v[252:255] cbsz:1\n\t"
    "ds_read_b64_tr_b8 v[92:93], v59 offset:24576\n\t"
    "ds_read_b64_tr_b8 v[94:95], v59 offset:25600\n\t"
    "v_cmp_eq_u32 vcc, 0, v53\n\t"
    "s_cbranch_vccz L_ok13_%=\n\t"
    "v_readlane_b32 s77, %[tx], 13\n\t"
    "s_mov_b32 s82, s77\n\t"
    "s_getpc_b64 s[58:59]\n\t"
    "s_branch L_slow1_%=\n\t"
    "L_ok13_%=:\n\t"
    "s_waitcnt lgkmcnt(12)\n\t"
    "v_mfma_f32_16x16x128_f8f6f4 v[192:195], v[64:71], v[144:151], v[192:195] cbsz:1\n\t"
    "ds_read_b64_tr_b8 v[96:97], v60 offset:16384\n\t"
    "ds_read_b64_tr_b8 v[98:99], v60 offset:17408\n\t"
    "v_mfma_f32_16x16x128_f8f6f4 v[224:227], v[64:71], v[152:159], v[224:227] cbsz:1\n\t"
    "ds_read_b64_tr_b8 v[100:101], v60 offset:24576\n\t"
    "ds_read_b64_tr_b8 v[102:103], v60 offset:25600\n\t"
    "v_readlane_b32 s77, %[tx], 15\n\t"
    "s_mov_b32 s75, s77\n\t"
    "buffer_load_dwordx4 v[176:179], %[xvoff], s[60:63], s75 offen offset:0 sc0 sc1\n\t"
    "s_waitcnt lgkmcnt(12)\n\t"
    "v_mfma_f32_16x16x128_f8f6f4 v[196:199], v[72:79], v[144:151], v[196:199] cbsz:1\n\t"
    "ds_read_b64_tr_b8 v[104:105], v61 offset:16384\n\t"
    "ds_read_b64_tr_b8 v[106:107], v61 offset:17408\n\t"
    "buffer_load_dwordx4 v[180:183], %[xvoff], s[60:63], s75 offen offset:1024 sc0 sc1\n\t"
    "v_mfma_f32_16x16x128_f8f6f4 v[228:231], v[72:79], v[152:159], v[228:231] cbsz:1\n\t"
    "ds_read_b64_tr_b8 v[108:109], v61 offset:24576\n\t"
    "ds_read_b64_tr_b8 v[110:111], v61 offset:25600\n\t"
    "s_waitcnt lgkmcnt(12)\n\t"
    "v_mfma_f32_16x16x128_f8f6f4 v[200:203], v[80:87], v[144:151], v[200:203] cbsz:1\n\t"
    "ds_read_b64_tr_b8 v[112:113], v62 offset:16384\n\t"
    "ds_read_b64_tr_b8 v[114:115], v62 offset:17408\n\t"
    "buffer_load_dwordx4 v[184:187], %[xvoff], s[60:63], s75 offen offset:2048 sc0 sc1\n\t"
    "v_mfma_f32_16x16x128_f8f6f4 v[232:235], v[80:87], v[152:159], v[232:235] cbsz:1\n\t"
    "ds_read_b64_tr_b8 v[116:117], v62 offset:24576\n\t"
    "ds_read_b64_tr_b8 v[118:119], v62 offset:25600\n\t"
    "s_waitcnt lgkmcnt(12)\n\t"
    "v_mfma_f32_16x16x128_f8f6f4 v[204:207], v[88:95], v[144:151], v[204:207] cbsz:1\n\t"
    "ds_read_b64_tr_b8 v[120:121], v63 offset:16384\n\t"
    "ds_read_b64_tr_b8 v[122:123], v63 offset:17408\n\t"
    "buffer_load_dwordx4 v[188:191], %[xvoff], s[60:63], s75 offen offset:3072 sc0 sc1\n\t"
    "v_mfma_f32_16x16x128_f8f6f4 v[236:239], v[88:95], v[152:159], v[236:239] cbsz:1\n\t"
    "ds_read_b64_tr_b8 v[124:125], v63 offset:24576\n\t"
    "ds_read_b64_tr_b8 v[126:127], v63 offset:25600\n\t"
    "s_waitcnt lgkmcnt(12)\n\t"
    "v_mfma_f32_16x16x128_f8f6f4 v[208:211], v[96:103], v[144:151], v[208:211] cbsz:1\n\t"
    "ds_read_b64_tr_b8 v[64:65], v56 offset:32768\n\t"
    "ds_read_b64_tr_b8 v[66:67], v56 offset:33792\n\t"
    "v_readlane_b32 s77, %[tw], 17\n\t"
    "s_mov_b32 s73, s77\n\t"
    "s_add_u32 m0, %[m0w], 16384\n\t"
    "s_nop 0\n\t"
    "buffer_load_dwordx4 %[wvoff], %[srdW], s73 offen lds\n\t"
    "v_mfma_f32_16x16x128_f8f6f4 v[240:243], v[96:103], v[152:159], v[240:243] cbsz:1\n\t"
    "ds_read_b64_tr_b8 v[68:69], v56 offset:40960\n\t"
    "ds_read_b64_tr_b8 v[70:71], v56 offset:41984\n\t"
    "s_waitcnt lgkmcnt(12)\n\t"
    "v_mfma_f32_16x16x128_f8f6f4 v[212:215], v[104:111], v[144:151], v[212:215] cbsz:1\n\t"
    "ds_read_b64_tr_b8 v[72:73], v57 offset:32768\n\t"
    "ds_read_b64_tr_b8 v[74:75], v57 offset:33792\n\t"
    "s_add_u32 m0, %[m0w], 17408\n\t"
    "s_add_u32 s73, s73, 131072\n\t"
    "buffer_load_dwordx4 %[wvoff], %[srdW], s73 offen lds\n\t"
    "v_mfma_f32_16x16x128_f8f6f4 v[244:247], v[104:111], v[152:159], v[244:247] cbsz:1\n\t"
    "ds_read_b64_tr_b8 v[76:77], v57 offset:40960\n\t"
    "ds_read_b64_tr_b8 v[78:79], v57 offset:41984\n\t"
    "s_waitcnt vmcnt(9)\n\t"
    "v_min3_u32 v53, v160, v161, v162\n\t"
    "v_min3_u32 v54, v163, v164, v165\n\t"
    "s_waitcnt lgkmcnt(12)\n\t"
    "v_mfma_f32_16x16x128_f8f6f4 v[216:219], v[112:119], v[144:151], v[216:219] cbsz:1\n\t"
    "ds_read_b64_tr_b8 v[80:81], v58 offset:32768\n\t"
    "ds_read_b64_tr_b8 v[82:83], v58 offset:33792\n\t"
    "v_min3_u32 v53, v53, v166, v167\n\t"
    "v_min3_u32 v54, v54, v168, v169\n\t"
    "v_mfma_f32_16x16x128_f8f6f4 v[248:251], v[112:119], v[152:159], v[248:251] cbsz:1\n\t"
    "ds_read_b64_tr_b8 v[84:85], v58 offset:40960\n\t"
    "ds_read_b64_tr_b8 v[86:87], v58 offset:41984\n\t"
    "v_min3_u32 v53, v53, v170, v171\n\t"
    "v_min3_u32 v54, v54, v172, v173\n\t"
    "s_waitcnt lgkmcnt(12)\n\t"
    "v_mfma_f32_16x16x128_f8f6f4 v[220:223], v[120:127], v[144:151], v[220:223] cbsz:1\n\t"
    "ds_read_b64_tr_b8 v[88:89], v59 offset:32768\n\t"
    "ds_read_b64_tr_b8 v[90:91], v59 offset:33792\n\t"
    "v_min3_u32 v53, v53, v174, v175\n\t"
    "v_min_u32 v53, v53, v54\n\t"
    "v_mfma_f32_16x16x128_f8f6f4 v[252:255], v[120:127], v[152:159], v[252:255] cbsz:1\n\t"
    "ds_read_b64_tr_b8 v[92:93], v59 offset:40960\n\t"
    "ds_read_b64_tr_b8 v[94:95], v59 offset:41984\n\t"
    "v_cmp_eq_u32 vcc, 0, v53\n\t"
    "s_cbranch_vccz L_ok14_%=\n\t"
    "v_readlane_b32 s77, %[tx], 14\n\t"
    "s_mov_b32 s82, s77\n\t"
    "s_getpc_b64 s[58:59]\n\t"
    "s_branch L_slow2_%=\n\t"
    "L_ok14_%=:\n\t"
    "s_waitcnt lgkmcnt(12)\n\t"
    "v_mfma_f32_16x16x128_f8f6f4 v[192:195], v[64:71], v[160:167], v[192:195] cbsz:1\n\t"
    "ds_read_b64_tr_b8 v[96:97], v60 offset:32768\n\t"
    "ds_read_b64_tr_b8 v[98:99], v60 offset:33792\n\t"
    "v_mfma_f32_16x16x128_f8f6f4 v[224:227], v[64:71], v[168:175], v[224:227] cbsz:1\n\t"
    "ds_read_b64_tr_b8 v[100:101], v60 offset:40960\n\t"
    "ds_read_b64_tr_b8 v[102:103], v60 offset:41984\n\t"
    "v_readlane_b32 s77, %[tx], 16\n\t"
    "s_mov_b32 s75, s77\n\t"
    "buffer_load_dwordx4 v[128:131], %[xvoff], s[60:63], s75 offen offset:0 sc0 sc1\n\t"
    "s_waitcnt lgkmcnt(12)\n\t"
    "v_mfma_f32_16x16x128_f8f6f4 v[196:199], v[72:79], v[160:167], v[196:199] cbsz:1\n\t"
    "ds_read_b64_tr_b8 v[104:105], v61 offset:32768\n\t"
    "ds_read_b64_tr_b8 v[106:107], v61 offset:33792\n\t"
    "buffer_load_dwordx4 v[132:135], %[xvoff], s[60:63], s75 offen offset:1024 sc0 sc1\n\t"
    "v_mfma_f32_16x16x128_f8f6f4 v[228:231], v[72:79], v[168:175], v[228:231] cbsz:1\n\t"
    "ds_read_b64_tr_b8 v[108:109], v61 offset:40960\n\t"
    "ds_read_b64_tr_b8 v[110:111], v61 offset:41984\n\t"
    "s_waitcnt lgkmcnt(12)\n\t"
    "v_mfma_f32_16x16x128_f8f6f4 v[200:203], v[80:87], v[160:167], v[200:203] cbsz:1\n\t"
    "ds_read_b64_tr_b8 v[112:113], v62 offset:32768\n\t"
    "ds_read_b64_tr_b8 v[114:115], v62 offset:33792\n\t"
    "buffer_load_dwordx4 v[136:139], %[xvoff], s[60:63], s75 offen offset:2048 sc0 sc1\n\t"
    "v_mfma_f32_16x16x128_f8f6f4 v[232:235], v[80:87], v[168:175], v[232:235] cbsz:1\n\t"
    "ds_read_b64_tr_b8 v[116:117], v62 offset:40960\n\t"
    "ds_read_b64_tr_b8 v[118:119], v62 offset:41984\n\t"
    "s_waitcnt lgkmcnt(12)\n\t"
    "v_mfma_f32_16x16x128_f8f6f4 v[204:207], v[88:95], v[160:167], v[204:207] cbsz:1\n\t"
    "ds_read_b64_tr_b8 v[120:121], v63 offset:32768\n\t"
    "ds_read_b64_tr_b8 v[122:123], v63 offset:33792\n\t"
    "buffer_load_dwordx4 v[140:143], %[xvoff], s[60:63], s75 offen offset:3072 sc0 sc1\n\t"
    "s_cmp_eq_u32 %[cpfw], 63\n\t"
    "s_cbranch_scc0 L_ccp2_%=\n\t"
    "s_mov_b32 s77, 16384\n\t"
    "buffer_load_dword v46, %[cvoff], s[84:87], s77 offen sc1\n\t"
    "s_branch L_dcp2_%=\n\t"
    "L_ccp2_%=:\n\t"
    "buffer_load_dword v46, v55, %[srdM], s79 offen\n\t"
    "L_dcp2_%=:\n\t"
    "v_mfma_f32_16x16x128_f8f6f4 v[236:239], v[88:95], v[168:175], v[236:239] cbsz:1\n\t"
    "ds_read_b64_tr_b8 v[124:125], v63 offset:40960\n\t"
    "ds_read_b64_tr_b8 v[126:127], v63 offset:41984\n\t"
    "s_waitcnt vmcnt(11)\n\t"
    "s_barrier\n\t"
    "s_waitcnt lgkmcnt(12)\n\t"
    "v_mfma_f32_16x16x128_f8f6f4 v[208:211], v[96:103], v[160:167], v[208:211] cbsz:1\n\t"
    "ds_read_b64_tr_b8 v[64:65], v56 offset:49152\n\t"
    "ds_read_b64_tr_b8 v[66:67], v56 offset:50176\n\t"
    "v_readlane_b32 s77, %[tw], 18\n\t"
    "s_mov_b32 s73, s77\n\t"
    "s_add_u32 m0, %[m0w], 32768\n\t"
    "s_nop 0\n\t"
    "buffer_load_dwordx4 %[wvoff], %[srdW], s73 offen lds\n\t"
    "v_mfma_f32_16x16x128_f8f6f4 v[240:243], v[96:103], v[168:175], v[240:243] cbsz:1\n\t"
    "ds_read_b64_tr_b8 v[68:69], v56 offset:57344\n\t"
    "ds_read_b64_tr_b8 v[70:71], v56 offset:58368\n\t"
    "s_waitcnt lgkmcnt(12)\n\t"
    "v_mfma_f32_16x16x128_f8f6f4 v[212:215], v[104:111], v[160:167], v[212:215] cbsz:1\n\t"
    "ds_read_b64_tr_b8 v[72:73], v57 offset:49152\n\t"
    "ds_read_b64_tr_b8 v[74:75], v57 offset:50176\n\t"
    "s_add_u32 m0, %[m0w], 33792\n\t"
    "s_add_u32 s73, s73, 131072\n\t"
    "buffer_load_dwordx4 %[wvoff], %[srdW], s73 offen lds\n\t"
    "v_mfma_f32_16x16x128_f8f6f4 v[244:247], v[104:111], v[168:175], v[244:247] cbsz:1\n\t"
    "ds_read_b64_tr_b8 v[76:77], v57 offset:57344\n\t"
    "ds_read_b64_tr_b8 v[78:79], v57 offset:58368\n\t"
    "s_waitcnt vmcnt(9)\n\t"
    "v_min3_u32 v53, v176, v177, v178\n\t"
    "v_min3_u32 v54, v179, v180, v181\n\t"
    "s_waitcnt lgkmcnt(12)\n\t"
    "v_mfma_f32_16x16x128_f8f6f4 v[216:219], v[112:119], v[160:167], v[216:219] cbsz:1\n\t"
    "ds_read_b64_tr_b8 v[80:81], v58 offset:49152\n\t"
    "ds_read_b64_tr_b8 v[82:83], v58 offset:50176\n\t"
    "v_readlane_b32 s77, %[tw], 30\n\t"
    "s_mov_b32 s73, s77\n\t"
    "s_add_u32 m0, %[m0w], 131072\n\t"
    "s_nop 0\n\t"
    "buffer_load_dwordx4 %[wvoff], %[srdW], s73 offen lds\n\t"
    "v_min3_u32 v53, v53, v182, v183\n\t"
    "v_min3_u32 v54, v54, v184, v185\n\t"
    "v_mfma_f32_16x16x128_f8f6f4 v[248:251], v[112:119], v[168:175], v[248:251] cbsz:1\n\t"
    "ds_read_b64_tr_b8 v[84:85], v58 offset:57344\n\t"
    "ds_read_b64_tr_b8 v[86:87], v58 offset:58368\n\t"
    "v_min3_u32 v53, v53, v186, v187\n\t"
    "v_min3_u32 v54, v54, v188, v189\n\t"
    "s_waitcnt lgkmcnt(12)\n\t"
    "v_mfma_f32_16x16x128_f8f6f4 v[220:223], v[120:127], v[160:167], v[220:223] cbsz:1\n\t"
    "ds_read_b64_tr_b8 v[88:89], v59 offset:49152\n\t"
    "ds_read_b64_tr_b8 v[90:91], v59 offset:50176\n\t"
    "s_add_u32 m0, %[m0w], 132096\n\t"
    "s_add_u32 s73, s73, 131072\n\t"
    "buffer_load_dwordx4 %[wvoff], %[srdW], s73 offen lds\n\t"
    "v_min3_u32 v53, v53, v190, v191\n\t"
    "v_min_u32 v53, v53, v54\n\t"
    "v_mfma_f32_16x16x128_f8f6f4 v[252:255], v[120:127], v[168:175], v[252:255] cbsz:1\n\t"
    "ds_read_b64_tr_b8 v[92:93], v59 offset:57344\n\t"
    "ds_read_b64_tr_b8 v[94:95], v59 offset:58368\n\t"
    "v_cmp_eq_u32 vcc, 0, v53\n\t"
    "s_cbranch_vccz L_ok15_%=\n\t"
    "v_readlane_b32 s77, %[tx], 15\n\t"
    "s_mov_b32 s82, s77\n\t"
    "s_getpc_b64 s[58:59]\n\t"
    "s_branch L_slow3_%=\n\t"
    "L_ok15_%=:\n\t"
    "s_waitcnt lgkmcnt(12)\n\t"
    "v_mfma_f32_16x16x128_f8f6f4 v[192:195], v[64:71], v[176:183], v[192:195] cbsz:1\n\t"
    "ds_read_b64_tr_b8 v[96:97], v60 offset:49152\n\t"
    "ds_read_b64_tr_b8 v[98:99], v60 offset:50176\n\t"
    "v_xor_b32 v56, 0x10000, v56\n\t"
    "v_mfma_f32_16x16x128_f8f6f4 v[224:227], v[64:71], v[184:191], v[224:227] cbsz:1\n\t"
    "ds_read_b64_tr_b8 v[100:101], v60 offset:57344\n\t"
    "ds_read_b64_tr_b8 v[102:103], v60 offset:58368\n\t"
    "v_readlane_b32 s77, %[tx], 17\n\t"
    "s_mov_b32 s75, s77\n\t"
    "buffer_load_dwordx4 v[144:147], %[xvoff], s[60:63], s75 offen offset:0 sc0 sc1\n\t"
    "v_xor_b32 v57, 0x10000, v57\n\t"
    "s_waitcnt lgkmcnt(12)\n\t"
    "v_mfma_f32_16x16x128_f8f6f4 v[196:199], v[72:79], v[176:183], v[196:199] cbsz:1\n\t"
    "ds_read_b64_tr_b8 v[104:105], v61 offset:49152\n\t"
    "ds_read_b64_tr_b8 v[106:107], v61 offset:50176\n\t"
    "buffer_load_dwordx4 v[148:151], %[xvoff], s[60:63], s75 offen offset:1024 sc0 sc1\n\t"
    "v_xor_b32 v58, 0x10000, v58\n\t"
    "v_mfma_f32_16x16x128_f8f6f4 v[228:231], v[72:79], v[184:191], v[228:231] cbsz:1\n\t"
    "ds_read_b64_tr_b8 v[108:109], v61 offset:57344\n\t"
    "ds_read_b64_tr_b8 v[110:111], v61 offset:58368\n\t"
    "v_xor_b32 v59, 0x10000, v59\n\t"
    "s_waitcnt lgkmcnt(12)\n\t"
    "v_mfma_f32_16x16x128_f8f6f4 v[200:203], v[80:87], v[176:183], v[200:203] cbsz:1\n\t"
    "ds_read_b64_tr_b8 v[112:113], v62 offset:49152\n\t"
    "ds_read_b64_tr_b8 v[114:115], v62 offset:50176\n\t"
    "buffer_load_dwordx4 v[152:155], %[xvoff], s[60:63], s75 offen offset:2048 sc0 sc1\n\t"
    "v_mfma_f32_16x16x128_f8f6f4 v[232:235], v[80:87], v[184:191], v[232:235] cbsz:1\n\t"
    "ds_read_b64_tr_b8 v[116:117], v62 offset:57344\n\t"
    "ds_read_b64_tr_b8 v[118:119], v62 offset:58368\n\t"
    "s_waitcnt lgkmcnt(12)\n\t"
    "v_mfma_f32_16x16x128_f8f6f4 v[204:207], v[88:95], v[176:183], v[204:207] cbsz:1\n\t"
    "ds_read_b64_tr_b8 v[120:121], v63 offset:49152\n\t"
    "ds_read_b64_tr_b8 v[122:123], v63 offset:50176\n\t"
    "buffer_load_dwordx4 v[156:159], %[xvoff], s[60:63], s75 offen offset:3072 sc0 sc1\n\t"
    "v_mfma_f32_16x16x128_f8f6f4 v[236:239], v[88:95], v[184:191], v[236:239] cbsz:1\n\t"
    "ds_read_b64_tr_b8 v[124:125], v63 offset:57344\n\t"
    "ds_read_b64_tr_b8 v[126:127], v63 offset:58368\n\t"
    "s_waitcnt lgkmcnt(12)\n\t"
    "v_mfma_f32_16x16x128_f8f6f4 v[208:211], v[96:103], v[176:183], v[208:211] cbsz:1\n\t"
    "ds_read_b64_tr_b8 v[64:65], v56 offset:0\n\t"
    "ds_read_b64_tr_b8 v[66:67], v56 offset:1024\n\t"
    "v_readlane_b32 s77, %[tw], 19\n\t"
    "s_mov_b32 s73, s77\n\t"
    "s_add_u32 m0, %[m0w], 49152\n\t"
    "s_nop 0\n\t"
    "buffer_load_dwordx4 %[wvoff], %[srdW], s73 offen lds\n\t"
    "v_xor_b32 v60, 0x10000, v60\n\t"
    "v_mfma_f32_16x16x128_f8f6f4 v[240:243], v[96:103], v[184:191], v[240:243] cbsz:1\n\t"
    "ds_read_b64_tr_b8 v[68:69], v56 offset:8192\n\t"
    "ds_read_b64_tr_b8 v[70:71], v56 offset:9216\n\t"
    "v_xor_b32 v61, 0x10000, v61\n\t"
    "s_waitcnt lgkmcnt(12)\n\t"
    "v_mfma_f32_16x16x128_f8f6f4 v[212:215], v[104:111], v[176:183], v[212:215] cbsz:1\n\t"
    "ds_read_b64_tr_b8 v[72:73], v57 offset:0\n\t"
    "ds_read_b64_tr_b8 v[74:75], v57 offset:1024\n\t"
    "s_add_u32 m0, %[m0w], 50176\n\t"
    "s_add_u32 s73, s73, 131072\n\t"
    "buffer_load_dwordx4 %[wvoff], %[srdW], s73 offen lds\n\t"
    "v_xor_b32 v62, 0x10000, v62\n\t"
    "v_mfma_f32_16x16x128_f8f6f4 v[244:247], v[104:111], v[184:191], v[244:247] cbsz:1\n\t"
    "ds_read_b64_tr_b8 v[76:77], v57 offset:8192\n\t"
    "ds_read_b64_tr_b8 v[78:79], v57 offset:9216\n\t"
    "s_waitcnt vmcnt(11)\n\t"
    "v_min3_u32 v53, v128, v129, v130\n\t"
    "v_min3_u32 v54, v131, v132, v133\n\t"
    "v_xor_b32 v63, 0x10000, v63\n\t"
    "s_waitcnt lgkmcnt(12)\n\t"
    "v_mfma_f32_16x16x128_f8f6f4 v[216:219], v[112:119], v[176:183], v[216:219] cbsz:1\n\t"
    "ds_read_b64_tr_b8 v[80:81], v58 offset:0\n\t"
    "ds_read_b64_tr_b8 v[82:83], v58 offset:1024\n\t"
    "v_min3_u32 v53, v53, v134, v135\n\t"
    "v_min3_u32 v54, v54, v136, v137\n\t"
    "v_mfma_f32_16x16x128_f8f6f4 v[248:251], v[112:119], v[184:191], v[248:251] cbsz:1\n\t"
    "ds_read_b64_tr_b8 v[84:85], v58 offset:8192\n\t"
    "ds_read_b64_tr_b8 v[86:87], v58 offset:9216\n\t"
    "v_min3_u32 v53, v53, v138, v139\n\t"
    "v_min3_u32 v54, v54, v140, v141\n\t"
    "s_waitcnt lgkmcnt(12)\n\t"
    "v_mfma_f32_16x16x128_f8f6f4 v[220:223], v[120:127], v[176:183], v[220:223] cbsz:1\n\t"
    "ds_read_b64_tr_b8 v[88:89], v59 offset:0\n\t"
    "ds_read_b64_tr_b8 v[90:91], v59 offset:1024\n\t"
    "v_min3_u32 v53, v53, v142, v143\n\t"
    "v_min_u32 v53, v53, v54\n\t"
    "v_mfma_f32_16x16x128_f8f6f4 v[252:255], v[120:127], v[184:191], v[252:255] cbsz:1\n\t"
    "ds_read_b64_tr_b8 v[92:93], v59 offset:8192\n\t"
    "ds_read_b64_tr_b8 v[94:95], v59 offset:9216\n\t"
    "v_cmp_eq_u32 vcc, 0, v53\n\t"
    "s_cbranch_vccz L_ok16_%=\n\t"
    "v_readlane_b32 s77, %[tx], 16\n\t"
    "s_mov_b32 s82, s77\n\t"
    "s_getpc_b64 s[58:59]\n\t"
    "s_branch L_slow0_%=\n\t"
    "L_ok16_%=:\n\t"
    "s_waitcnt lgkmcnt(12)\n\t"
    "v_mfma_f32_16x16x128_f8f6f4 v[192:195], v[64:71], v[128:135], v[192:195] cbsz:1\n\t"
    "ds_read_b64_tr_b8 v[96:97], v60 offset:0\n\t"
    "ds_read_b64_tr_b8 v[98:99], v60 offset:1024\n\t"
    "v_mfma_f32_16x16x128_f8f6f4 v[224:227], v[64:71], v[136:143], v[224:227] cbsz:1\n\t"
    "ds_read_b64_tr_b8 v[100:101], v60 offset:8192\n\t"
    "ds_read_b64_tr_b8 v[102:103], v60 offset:9216\n\t"
    "v_readlane_b32 s77, %[tx], 18\n\t"
    "s_mov_b32 s75, s77\n\t"
    "buffer_load_dwordx4 v[160:163], %[xvoff], s[60:63], s75 offen offset:0 sc0 sc1\n\t"
    "s_waitcnt lgkmcnt(12)\n\t"
    "v_mfma_f32_16x16x128_f8f6f4 v[196:199], v[72:79], v[128:135], v[196:199] cbsz:1\n\t"
    "ds_read_b64_tr_b8 v[104:105], v61 offset:0\n\t"
    "ds_read_b64_tr_b8 v[106:107], v61 offset:1024\n\t"
    "buffer_load_dwordx4 v[164:167], %[xvoff], s[60:63], s75 offen offset:1024 sc0 sc1\n\t"
    "v_mfma_f32_16x16x128_f8f6f4 v[228:231], v[72:79], v[136:143], v[228:231] cbsz:1\n\t"
    "ds_read_b64_tr_b8 v[108:109], v61 offset:8192\n\t"
    "ds_read_b64_tr_b8 v[110:111], v61 offset:9216\n\t"
    "s_waitcnt lgkmcnt(12)\n\t"
    "v_mfma_f32_16x16x128_f8f6f4 v[200:203], v[80:87], v[128:135], v[200:203] cbsz:1\n\t"
    "ds_read_b64_tr_b8 v[112:113], v62 offset:0\n\t"
    "ds_read_b64_tr_b8 v[114:115], v62 offset:1024\n\t"
    "buffer_load_dwordx4 v[168:171], %[xvoff], s[60:63], s75 offen offset:2048 sc0 sc1\n\t"
    "v_mfma_f32_16x16x128_f8f6f4 v[232:235], v[80:87], v[136:143], v[232:235] cbsz:1\n\t"
    "ds_read_b64_tr_b8 v[116:117], v62 offset:8192\n\t"
    "ds_read_b64_tr_b8 v[118:119], v62 offset:9216\n\t"
    "s_waitcnt lgkmcnt(12)\n\t"
    "v_mfma_f32_16x16x128_f8f6f4 v[204:207], v[88:95], v[128:135], v[204:207] cbsz:1\n\t"
    "ds_read_b64_tr_b8 v[120:121], v63 offset:0\n\t"
    "ds_read_b64_tr_b8 v[122:123], v63 offset:1024\n\t"
    "buffer_load_dwordx4 v[172:175], %[xvoff], s[60:63], s75 offen offset:3072 sc0 sc1\n\t"
    "v_mfma_f32_16x16x128_f8f6f4 v[236:239], v[88:95], v[136:143], v[236:239] cbsz:1\n\t"
    "ds_read_b64_tr_b8 v[124:125], v63 offset:8192\n\t"
    "ds_read_b64_tr_b8 v[126:127], v63 offset:9216\n\t"
    "s_waitcnt vmcnt(12)\n\t"
    "s_barrier\n\t"
    "s_waitcnt lgkmcnt(12)\n\t"
    "v_mfma_f32_16x16x128_f8f6f4 v[208:211], v[96:103], v[128:135], v[208:211] cbsz:1\n\t"
    "ds_read_b64_tr_b8 v[64:65], v56 offset:16384\n\t"
    "ds_read_b64_tr_b8 v[66:67], v56 offset:17408\n\t"
    "v_readlane_b32 s77, %[tw], 20\n\t"
    "s_mov_b32 s73, s77\n\t"
    "s_add_u32 m0, %[m0w], 65536\n\t"
    "s_nop 0\n\t"
    "buffer_load_dwordx4 %[wvoff], %[srdW], s73 offen lds\n\t"
    "v_mfma_f32_16x16x128_f8f6f4 v[240:243], v[96:103], v[136:143], v[240:243] cbsz:1\n\t"
    "ds_read_b64_tr_b8 v[68:69], v56 offset:24576\n\t"
    "ds_read_b64_tr_b8 v[70:71], v56 offset:25600\n\t"
    "s_cmp_eq_u32 %[pfw], 7\n\t"
    "s_cbranch_scc0 L_cpf7_%=\n\t"
    "buffer_load_dword v46, %[pfvoff], %[srdPF], %[pfso] offen sc1\n\t"
    "s_branch L_dpf7_%=\n\t"
    "L_cpf7_%=:\n\t"
    "buffer_load_dword v46, v55, %[srdM], s79 offen\n\t"
    "L_dpf7_%=:\n\t"
    "s_waitcnt lgkmcnt(12)\n\t"
    "v_mfma_f32_16x16x128_f8f6f4 v[212:215], v[104:111], v[128:135], v[212:215] cbsz:1\n\t"
    "ds_read_b64_tr_b8 v[72:73], v57 offset:16384\n\t"
    "ds_read_b64_tr_b8 v[74:75], v57 offset:17408\n\t"
    "s_add_u32 m0, %[m0w], 66560\n\t"
    "s_add_u32 s73, s73, 131072\n\t"
    "buffer_load_dwordx4 %[wvoff], %[srdW], s73 offen lds\n\t"
    "v_mfma_f32_16x16x128_f8f6f4 v[244:247], v[104:111], v[136:143], v[244:247] cbsz:1\n\t"
    "ds_read_b64_tr_b8 v[76:77], v57 offset:24576\n\t"
    "ds_read_b64_tr_b8 v[78:79], v57 offset:25600\n\t"
    "s_waitcnt vmcnt(9)\n\t"
    "v_min3_u32 v53, v144, v145, v146\n\t"
    "v_min3_u32 v54, v147, v148, v149\n\t"
    "s_waitcnt lgkmcnt(12)\n\t"
    "v_mfma_f32_16x16x128_f8f6f4 v[216:219], v[112:119], v[128:135], v[216:219] cbsz:1\n\t"
    "ds_read_b64_tr_b8 v[80:81], v58 offset:16384\n\t"
    "ds_read_b64_tr_b8 v[82:83], v58 offset:17408\n\t"
    "v_readlane_b32 s77, %[tw], 31\n\t"
    "s_mov_b32 s73, s77\n\t"
    "s_add_u32 m0, %[m0w], 147456\n\t"
    "s_nop 0\n\t"
    "buffer_load_dwordx4 %[wvoff], %[srdW], s73 offen lds\n\t"
    "v_min3_u32 v53, v53, v150, v151\n\t"
    "v_min3_u32 v54, v54, v152, v153\n\t"
    "v_mfma_f32_16x16x128_f8f6f4 v[248:251], v[112:119], v[136:143], v[248:251] cbsz:1\n\t"
    "ds_read_b64_tr_b8 v[84:85], v58 offset:24576\n\t"
    "ds_read_b64_tr_b8 v[86:87], v58 offset:25600\n\t"
    "v_min3_u32 v53, v53, v154, v155\n\t"
    "v_min3_u32 v54, v54, v156, v157\n\t"
    "s_waitcnt lgkmcnt(12)\n\t"
    "v_mfma_f32_16x16x128_f8f6f4 v[220:223], v[120:127], v[128:135], v[220:223] cbsz:1\n\t"
    "ds_read_b64_tr_b8 v[88:89], v59 offset:16384\n\t"
    "ds_read_b64_tr_b8 v[90:91], v59 offset:17408\n\t"
    "s_add_u32 m0, %[m0w], 148480\n\t"
    "s_add_u32 s73, s73, 131072\n\t"
    "buffer_load_dwordx4 %[wvoff], %[srdW], s73 offen lds\n\t"
    "v_min3_u32 v53, v53, v158, v159\n\t"
    "v_min_u32 v53, v53, v54\n\t"
    "v_mfma_f32_16x16x128_f8f6f4 v[252:255], v[120:127], v[136:143], v[252:255] cbsz:1\n\t"
    "ds_read_b64_tr_b8 v[92:93], v59 offset:24576\n\t"
    "ds_read_b64_tr_b8 v[94:95], v59 offset:25600\n\t"
    "v_cmp_eq_u32 vcc, 0, v53\n\t"
    "s_cbranch_vccz L_ok17_%=\n\t"
    "v_readlane_b32 s77, %[tx], 17\n\t"
    "s_mov_b32 s82, s77\n\t"
    "s_getpc_b64 s[58:59]\n\t"
    "s_branch L_slow1_%=\n\t"
    "L_ok17_%=:\n\t"
    "s_waitcnt lgkmcnt(12)\n\t"
    "v_mfma_f32_16x16x128_f8f6f4 v[192:195], v[64:71], v[144:151], v[192:195] cbsz:1\n\t"
    "ds_read_b64_tr_b8 v[96:97], v60 offset:16384\n\t"
    "ds_read_b64_tr_b8 v[98:99], v60 offset:17408\n\t"
    "s_cmp_eq_u32 %[wid], 7\n\t"
    "s_cbranch_scc1 L_z7%=\n\t"
    "v_mov_b32 v48, 0\n\t"
    "v_mov_b32 v49, 0\n\t"
    "v_mov_b32 v50, 0\n\t"
    "v_mov_b32 v51, 0\n\t"
    "buffer_store_dwordx4 v[48:51], %[xvoff], %[srdM], s72 offen sc0 sc1\n\t"
    "s_branch L_z8%=\n\t"
    "L_z7%=:\n\t"
    "v_mov_b32 v48, s68\n\t"
    "buffer_store_dword v48, v55, %[srdM], %[ctroff] offen sc0 sc1\n\t"
    "L_z8%=:\n\t"
    "v_mfma_f32_16x16x128_f8f6f4 v[224:227], v[64:71], v[152:159], v[224:227] cbsz:1\n\t"
    "ds_read_b64_tr_b8 v[100:101], v60 offset:24576\n\t"
    "ds_read_b64_tr_b8 v[102:103], v60 offset:25600\n\t"
    "v_readlane_b32 s77, %[tx], 19\n\t"
    "s_mov_b32 s75, s77\n\t"
    "buffer_load_dwordx4 v[176:179], %[xvoff], s[60:63], s75 offen offset:0 sc0 sc1\n\t"
    "s_waitcnt lgkmcnt(12)\n\t"
    "v_mfma_f32_16x16x128_f8f6f4 v[196:199], v[72:79], v[144:151], v[196:199] cbsz:1\n\t"
    "ds_read_b64_tr_b8 v[104:105], v61 offset:16384\n\t"
    "ds_read_b64_tr_b8 v[106:107], v61 offset:17408\n\t"
    "buffer_load_dwordx4 v[180:183], %[xvoff], s[60:63], s75 offen offset:1024 sc0 sc1\n\t"
    "v_mfma_f32_16x16x128_f8f6f4 v[228:231], v[72:79], v[152:159], v[228:231] cbsz:1\n\t"
    "ds_read_b64_tr_b8 v[108:109], v61 offset:24576\n\t"
    "ds_read_b64_tr_b8 v[110:111], v61 offset:25600\n\t"
    "s_waitcnt lgkmcnt(12)\n\t"
    "v_mfma_f32_16x16x128_f8f6f4 v[200:203], v[80:87], v[144:151], v[200:203] cbsz:1\n\t"
    "ds_read_b64_tr_b8 v[112:113], v62 offset:16384\n\t"
    "ds_read_b64_tr_b8 v[114:115], v62 offset:17408\n\t"
    "buffer_load_dwordx4 v[184:187], %[xvoff], s[60:63], s75 offen offset:2048 sc0 sc1\n\t"
    "v_mfma_f32_16x16x128_f8f6f4 v[232:235], v[80:87], v[152:159], v[232:235] cbsz:1\n\t"
    "ds_read_b64_tr_b8 v[116:117], v62 offset:24576\n\t"
    "ds_read_b64_tr_b8 v[118:119], v62 offset:25600\n\t"
    "s_waitcnt lgkmcnt(12)\n\t"
    "v_mfma_f32_16x16x128_f8f6f4 v[204:207], v[88:95], v[144:151], v[204:207] cbsz:1\n\t"
    "ds_read_b64_tr_b8 v[120:121], v63 offset:16384\n\t"
    "ds_read_b64_tr_b8 v[122:123], v63 offset:17408\n\t"
    "buffer_load_dwordx4 v[188:191], %[xvoff], s[60:63], s75 offen offset:3072 sc0 sc1\n\t"
    "v_mfma_f32_16x16x128_f8f6f4 v[236:239], v[88:95], v[152:159], v[236:239] cbsz:1\n\t"
    "ds_read_b64_tr_b8 v[124:125], v63 offset:24576\n\t"
    "ds_read_b64_tr_b8 v[126:127], v63 offset:25600\n\t"
    "s_waitcnt lgkmcnt(12)\n\t"
    "v_mfma_f32_16x16x128_f8f6f4 v[208:211], v[96:103], v[144:151], v[208:211] cbsz:1\n\t"
    "ds_read_b64_tr_b8 v[64:65], v56 offset:32768\n\t"
    "ds_read_b64_tr_b8 v[66:67], v56 offset:33792\n\t"
    "v_readlane_b32 s77, %[tw], 21\n\t"
    "s_mov_b32 s73, s77\n\t"
    "s_add_u32 m0, %[m0w], 81920\n\t"
    "s_nop 0\n\t"
    "buffer_load_dwordx4 %[wvoff], %[srdW], s73 offen lds\n\t"
    "v_mfma_f32_16x16x128_f8f6f4 v[240:243], v[96:103], v[152:159], v[240:243] cbsz:1\n\t"
    "ds_read_b64_tr_b8 v[68:69], v56 offset:40960\n\t"
    "ds_read_b64_tr_b8 v[70:71], v56 offset:41984\n\t"
    "s_waitcnt lgkmcnt(12)\n\t"
    "v_mfma_f32_16x16x128_f8f6f4 v[212:215], v[104:111], v[144:151], v[212:215] cbsz:1\n\t"
    "ds_read_b64_tr_b8 v[72:73], v57 offset:32768\n\t"
    "ds_read_b64_tr_b8 v[74:75], v57 offset:33792\n\t"
    "s_add_u32 m0, %[m0w], 82944\n\t"
    "s_add_u32 s73, s73, 131072\n\t"
    "buffer_load_dwordx4 %[wvoff], %[srdW], s73 offen lds\n\t"
    "v_mfma_f32_16x16x128_f8f6f4 v[244:247], v[104:111], v[152:159], v[244:247] cbsz:1\n\t"
    "ds_read_b64_tr_b8 v[76:77], v57 offset:40960\n\t"
    "ds_read_b64_tr_b8 v[78:79], v57 offset:41984\n\t"
    "s_waitcnt vmcnt(12)\n\t"
    "v_min3_u32 v53, v160, v161, v162\n\t"
    "v_min3_u32 v54, v163, v164, v165\n\t"
    "s_waitcnt lgkmcnt(12)\n\t"
    "v_mfma_f32_16x16x128_f8f6f4 v[216:219], v[112:119], v[144:151], v[216:219] cbsz:1\n\t"
    "ds_read_b64_tr_b8 v[80:81], v58 offset:32768\n\t"
    "ds_read_b64_tr_b8 v[82:83], v58 offset:33792\n\t"
    "v_min3_u32 v53, v53, v166, v167\n\t"
    "v_min3_u32 v54, v54, v168, v169\n\t"
    "v_mfma_f32_16x16x128_f8f6f4 v[248:251], v[112:119], v[152:159], v[248:251] cbsz:1\n\t"
    "ds_read_b64_tr_b8 v[84:85], v58 offset:40960\n\t"
    "ds_read_b64_tr_b8 v[86:87], v58 offset:41984\n\t"
    "v_min3_u32 v53, v53, v170, v171\n\t"
    "v_min3_u32 v54, v54, v172, v173\n\t"
    "s_waitcnt lgkmcnt(12)\n\t"
    "v_mfma_f32_16x16x128_f8f6f4 v[220:223], v[120:127], v[144:151], v[220:223] cbsz:1\n\t"
    "ds_read_b64_tr_b8 v[88:89], v59 offset:32768\n\t"
    "ds_read_b64_tr_b8 v[90:91], v59 offset:33792\n\t"
    "v_min3_u32 v53, v53, v174, v175\n\t"
    "v_min_u32 v53, v53, v54\n\t"
    "v_mfma_f32_16x16x128_f8f6f4 v[252:255], v[120:127], v[152:159], v[252:255] cbsz:1\n\t"
    "ds_read_b64_tr_b8 v[92:93], v59 offset:40960\n\t"
    "ds_read_b64_tr_b8 v[94:95], v59 offset:41984\n\t"
    "v_cmp_eq_u32 vcc, 0, v53\n\t"
    "s_cbranch_vccz L_ok18_%=\n\t"
    "v_readlane_b32 s77, %[tx], 18\n\t"
    "s_mov_b32 s82, s77\n\t"
    "s_getpc_b64 s[58:59]\n\t"
    "s_branch L_slow2_%=\n\t"
    "L_ok18_%=:\n\t"
    "s_waitcnt lgkmcnt(12)\n\t"
    "v_mfma_f32_16x16x128_f8f6f4 v[192:195], v[64:71], v[160:167], v[192:195] cbsz:1\n\t"
    "ds_read_b64_tr_b8 v[96:97], v60 offset:32768\n\t"
    "ds_read_b64_tr_b8 v[98:99], v60 offset:33792\n\t"
    "v_mfma_f32_16x16x128_f8f6f4 v[224:227], v[64:71], v[168:175], v[224:227] cbsz:1\n\t"
    "ds_read_b64_tr_b8 v[100:101], v60 offset:40960\n\t"
    "ds_read_b64_tr_b8 v[102:103], v60 offset:41984\n\t"
    "v_readlane_b32 s77, %[tx], 20\n\t"
    "s_mov_b32 s75, s77\n\t"
    "buffer_load_dwordx4 v[128:131], %[xvoff], s[60:63], s75 offen offset:0 sc0 sc1\n\t"
    "s_waitcnt lgkmcnt(12)\n\t"
    "v_mfma_f32_16x16x128_f8f6f4 v[196:199], v[72:79], v[160:167], v[196:199] cbsz:1\n\t"
    "ds_read_b64_tr_b8 v[104:105], v61 offset:32768\n\t"
    "ds_read_b64_tr_b8 v[106:107], v61 offset:33792\n\t"
    "buffer_load_dwordx4 v[132:135], %[xvoff], s[60:63], s75 offen offset:1024 sc0 sc1\n\t"
    "v_mfma_f32_16x16x128_f8f6f4 v[228:231], v[72:79], v[168:175], v[228:231] cbsz:1\n\t"
    "ds_read_b64_tr_b8 v[108:109], v61 offset:40960\n\t"
    "ds_read_b64_tr_b8 v[110:111], v61 offset:41984\n\t"
    "s_waitcnt lgkmcnt(12)\n\t"
    "v_mfma_f32_16x16x128_f8f6f4 v[200:203], v[80:87], v[160:167], v[200:203] cbsz:1\n\t"
    "ds_read_b64_tr_b8 v[112:113], v62 offset:32768\n\t"
    "ds_read_b64_tr_b8 v[114:115], v62 offset:33792\n\t"
    "buffer_load_dwordx4 v[136:139], %[xvoff], s[60:63], s75 offen offset:2048 sc0 sc1\n\t"
    "v_mfma_f32_16x16x128_f8f6f4 v[232:235], v[80:87], v[168:175], v[232:235] cbsz:1\n\t"
    "ds_read_b64_tr_b8 v[116:117], v62 offset:40960\n\t"
    "ds_read_b64_tr_b8 v[118:119], v62 offset:41984\n\t"
    "s_waitcnt lgkmcnt(12)\n\t"
    "v_mfma_f32_16x16x128_f8f6f4 v[204:207], v[88:95], v[160:167], v[204:207] cbsz:1\n\t"
    "ds_read_b64_tr_b8 v[120:121], v63 offset:32768\n\t"
    "ds_read_b64_tr_b8 v[122:123], v63 offset:33792\n\t"
    "buffer_load_dwordx4 v[140:143], %[xvoff], s[60:63], s75 offen offset:3072 sc0 sc1\n\t"
    "v_mfma_f32_16x16x128_f8f6f4 v[236:239], v[88:95], v[168:175], v[236:239] cbsz:1\n\t"
    "ds_read_b64_tr_b8 v[124:125], v63 offset:40960\n\t"
    "ds_read_b64_tr_b8 v[126:127], v63 offset:41984\n\t"
    "s_waitcnt vmcnt(13)\n\t"
    "s_barrier\n\t"
    "s_waitcnt lgkmcnt(12)\n\t"
    "v_mfma_f32_16x16x128_f8f6f4 v[208:211], v[96:103], v[160:167], v[208:211] cbsz:1\n\t"
    "ds_read_b64_tr_b8 v[64:65], v56 offset:49152\n\t"
    "ds_read_b64_tr_b8 v[66:67], v56 offset:50176\n\t"
    "v_readlane_b32 s77, %[tw], 22\n\t"
    "s_mov_b32 s73, s77\n\t"
    "s_add_u32 m0, %[m0w], 98304\n\t"
    "s_nop 0\n\t"
    "buffer_load_dwordx4 %[wvoff], %[srdW], s73 offen lds\n\t"
    "v_mfma_f32_16x16x128_f8f6f4 v[240:243], v[96:103], v[168:175], v[240:243] cbsz:1\n\t"
    "ds_read_b64_tr_b8 v[68:69], v56 offset:57344\n\t"
    "ds_read_b64_tr_b8 v[70:71], v56 offset:58368\n\t"
    "s_waitcnt lgkmcnt(12)\n\t"
    "v_mfma_f32_16x16x128_f8f6f4 v[212:215], v[104:111], v[160:167], v[212:215] cbsz:1\n\t"
    "ds_read_b64_tr_b8 v[72:73], v57 offset:49152\n\t"
    "ds_read_b64_tr_b8 v[74:75], v57 offset:50176\n\t"
    "s_add_u32 m0, %[m0w], 99328\n\t"
    "s_add_u32 s73, s73, 131072\n\t"
    "buffer_load_dwordx4 %[wvoff], %[srdW], s73 offen lds\n\t"
    "v_mfma_f32_16x16x128_f8f6f4 v[244:247], v[104:111], v[168:175], v[244:247] cbsz:1\n\t"
    "ds_read_b64_tr_b8 v[76:77], v57 offset:57344\n\t"
    "ds_read_b64_tr_b8 v[78:79], v57 offset:58368\n\t"
    "s_waitcnt vmcnt(8)\n\t"
    "v_min3_u32 v53, v176, v177, v178\n\t"
    "v_min3_u32 v54, v179, v180, v181\n\t"
    "s_waitcnt lgkmcnt(12)\n\t"
    "v_mfma_f32_16x16x128_f8f6f4 v[216:219], v[112:119], v[160:167], v[216:219] cbsz:1\n\t"
    "ds_read_b64_tr_b8 v[80:81], v58 offset:49152\n\t"
    "ds_read_b64_tr_b8 v[82:83], v58 offset:50176\n\t"
    "v_min3_u32 v53, v53, v182, v183\n\t"
    "v_min3_u32 v54, v54, v184, v185\n\t"
    "v_mfma_f32_16x16x128_f8f6f4 v[248:251], v[112:119], v[168:175], v[248:251] cbsz:1\n\t"
    "ds_read_b64_tr_b8 v[84:85], v58 offset:57344\n\t"
    "ds_read_b64_tr_b8 v[86:87], v58 offset:58368\n\t"
    "v_min3_u32 v53, v53, v186, v187\n\t"
    "v_min3_u32 v54, v54, v188, v189\n\t"
    "s_waitcnt lgkmcnt(12)\n\t"
    "v_mfma_f32_16x16x128_f8f6f4 v[220:223], v[120:127], v[160:167], v[220:223] cbsz:1\n\t"
    "ds_read_b64_tr_b8 v[88:89], v59 offset:49152\n\t"
    "ds_read_b64_tr_b8 v[90:91], v59 offset:50176\n\t"
    "v_min3_u32 v53, v53, v190, v191\n\t"
    "v_min_u32 v53, v53, v54\n\t"
    "v_mfma_f32_16x16x128_f8f6f4 v[252:255], v[120:127], v[168:175], v[252:255] cbsz:1\n\t"
    "ds_read_b64_tr_b8 v[92:93], v59 offset:57344\n\t"
    "ds_read_b64_tr_b8 v[94:95], v59 offset:58368\n\t"
    "v_cmp_eq_u32 vcc, 0, v53\n\t"
    "s_cbranch_vccz L_ok19_%=\n\t"
    "v_readlane_b32 s77, %[tx], 19\n\t"
    "s_mov_b32 s82, s77\n\t"
    "s_getpc_b64 s[58:59]\n\t"
    "s_branch L_slow3_%=\n\t"
    "L_ok19_%=:\n\t"
    "s_waitcnt lgkmcnt(12)\n\t"
    "v_mfma_f32_16x16x128_f8f6f4 v[192:195], v[64:71], v[176:183], v[192:195] cbsz:1\n\t"
    "ds_read_b64_tr_b8 v[96:97], v60 offset:49152\n\t"
    "ds_read_b64_tr_b8 v[98:99], v60 offset:50176\n\t"
    "v_xor_b32 v56, 0x10000, v56\n\t"
    "v_mfma_f32_16x16x128_f8f6f4 v[224:227], v[64:71], v[184:191], v[224:227] cbsz:1\n\t"
    "ds_read_b64_tr_b8 v[100:101], v60 offset:57344\n\t"
    "ds_read_b64_tr_b8 v[102:103], v60 offset:58368\n\t"
    "v_readlane_b32 s77, %[tx], 21\n\t"
    "s_mov_b32 s75, s77\n\t"
    "buffer_load_dwordx4 v[144:147], %[xvoff], s[60:63], s75 offen offset:0 sc0 sc1\n\t"
    "v_xor_b32 v57, 0x10000, v57\n\t"
    "s_waitcnt lgkmcnt(12)\n\t"
    "v_mfma_f32_16x16x128_f8f6f4 v[196:199], v[72:79], v[176:183], v[196:199] cbsz:1\n\t"
    "ds_read_b64_tr_b8 v[104:105], v61 offset:49152\n\t"
    "ds_read_b64_tr_b8 v[106:107], v61 offset:50176\n\t"
    "buffer_load_dwordx4 v[148:151], %[xvoff], s[60:63], s75 offen offset:1024 sc0 sc1\n\t"
    "v_xor_b32 v58, 0x10000, v58\n\t"
    "v_mfma_f32_16x16x128_f8f6f4 v[228:231], v[72:79], v[184:191], v[228:231] cbsz:1\n\t"
    "ds_read_b64_tr_b8 v[108:109], v61 offset:57344\n\t"
    "ds_read_b64_tr_b8 v[110:111], v61 offset:58368\n\t"
    "v_xor_b32 v59, 0x10000, v59\n\t"
    "s_waitcnt lgkmcnt(12)\n\t"
    "v_mfma_f32_16x16x128_f8f6f4 v[200:203], v[80:87], v[176:183], v[200:203] cbsz:1\n\t"
    "ds_read_b64_tr_b8 v[112:113], v62 offset:49152\n\t"
    "ds_read_b64_tr_b8 v[114:115], v62 offset:50176\n\t"
    "buffer_load_dwordx4 v[152:155], %[xvoff], s[60:63], s75 offen offset:2048 sc0 sc1\n\t"
    "v_mfma_f32_16x16x128_f8f6f4 v[232:235], v[80:87], v[184:191], v[232:235] cbsz:1\n\t"
    "ds_read_b64_tr_b8 v[116:117], v62 offset:57344\n\t"
    "ds_read_b64_tr_b8 v[118:119], v62 offset:58368\n\t"
    "s_waitcnt lgkmcnt(12)\n\t"
    "v_mfma_f32_16x16x128_f8f6f4 v[204:207], v[88:95], v[176:183], v[204:207] cbsz:1\n\t"
    "ds_read_b64_tr_b8 v[120:121], v63 offset:49152\n\t"
    "ds_read_b64_tr_b8 v[122:123], v63 offset:50176\n\t"
    "buffer_load_dwordx4 v[156:159], %[xvoff], s[60:63], s75 offen offset:3072 sc0 sc1\n\t"
    "v_mfma_f32_16x16x128_f8f6f4 v[236:239], v[88:95], v[184:191], v[236:239] cbsz:1\n\t"
    "ds_read_b64_tr_b8 v[124:125], v63 offset:57344\n\t"
    "ds_read_b64_tr_b8 v[126:127], v63 offset:58368\n\t"
    "s_waitcnt lgkmcnt(12)\n\t"
    "v_mfma_f32_16x16x128_f8f6f4 v[208:211], v[96:103], v[176:183], v[208:211] cbsz:1\n\t"
    "ds_read_b64_tr_b8 v[64:65], v56 offset:0\n\t"
    "ds_read_b64_tr_b8 v[66:67], v56 offset:1024\n\t"
    "v_readlane_b32 s77, %[tw], 23\n\t"
    "s_mov_b32 s73, s77\n\t"
    "s_add_u32 m0, %[m0w], 114688\n\t"
    "s_nop 0\n\t"
    "buffer_load_dwordx4 %[wvoff], %[srdW], s73 offen lds\n\t"
    "v_xor_b32 v60, 0x10000, v60\n\t"
    "v_mfma_f32_16x16x128_f8f6f4 v[240:243], v[96:103], v[184:191], v[240:243] cbsz:1\n\t"
    "ds_read_b64_tr_b8 v[68:69], v56 offset:8192\n\t"
    "ds_read_b64_tr_b8 v[70:71], v56 offset:9216\n\t"
    "v_xor_b32 v61, 0x10000, v61\n\t"
    "s_waitcnt lgkmcnt(12)\n\t"
    "v_mfma_f32_16x16x128_f8f6f4 v[212:215], v[104:111], v[176:183], v[212:215] cbsz:1\n\t"
    "ds_read_b64_tr_b8 v[72:73], v57 offset:0\n\t"
    "ds_read_b64_tr_b8 v[74:75], v57 offset:1024\n\t"
    "s_add_u32 m0, %[m0w], 115712\n\t"
    "s_add_u32 s73, s73, 131072\n\t"
    "buffer_load_dwordx4 %[wvoff], %[srdW], s73 offen lds\n\t"
    "v_xor_b32 v62, 0x10000, v62\n\t"
    "v_mfma_f32_16x16x128_f8f6f4 v[244:247], v[104:111], v[184:191], v[244:247] cbsz:1\n\t"
    "ds_read_b64_tr_b8 v[76:77], v57 offset:8192\n\t"
    "ds_read_b64_tr_b8 v[78:79], v57 offset:9216\n\t"
    "s_waitcnt vmcnt(8)\n\t"
    "v_min3_u32 v53, v128, v129, v130\n\t"
    "v_min3_u32 v54, v131, v132, v133\n\t"
    "v_xor_b32 v63, 0x10000, v63\n\t"
    "s_waitcnt lgkmcnt(12)\n\t"
    "v_mfma_f32_16x16x128_f8f6f4 v[216:219], v[112:119], v[176:183], v[216:219] cbsz:1\n\t"
    "ds_read_b64_tr_b8 v[80:81], v58 offset:0\n\t"
    "ds_read_b64_tr_b8 v[82:83], v58 offset:1024\n\t"
    "v_min3_u32 v53, v53, v134, v135\n\t"
    "v_min3_u32 v54, v54, v136, v137\n\t"
    "v_mfma_f32_16x16x128_f8f6f4 v[248:251], v[112:119], v[184:191], v[248:251] cbsz:1\n\t"
    "ds_read_b64_tr_b8 v[84:85], v58 offset:8192\n\t"
    "ds_read_b64_tr_b8 v[86:87], v58 offset:9216\n\t"
    "v_min3_u32 v53, v53, v138, v139\n\t"
    "v_min3_u32 v54, v54, v140, v141\n\t"
    "s_waitcnt lgkmcnt(12)\n\t"
    "v_mfma_f32_16x16x128_f8f6f4 v[220:223], v[120:127], v[176:183], v[220:223] cbsz:1\n\t"
    "ds_read_b64_tr_b8 v[88:89], v59 offset:0\n\t"
    "ds_read_b64_tr_b8 v[90:91], v59 offset:1024\n\t"
    "v_min3_u32 v53, v53, v142, v143\n\t"
    "v_min_u32 v53, v53, v54\n\t"
    "v_mfma_f32_16x16x128_f8f6f4 v[252:255], v[120:127], v[184:191], v[252:255] cbsz:1\n\t"
    "ds_read_b64_tr_b8 v[92:93], v59 offset:8192\n\t"
    "ds_read_b64_tr_b8 v[94:95], v59 offset:9216\n\t"
    "v_cmp_eq_u32 vcc, 0, v53\n\t"
    "s_cbranch_vccz L_ok20_%=\n\t"
    "v_readlane_b32 s77, %[tx], 20\n\t"
    "s_mov_b32 s82, s77\n\t"
    "s_getpc_b64 s[58:59]\n\t"
    "s_branch L_slow0_%=\n\t"
    "L_ok20_%=:\n\t"
    "s_waitcnt lgkmcnt(12)\n\t"
    "v_mfma_f32_16x16x128_f8f6f4 v[192:195], v[64:71], v[128:135], v[192:195] cbsz:1\n\t"
    "ds_read_b64_tr_b8 v[96:97], v60 offset:0\n\t"
    "ds_read_b64_tr_b8 v[98:99], v60 offset:1024\n\t"
    "v_mfma_f32_16x16x128_f8f6f4 v[224:227], v[64:71], v[136:143], v[224:227] cbsz:1\n\t"
    "ds_read_b64_tr_b8 v[100:101], v60 offset:8192\n\t"
    "ds_read_b64_tr_b8 v[102:103], v60 offset:9216\n\t"
    "v_readlane_b32 s77, %[tx], 22\n\t"
    "s_mov_b32 s75, s77\n\t"
    "buffer_load_dwordx4 v[160:163], %[xvoff], s[60:63], s75 offen offset:0 sc0 sc1\n\t"
    "s_waitcnt lgkmcnt(12)\n\t"
    "v_mfma_f32_16x16x128_f8f6f4 v[196:199], v[72:79], v[128:135], v[196:199] cbsz:1\n\t"
    "ds_read_b64_tr_b8 v[104:105], v61 offset:0\n\t"
    "ds_read_b64_tr_b8 v[106:107], v61 offset:1024\n\t"
    "buffer_load_dwordx4 v[164:167], %[xvoff], s[60:63], s75 offen offset:1024 sc0 sc1\n\t"
    "v_mfma_f32_16x16x128_f8f6f4 v[228:231], v[72:79], v[136:143], v[228:231] cbsz:1\n\t"
    "ds_read_b64_tr_b8 v[108:109], v61 offset:8192\n\t"
    "ds_read_b64_tr_b8 v[110:111], v61 offset:9216\n\t"
    "s_waitcnt lgkmcnt(12)\n\t"
    "v_mfma_f32_16x16x128_f8f6f4 v[200:203], v[80:87], v[128:135], v[200:203] cbsz:1\n\t"
    "ds_read_b64_tr_b8 v[112:113], v62 offset:0\n\t"
    "ds_read_b64_tr_b8 v[114:115], v62 offset:1024\n\t"
    "buffer_load_dwordx4 v[168:171], %[xvoff], s[60:63], s75 offen offset:2048 sc0 sc1\n\t"
    "v_mfma_f32_16x16x128_f8f6f4 v[232:235], v[80:87], v[136:143], v[232:235] cbsz:1\n\t"
    "ds_read_b64_tr_b8 v[116:117], v62 offset:8192\n\t"
    "ds_read_b64_tr_b8 v[118:119], v62 offset:9216\n\t"
    "s_waitcnt lgkmcnt(12)\n\t"
    "v_mfma_f32_16x16x128_f8f6f4 v[204:207], v[88:95], v[128:135], v[204:207] cbsz:1\n\t"
    "ds_read_b64_tr_b8 v[120:121], v63 offset:0\n\t"
    "ds_read_b64_tr_b8 v[122:123], v63 offset:1024\n\t"
    "buffer_load_dwordx4 v[172:175], %[xvoff], s[60:63], s75 offen offset:3072 sc0 sc1\n\t"
    "v_mfma_f32_16x16x128_f8f6f4 v[236:239], v[88:95], v[136:143], v[236:239] cbsz:1\n\t"
    "ds_read_b64_tr_b8 v[124:125], v63 offset:8192\n\t"
    "ds_read_b64_tr_b8 v[126:127], v63 offset:9216\n\t"
    "s_waitcnt vmcnt(10)\n\t"
    "s_barrier\n\t"
    "s_waitcnt lgkmcnt(12)\n\t"
    "v_mfma_f32_16x16x128_f8f6f4 v[208:211], v[96:103], v[128:135], v[208:211] cbsz:1\n\t"
    "ds_read_b64_tr_b8 v[64:65], v56 offset:16384\n\t"
    "ds_read_b64_tr_b8 v[66:67], v56 offset:17408\n\t"
    "v_readlane_b32 s77, %[tw], 24\n\t"
    "s_mov_b32 s73, s77\n\t"
    "s_add_u32 m0, %[m0w], 0\n\t"
    "s_nop 0\n\t"
    "buffer_load_dwordx4 %[wvoff], %[srdW], s73 offen lds\n\t"
    "v_mfma_f32_16x16x128_f8f6f4 v[240:243], v[96:103], v[136:143], v[240:243] cbsz:1\n\t"
    "ds_read_b64_tr_b8 v[68:69], v56 offset:24576\n\t"
    "ds_read_b64_tr_b8 v[70:71], v56 offset:25600\n\t"
    "s_waitcnt lgkmcnt(12)\n\t"
    "v_mfma_f32_16x16x128_f8f6f4 v[212:215], v[104:111], v[128:135], v[212:215] cbsz:1\n\t"
    "ds_read_b64_tr_b8 v[72:73], v57 offset:16384\n\t"
    "ds_read_b64_tr_b8 v[74:75], v57 offset:17408\n\t"
    "s_add_u32 m0, %[m0w], 1024\n\t"
    "s_add_u32 s73, s73, 131072\n\t"
    "buffer_load_dwordx4 %[wvoff], %[srdW], s73 offen lds\n\t"
    "s_cmp_eq_u32 %[cpfw], 63\n\t"
    "s_cbranch_scc0 L_ccp3_%=\n\t"
    "s_mov_b32 s77, 24576\n\t"
    "buffer_load_dword v46, %[cvoff], s[84:87], s77 offen sc1\n\t"
    "s_branch L_dcp3_%=\n\t"
    "L_ccp3_%=:\n\t"
    "buffer_load_dword v46, v55, %[srdM], s79 offen\n\t"
    "L_dcp3_%=:\n\t"
    "v_mfma_f32_16x16x128_f8f6f4 v[244:247], v[104:111], v[136:143], v[244:247] cbsz:1\n\t"
    "ds_read_b64_tr_b8 v[76:77], v57 offset:24576\n\t"
    "ds_read_b64_tr_b8 v[78:79], v57 offset:25600\n\t"
    "s_waitcnt vmcnt(9)\n\t"
    "v_min3_u32 v53, v144, v145, v146\n\t"
    "v_min3_u32 v54, v147, v148, v149\n\t"
    "s_waitcnt lgkmcnt(12)\n\t"
    "v_mfma_f32_16x16x128_f8f6f4 v[216:219], v[112:119], v[128:135], v[216:219] cbsz:1\n\t"
    "ds_read_b64_tr_b8 v[80:81], v58 offset:16384\n\t"
    "ds_read_b64_tr_b8 v[82:83], v58 offset:17408\n\t"
    "v_min3_u32 v53, v53, v150, v151\n\t"
    "v_min3_u32 v54, v54, v152, v153\n\t"
    "v_mfma_f32_16x16x128_f8f6f4 v[248:251], v[112:119], v[136:143], v[248:251] cbsz:1\n\t"
    "ds_read_b64_tr_b8 v[84:85], v58 offset:24576\n\t"
    "ds_read_b64_tr_b8 v[86:87], v58 offset:25600\n\t"
    "v_min3_u32 v53, v53, v154, v155\n\t"
    "v_min3_u32 v54, v54, v156, v157\n\t"
    "s_waitcnt lgkmcnt(12)\n\t"
    "v_mfma_f32_16x16x128_f8f6f4 v[220:223], v[120:127], v[128:135], v[220:223] cbsz:1\n\t"
    "ds_read_b64_tr_b8 v[88:89], v59 offset:16384\n\t"
    "ds_read_b64_tr_b8 v[90:91], v59 offset:17408\n\t"
    "v_min3_u32 v53, v53, v158, v159\n\t"
    "v_min_u32 v53, v53, v54\n\t"
    "v_mfma_f32_16x16x128_f8f6f4 v[252:255], v[120:127], v[136:143], v[252:255] cbsz:1\n\t"
    "ds_read_b64_tr_b8 v[92:93], v59 offset:24576\n\t"
    "ds_read_b64_tr_b8 v[94:95], v59 offset:25600\n\t"
    "v_cmp_eq_u32 vcc, 0, v53\n\t"
    "s_cbranch_vccz L_ok21_%=\n\t"
    "v_readlane_b32 s77, %[tx], 21\n\t"
    "s_mov_b32 s82, s77\n\t"
    "s_getpc_b64 s[58:59]\n\t"
    "s_branch L_slow1_%=\n\t"
    "L_ok21_%=:\n\t"
    "s_waitcnt lgkmcnt(12)\n\t"
    "v_mfma_f32_16x16x128_f8f6f4 v[192:195], v[64:71], v[144:151], v[192:195] cbsz:1\n\t"
    "ds_read_b64_tr_b8 v[96:97], v60 offset:16384\n\t"
    "ds_read_b64_tr_b8 v[98:99], v60 offset:17408\n\t"
    "v_mfma_f32_16x16x128_f8f6f4 v[224:227], v[64:71], v[152:159], v[224:227] cbsz:1\n\t"
    "ds_read_b64_tr_b8 v[100:101], v60 offset:24576\n\t"
    "ds_read_b64_tr_b8 v[102:103], v60 offset:25600\n\t"
    "v_readlane_b32 s77, %[tx], 23\n\t"
    "s_mov_b32 s75, s77\n\t"
    "buffer_load_dwordx4 v[176:179], %[xvoff], s[60:63], s75 offen offset:0 sc0 sc1\n\t"
    "s_waitcnt lgkmcnt(12)\n\t"
    "v_mfma_f32_16x16x128_f8f6f4 v[196:199], v[72:79], v[144:151], v[196:199] cbsz:1\n\t"
    "ds_read_b64_tr_b8 v[104:105], v61 offset:16384\n\t"
    "ds_read_b64_tr_b8 v[106:107], v61 offset:17408\n\t"
    "buffer_load_dwordx4 v[180:183], %[xvoff], s[60:63], s75 offen offset:1024 sc0 sc1\n\t"
    "v_mfma_f32_16x16x128_f8f6f4 v[228:231], v[72:79], v[152:159], v[228:231] cbsz:1\n\t"
    "ds_read_b64_tr_b8 v[108:109], v61 offset:24576\n\t"
    "ds_read_b64_tr_b8 v[110:111], v61 offset:25600\n\t"
    "s_waitcnt lgkmcnt(12)\n\t"
    "v_mfma_f32_16x16x128_f8f6f4 v[200:203], v[80:87], v[144:151], v[200:203] cbsz:1\n\t"
    "ds_read_b64_tr_b8 v[112:113], v62 offset:16384\n\t"
    "ds_read_b64_tr_b8 v[114:115], v62 offset:17408\n\t"
    "buffer_load_dwordx4 v[184:187], %[xvoff], s[60:63], s75 offen offset:2048 sc0 sc1\n\t"
    "v_mfma_f32_16x16x128_f8f6f4 v[232:235], v[80:87], v[152:159], v[232:235] cbsz:1\n\t"
    "ds_read_b64_tr_b8 v[116:117], v62 offset:24576\n\t"
    "ds_read_b64_tr_b8 v[118:119], v62 offset:25600\n\t"
    "s_waitcnt lgkmcnt(12)\n\t"
    "v_mfma_f32_16x16x128_f8f6f4 v[204:207], v[88:95], v[144:151], v[204:207] cbsz:1\n\t"
    "ds_read_b64_tr_b8 v[120:121], v63 offset:16384\n\t"
    "ds_read_b64_tr_b8 v[122:123], v63 offset:17408\n\t"
    "buffer_load_dwordx4 v[188:191], %[xvoff], s[60:63], s75 offen offset:3072 sc0 sc1\n\t"
    "v_mfma_f32_16x16x128_f8f6f4 v[236:239], v[88:95], v[152:159], v[236:239] cbsz:1\n\t"
    "ds_read_b64_tr_b8 v[124:125], v63 offset:24576\n\t"
    "ds_read_b64_tr_b8 v[126:127], v63 offset:25600\n\t"
    "s_waitcnt lgkmcnt(12)\n\t"
    "v_mfma_f32_16x16x128_f8f6f4 v[208:211], v[96:103], v[144:151], v[208:211] cbsz:1\n\t"
    "ds_read_b64_tr_b8 v[64:65], v56 offset:32768\n\t"
    "ds_read_b64_tr_b8 v[66:67], v56 offset:33792\n\t"
    "v_readlane_b32 s77, %[tw], 25\n\t"
    "s_mov_b32 s73, s77\n\t"
    "s_add_u32 m0, %[m0w], 16384\n\t"
    "s_nop 0\n\t"
    "buffer_load_dwordx4 %[wvoff], %[srdW], s73 offen lds\n\t"
    "v_mfma_f32_16x16x128_f8f6f4 v[240:243], v[96:103], v[152:159], v[240:243] cbsz:1\n\t"
    "ds_read_b64_tr_b8 v[68:69], v56 offset:40960\n\t"
    "ds_read_b64_tr_b8 v[70:71], v56 offset:41984\n\t"
    "s_waitcnt lgkmcnt(12)\n\t"
    "v_mfma_f32_16x16x128_f8f6f4 v[212:215], v[104:111], v[144:151], v[212:215] cbsz:1\n\t"
    "ds_read_b64_tr_b8 v[72:73], v57 offset:32768\n\t"
    "ds_read_b64_tr_b8 v[74:75], v57 offset:33792\n\t"
    "s_add_u32 m0, %[m0w], 17408\n\t"
    "s_add_u32 s73, s73, 131072\n\t"
    "buffer_load_dwordx4 %[wvoff], %[srdW], s73 offen lds\n\t"
    "v_mfma_f32_16x16x128_f8f6f4 v[244:247], v[104:111], v[152:159], v[244:247] cbsz:1\n\t"
    "ds_read_b64_tr_b8 v[76:77], v57 offset:40960\n\t"
    "ds_read_b64_tr_b8 v[78:79], v57 offset:41984\n\t"
    "s_waitcnt vmcnt(9)\n\t"
    "v_min3_u32 v53, v160, v161, v162\n\t"
    "v_min3_u32 v54, v163, v164, v165\n\t"
    "s_waitcnt lgkmcnt(12)\n\t"
    "v_mfma_f32_16x16x128_f8f6f4 v[216:219], v[112:119], v[144:151], v[216:219] cbsz:1\n\t"
    "ds_read_b64_tr_b8 v[80:81], v58 offset:32768\n\t"
    "ds_read_b64_tr_b8 v[82:83], v58 offset:33792\n\t"
    "v_min3_u32 v53, v53, v166, v167\n\t"
    "v_min3_u32 v54, v54, v168, v169\n\t"
    "v_mfma_f32_16x16x128_f8f6f4 v[248:251], v[112:119], v[152:159], v[248:251] cbsz:1\n\t"
    "ds_read_b64_tr_b8 v[84:85], v58 offset:40960\n\t"
    "ds_read_b64_tr_b8 v[86:87], v58 offset:41984\n\t"
    "v_min3_u32 v53, v53, v170, v171\n\t"
    "v_min3_u32 v54, v54, v172, v173\n\t"
    "s_waitcnt lgkmcnt(12)\n\t"
    "v_mfma_f32_16x16x128_f8f6f4 v[220:223], v[120:127], v[144:151], v[220:223] cbsz:1\n\t"
    "ds_read_b64_tr_b8 v[88:89], v59 offset:32768\n\t"
    "ds_read_b64_tr_b8 v[90:91], v59 offset:33792\n\t"
    "v_min3_u32 v53, v53, v174, v175\n\t"
    "v_min_u32 v53, v53, v54\n\t"
    "v_mfma_f32_16x16x128_f8f6f4 v[252:255], v[120:127], v[152:159], v[252:255] cbsz:1\n\t"
    "ds_read_b64_tr_b8 v[92:93], v59 offset:40960\n\t"
    "ds_read_b64_tr_b8 v[94:95], v59 offset:41984\n\t"
    "v_cmp_eq_u32 vcc, 0, v53\n\t"
    "s_cbranch_vccz L_ok22_%=\n\t"
    "v_readlane_b32 s77, %[tx], 22\n\t"
    "s_mov_b32 s82, s77\n\t"
    "s_getpc_b64 s[58:59]\n\t"
    "s_branch L_slow2_%=\n\t"
    "L_ok22_%=:\n\t"
    "s_waitcnt lgkmcnt(12)\n\t"
    "v_mfma_f32_16x16x128_f8f6f4 v[192:195], v[64:71], v[160:167], v[192:195] cbsz:1\n\t"
    "ds_read_b64_tr_b8 v[96:97], v60 offset:32768\n\t"
    "ds_read_b64_tr_b8 v[98:99], v60 offset:33792\n\t"
    "v_mfma_f32_16x16x128_f8f6f4 v[224:227], v[64:71], v[168:175], v[224:227] cbsz:1\n\t"
    "ds_read_b64_tr_b8 v[100:101], v60 offset:40960\n\t"
    "ds_read_b64_tr_b8 v[102:103], v60 offset:41984\n\t"
    "v_readlane_b32 s77, %[tx], 24\n\t"
    "s_mov_b32 s75, s77\n\t"
    "buffer_load_dwordx4 v[128:131], %[xvoff], s[60:63], s75 offen offset:0 sc0 sc1\n\t"
    "s_waitcnt lgkmcnt(12)\n\t"
    "v_mfma_f32_16x16x128_f8f6f4 v[196:199], v[72:79], v[160:167], v[196:199] cbsz:1\n\t"
    "ds_read_b64_tr_b8 v[104:105], v61 offset:32768\n\t"
    "ds_read_b64_tr_b8 v[106:107], v61 offset:33792\n\t"
    "buffer_load_dwordx4 v[132:135], %[xvoff], s[60:63], s75 offen offset:1024 sc0 sc1\n\t"
    "v_mfma_f32_16x16x128_f8f6f4 v[228:231], v[72:79], v[168:175], v[228:231] cbsz:1\n\t"
    "ds_read_b64_tr_b8 v[108:109], v61 offset:40960\n\t"
    "ds_read_b64_tr_b8 v[110:111], v61 offset:41984\n\t"
    "s_waitcnt lgkmcnt(12)\n\t"
    "v_mfma_f32_16x16x128_f8f6f4 v[200:203], v[80:87], v[160:167], v[200:203] cbsz:1\n\t"
    "ds_read_b64_tr_b8 v[112:113], v62 offset:32768\n\t"
    "ds_read_b64_tr_b8 v[114:115], v62 offset:33792\n\t"
    "buffer_load_dwordx4 v[136:139], %[xvoff], s[60:63], s75 offen offset:2048 sc0 sc1\n\t"
    "v_mfma_f32_16x16x128_f8f6f4 v[232:235], v[80:87], v[168:175], v[232:235] cbsz:1\n\t"
    "ds_read_b64_tr_b8 v[116:117], v62 offset:40960\n\t"
    "ds_read_b64_tr_b8 v[118:119], v62 offset:41984\n\t"
    "s_waitcnt lgkmcnt(12)\n\t"
    "v_mfma_f32_16x16x128_f8f6f4 v[204:207], v[88:95], v[160:167], v[204:207] cbsz:1\n\t"
    "ds_read_b64_tr_b8 v[120:121], v63 offset:32768\n\t"
    "ds_read_b64_tr_b8 v[122:123], v63 offset:33792\n\t"
    "buffer_load_dwordx4 v[140:143], %[xvoff], s[60:63], s75 offen offset:3072 sc0 sc1\n\t"
    "v_mfma_f32_16x16x128_f8f6f4 v[236:239], v[88:95], v[168:175], v[236:239] cbsz:1\n\t"
    "ds_read_b64_tr_b8 v[124:125], v63 offset:40960\n\t"
    "ds_read_b64_tr_b8 v[126:127], v63 offset:41984\n\t"
    "s_waitcnt vmcnt(11)\n\t"
    "s_barrier\n\t"
    "s_cmp_lg_u32 s74, 0\n\t"
    "s_cbranch_scc1 L_pr%=\n\t"
    "s_setprio 3\n\t"
    "L_pr%=:\n\t"
    "s_waitcnt lgkmcnt(12)\n\t"
    "v_mfma_f32_16x16x128_f8f6f4 v[208:211], v[96:103], v[160:167], v[208:211] cbsz:1\n\t"
    "ds_read_b64_tr_b8 v[64:65], v56 offset:49152\n\t"
    "ds_read_b64_tr_b8 v[66:67], v56 offset:50176\n\t"
    "v_readlane_b32 s77, %[tw], 26\n\t"
    "s_mov_b32 s73, s77\n\t"
    "s_add_u32 m0, %[m0w], 32768\n\t"
    "s_nop 0\n\t"
    "buffer_load_dwordx4 %[wvoff], %[srdW], s73 offen lds\n\t"
    "s_add_u32 m0, %[m0w], 33792\n\t"
    "s_add_u32 s73, s73, 131072\n\t"
    "buffer_load_dwordx4 %[wvoff], %[srdW], s73 offen lds\n\t"
    "v_mfma_f32_16x16x128_f8f6f4 v[240:243], v[96:103], v[168:175], v[240:243] cbsz:1\n\t"
    "ds_read_b64_tr_b8 v[68:69], v56 offset:57344\n\t"
    "ds_read_b64_tr_b8 v[70:71], v56 offset:58368\n\t"
    "v_readlane_b32 s77, %[tw], 27\n\t"
    "s_mov_b32 s73, s77\n\t"
    "s_add_u32 m0, %[m0w], 49152\n\t"
    "s_nop 0\n\t"
    "buffer_load_dwordx4 %[wvoff], %[srdW], s73 offen lds\n\t"
    "s_add_u32 m0, %[m0w], 50176\n\t"
    "s_add_u32 s73, s73, 131072\n\t"
    "buffer_load_dwordx4 %[wvoff], %[srdW], s73 offen lds\n\t"
    "s_waitcnt lgkmcnt(12)\n\t"
    "v_mfma_f32_16x16x128_f8f6f4 v[212:215], v[104:111], v[160:167], v[212:215] cbsz:1\n\t"
    "ds_read_b64_tr_b8 v[72:73], v57 offset:49152\n\t"
    "ds_read_b64_tr_b8 v[74:75], v57 offset:50176\n\t"
    "v_readlane_b32 s77, %[tw], 28\n\t"
    "s_mov_b32 s73, s77\n\t"
    "s_add_u32 m0, %[m0w], 65536\n\t"
    "s_nop 0\n\t"
    "buffer_load_dwordx4 %[wvoff], %[srdW], s73 offen lds\n\t"
    "s_add_u32 m0, %[m0w], 66560\n\t"
    "s_add_u32 s73, s73, 131072\n\t"
    "buffer_load_dwordx4 %[wvoff], %[srdW], s73 offen lds\n\t"
    "v_mfma_f32_16x16x128_f8f6f4 v[244:247], v[104:111], v[168:175], v[244:247] cbsz:1\n\t"
    "ds_read_b64_tr_b8 v[76:77], v57 offset:57344\n\t"
    "ds_read_b64_tr_b8 v[78:79], v57 offset:58368\n\t"
    "v_readlane_b32 s77, %[tw], 29\n\t"
    "s_mov_b32 s73, s77\n\t"
    "s_add_u32 m0, %[m0w], 81920\n\t"
    "s_nop 0\n\t"
    "buffer_load_dwordx4 %[wvoff], %[srdW], s73 offen lds\n\t"
    "s_add_u32 m0, %[m0w], 82944\n\t"
    "s_add_u32 s73, s73, 131072\n\t"
    "buffer_load_dwordx4 %[wvoff], %[srdW], s73 offen lds\n\t"
    "s_waitcnt vmcnt(14)\n\t"
    "v_min3_u32 v53, v176, v177, v178\n\t"
    "v_min3_u32 v54, v179, v180, v181\n\t"
    "s_waitcnt lgkmcnt(12)\n\t"
    "v_mfma_f32_16x16x128_f8f6f4 v[216:219], v[112:119], v[160:167], v[216:219] cbsz:1\n\t"
    "ds_read_b64_tr_b8 v[80:81], v58 offset:49152\n\t"
    "ds_read_b64_tr_b8 v[82:83], v58 offset:50176\n\t"
    "v_min3_u32 v53, v53, v182, v183\n\t"
    "v_min3_u32 v54, v54, v184, v185\n\t"
    "v_mfma_f32_16x16x128_f8f6f4 v[248:251], v[112:119], v[168:175], v[248:251] cbsz:1\n\t"
    "ds_read_b64_tr_b8 v[84:85], v58 offset:57344\n\t"
    "ds_read_b64_tr_b8 v[86:87], v58 offset:58368\n\t"
    "v_min3_u32 v53, v53, v186, v187\n\t"
    "v_min3_u32 v54, v54, v188, v189\n\t"
    "s_waitcnt lgkmcnt(12)\n\t"
    "v_mfma_f32_16x16x128_f8f6f4 v[220:223], v[120:127], v[160:167], v[220:223] cbsz:1\n\t"
    "ds_read_b64_tr_b8 v[88:89], v59 offset:49152\n\t"
    "ds_read_b64_tr_b8 v[90:91], v59 offset:50176\n\t"
    "v_min3_u32 v53, v53, v190, v191\n\t"
    "v_min_u32 v53, v53, v54\n\t"
    "v_mfma_f32_16x16x128_f8f6f4 v[252:255], v[120:127], v[168:175], v[252:255] cbsz:1\n\t"
    "ds_read_b64_tr_b8 v[92:93], v59 offset:57344\n\t"
    "ds_read_b64_tr_b8 v[94:95], v59 offset:58368\n\t"
    "v_cmp_eq_u32 vcc, 0, v53\n\t"
    "s_cmp_lg_u32 s74, 1\n\t"
    "s_cbranch_scc1 L_gbb1_%=\n\t"
    "s_waitcnt vmcnt(4)\n\t"
    "s_barrier\n\t"
    "L_gbb1_%=:\n\t"
    "s_cbranch_vccz L_ok23_%=\n\t"
    "v_readlane_b32 s77, %[tx], 23\n\t"
    "s_mov_b32 s82, s77\n\t"
    "s_getpc_b64 s[58:59]\n\t"
    "s_branch L_slow3_%=\n\t"
    "L_ok23_%=:\n\t"
    "s_waitcnt lgkmcnt(12)\n\t"
    "v_mfma_f32_16x16x128_f8f6f4 v[192:195], v[64:71], v[176:183], v[192:195] cbsz:1\n\t"
    "ds_read_b64_tr_b8 v[96:97], v60 offset:49152\n\t"
    "ds_read_b64_tr_b8 v[98:99], v60 offset:50176\n\t"
    "v_xor_b32 v56, 0x10000, v56\n\t"
    "v_mfma_f32_16x16x128_f8f6f4 v[224:227], v[64:71], v[184:191], v[224:227] cbsz:1\n\t"
    "ds_read_b64_tr_b8 v[100:101], v60 offset:57344\n\t"
    "ds_read_b64_tr_b8 v[102:103], v60 offset:58368\n\t"
    "v_readlane_b32 s77, %[tx], 25\n\t"
    "s_mov_b32 s75, s77\n\t"
    "buffer_load_dwordx4 v[144:147], %[xvoff], s[60:63], s75 offen offset:0 sc0 sc1\n\t"
    "v_xor_b32 v57, 0x10000, v57\n\t"
    "s_waitcnt lgkmcnt(12)\n\t"
    "v_mfma_f32_16x16x128_f8f6f4 v[196:199], v[72:79], v[176:183], v[196:199] cbsz:1\n\t"
    "ds_read_b64_tr_b8 v[104:105], v61 offset:49152\n\t"
    "ds_read_b64_tr_b8 v[106:107], v61 offset:50176\n\t"
    "buffer_load_dwordx4 v[148:151], %[xvoff], s[60:63], s75 offen offset:1024 sc0 sc1\n\t"
    "v_xor_b32 v58, 0x10000, v58\n\t"
    "v_mfma_f32_16x16x128_f8f6f4 v[228:231], v[72:79], v[184:191], v[228:231] cbsz:1\n\t"
    "ds_read_b64_tr_b8 v[108:109], v61 offset:57344\n\t"
    "ds_read_b64_tr_b8 v[110:111], v61 offset:58368\n\t"
    "v_xor_b32 v59, 0x10000, v59\n\t"
    "s_waitcnt lgkmcnt(12)\n\t"
    "v_mfma_f32_16x16x128_f8f6f4 v[200:203], v[80:87], v[176:183], v[200:203] cbsz:1\n\t"
    "ds_read_b64_tr_b8 v[112:113], v62 offset:49152\n\t"
    "ds_read_b64_tr_b8 v[114:115], v62 offset:50176\n\t"
    "buffer_load_dwordx4 v[152:155], %[xvoff], s[60:63], s75 offen offset:2048 sc0 sc1\n\t"
    "v_mfma_f32_16x16x128_f8f6f4 v[232:235], v[80:87], v[184:191], v[232:235] cbsz:1\n\t"
    "ds_read_b64_tr_b8 v[116:117], v62 offset:57344\n\t"
    "ds_read_b64_tr_b8 v[118:119], v62 offset:58368\n\t"
    "s_waitcnt lgkmcnt(12)\n\t"
    "v_mfma_f32_16x16x128_f8f6f4 v[204:207], v[88:95], v[176:183], v[204:207] cbsz:1\n\t"
    "ds_read_b64_tr_b8 v[120:121], v63 offset:49152\n\t"
    "ds_read_b64_tr_b8 v[122:123], v63 offset:50176\n\t"
    "buffer_load_dwordx4 v[156:159], %[xvoff], s[60:63], s75 offen offset:3072 sc0 sc1\n\t"
    "v_mfma_f32_16x16x128_f8f6f4 v[236:239], v[88:95], v[184:191], v[236:239] cbsz:1\n\t"
    "ds_read_b64_tr_b8 v[124:125], v63 offset:57344\n\t"
    "ds_read_b64_tr_b8 v[126:127], v63 offset:58368\n\t"
    "s_cmp_lg_u32 s74, 1\n\t"
    "s_cbranch_scc1 L_gbb2_%=\n\t"
    "s_waitcnt vmcnt(4)\n\t"
    "s_barrier\n\t"
    "L_gbb2_%=:\n\t"
    "s_waitcnt lgkmcnt(12)\n\t"
    "v_mfma_f32_16x16x128_f8f6f4 v[208:211], v[96:103], v[176:183], v[208:211] cbsz:1\n\t"
    "ds_read_b64_tr_b8 v[64:65], v56 offset:0\n\t"
    "ds_read_b64_tr_b8 v[66:67], v56 offset:1024\n\t"
    "v_xor_b32 v60, 0x10000, v60\n\t"
    "v_mfma_f32_16x16x128_f8f6f4 v[240:243], v[96:103], v[184:191], v[240:243] cbsz:1\n\t"
    "ds_read_b64_tr_b8 v[68:69], v56 offset:8192\n\t"
    "ds_read_b64_tr_b8 v[70:71], v56 offset:9216\n\t"
    "v_xor_b32 v61, 0x10000, v61\n\t"
    "s_waitcnt lgkmcnt(12)\n\t"
    "v_mfma_f32_16x16x128_f8f6f4 v[212:215], v[104:111], v[176:183], v[212:215] cbsz:1\n\t"
    "ds_read_b64_tr_b8 v[72:73], v57 offset:0\n\t"
    "ds_read_b64_tr_b8 v[74:75], v57 offset:1024\n\t"
    "v_xor_b32 v62, 0x10000, v62\n\t"
    "v_mfma_f32_16x16x128_f8f6f4 v[244:247], v[104:111], v[184:191], v[244:247] cbsz:1\n\t"
    "ds_read_b64_tr_b8 v[76:77], v57 offset:8192\n\t"
    "ds_read_b64_tr_b8 v[78:79], v57 offset:9216\n\t"
    "s_waitcnt vmcnt(12)\n\t"
    "v_min3_u32 v53, v128, v129, v130\n\t"
    "v_min3_u32 v54, v131, v132, v133\n\t"
    "v_xor_b32 v63, 0x10000, v63\n\t"
    "s_waitcnt lgkmcnt(12)\n\t"
    "v_mfma_f32_16x16x128_f8f6f4 v[216:219], v[112:119], v[176:183], v[216:219] cbsz:1\n\t"
    "ds_read_b64_tr_b8 v[80:81], v58 offset:0\n\t"
    "ds_read_b64_tr_b8 v[82:83], v58 offset:1024\n\t"
    "v_min3_u32 v53, v53, v134, v135\n\t"
    "v_min3_u32 v54, v54, v136, v137\n\t"
    "v_mfma_f32_16x16x128_f8f6f4 v[248:251], v[112:119], v[184:191], v[248:251] cbsz:1\n\t"
    "ds_read_b64_tr_b8 v[84:85], v58 offset:8192\n\t"
    "ds_read_b64_tr_b8 v[86:87], v58 offset:9216\n\t"
    "v_min3_u32 v53, v53, v138, v139\n\t"
    "v_min3_u32 v54, v54, v140, v141\n\t"
    "s_waitcnt lgkmcnt(12)\n\t"
    "v_mfma_f32_16x16x128_f8f6f4 v[220:223], v[120:127], v[176:183], v[220:223] cbsz:1\n\t"
    "ds_read_b64_tr_b8 v[88:89], v59 offset:0\n\t"
    "ds_read_b64_tr_b8 v[90:91], v59 offset:1024\n\t"
    "v_min3_u32 v53, v53, v142, v143\n\t"
    "v_min_u32 v53, v53, v54\n\t"
    "v_mfma_f32_16x16x128_f8f6f4 v[252:255], v[120:127], v[184:191], v[252:255] cbsz:1\n\t"
    "ds_read_b64_tr_b8 v[92:93], v59 offset:8192\n\t"
    "ds_read_b64_tr_b8 v[94:95], v59 offset:9216\n\t"
    "v_cmp_eq_u32 vcc, 0, v53\n\t"
    "s_cbranch_vccz L_ok24_%=\n\t"
    "v_readlane_b32 s77, %[tx], 24\n\t"
    "s_mov_b32 s82, s77\n\t"
    "s_getpc_b64 s[58:59]\n\t"
    "s_branch L_slow0_%=\n\t"
    "L_ok24_%=:\n\t"
    "s_waitcnt lgkmcnt(12)\n\t"
    "v_mfma_f32_16x16x128_f8f6f4 v[192:195], v[64:71], v[128:135], v[192:195] cbsz:1\n\t"
    "ds_read_b64_tr_b8 v[96:97], v60 offset:0\n\t"
    "ds_read_b64_tr_b8 v[98:99], v60 offset:1024\n\t"
    "v_mfma_f32_16x16x128_f8f6f4 v[224:227], v[64:71], v[136:143], v[224:227] cbsz:1\n\t"
    "ds_read_b64_tr_b8 v[100:101], v60 offset:8192\n\t"
    "ds_read_b64_tr_b8 v[102:103], v60 offset:9216\n\t"
    "v_readlane_b32 s77, %[tx], 26\n\t"
    "s_mov_b32 s75, s77\n\t"
    "buffer_load_dwordx4 v[160:163], %[xvoff], s[60:63], s75 offen offset:0 sc0 sc1\n\t"
    "s_waitcnt lgkmcnt(12)\n\t"
    "v_mfma_f32_16x16x128_f8f6f4 v[196:199], v[72:79], v[128:135], v[196:199] cbsz:1\n\t"
    "ds_read_b64_tr_b8 v[104:105], v61 offset:0\n\t"
    "ds_read_b64_tr_b8 v[106:107], v61 offset:1024\n\t"
    "buffer_load_dwordx4 v[164:167], %[xvoff], s[60:63], s75 offen offset:1024 sc0 sc1\n\t"
    "v_mfma_f32_16x16x128_f8f6f4 v[228:231], v[72:79], v[136:143], v[228:231] cbsz:1\n\t"
    "ds_read_b64_tr_b8 v[108:109], v61 offset:8192\n\t"
    "ds_read_b64_tr_b8 v[110:111], v61 offset:9216\n\t"
    "s_waitcnt lgkmcnt(12)\n\t"
    "v_mfma_f32_16x16x128_f8f6f4 v[200:203], v[80:87], v[128:135], v[200:203] cbsz:1\n\t"
    "ds_read_b64_tr_b8 v[112:113], v62 offset:0\n\t"
    "ds_read_b64_tr_b8 v[114:115], v62 offset:1024\n\t"
    "buffer_load_dwordx4 v[168:171], %[xvoff], s[60:63], s75 offen offset:2048 sc0 sc1\n\t"
    "v_mfma_f32_16x16x128_f8f6f4 v[232:235], v[80:87], v[136:143], v[232:235] cbsz:1\n\t"
    "ds_read_b64_tr_b8 v[116:117], v62 offset:8192\n\t"
    "ds_read_b64_tr_b8 v[118:119], v62 offset:9216\n\t"
    "s_waitcnt lgkmcnt(12)\n\t"
    "v_mfma_f32_16x16x128_f8f6f4 v[204:207], v[88:95], v[128:135], v[204:207] cbsz:1\n\t"
    "ds_read_b64_tr_b8 v[120:121], v63 offset:0\n\t"
    "ds_read_b64_tr_b8 v[122:123], v63 offset:1024\n\t"
    "buffer_load_dwordx4 v[172:175], %[xvoff], s[60:63], s75 offen offset:3072 sc0 sc1\n\t"
    "v_mfma_f32_16x16x128_f8f6f4 v[236:239], v[88:95], v[136:143], v[236:239] cbsz:1\n\t"
    "ds_read_b64_tr_b8 v[124:125], v63 offset:8192\n\t"
    "ds_read_b64_tr_b8 v[126:127], v63 offset:9216\n\t"
    "s_cmp_lg_u32 s74, 0\n\t"
    "s_cbranch_scc1 L_gba1_%=\n\t"
    "s_waitcnt vmcnt(12)\n\t"
    "s_barrier\n\t"
    "L_gba1_%=:\n\t"
    "s_waitcnt lgkmcnt(12)\n\t"
    "v_mfma_f32_16x16x128_f8f6f4 v[208:211], v[96:103], v[128:135], v[208:211] cbsz:1\n\t"
    "ds_read_b64_tr_b8 v[64:65], v56 offset:16384\n\t"
    "ds_read_b64_tr_b8 v[66:67], v56 offset:17408\n\t"
    "v_mfma_f32_16x16x128_f8f6f4 v[240:243], v[96:103], v[136:143], v[240:243] cbsz:1\n\t"
    "ds_read_b64_tr_b8 v[68:69], v56 offset:24576\n\t"
    "ds_read_b64_tr_b8 v[70:71], v56 offset:25600\n\t"
    "s_waitcnt lgkmcnt(12)\n\t"
    "v_mfma_f32_16x16x128_f8f6f4 v[212:215], v[104:111], v[128:135], v[212:215] cbsz:1\n\t"
    "ds_read_b64_tr_b8 v[72:73], v57 offset:16384\n\t"
    "ds_read_b64_tr_b8 v[74:75], v57 offset:17408\n\t"
    "v_mfma_f32_16x16x128_f8f6f4 v[244:247], v[104:111], v[136:143], v[244:247] cbsz:1\n\t"
    "ds_read_b64_tr_b8 v[76:77], v57 offset:24576\n\t"
    "ds_read_b64_tr_b8 v[78:79], v57 offset:25600\n\t"
    "s_waitcnt vmcnt(4)\n\t"
    "v_min3_u32 v53, v144, v145, v146\n\t"
    "v_min3_u32 v54, v147, v148, v149\n\t"
    "s_waitcnt lgkmcnt(12)\n\t"
    "v_mfma_f32_16x16x128_f8f6f4 v[216:219], v[112:119], v[128:135], v[216:219] cbsz:1\n\t"
    "ds_read_b64_tr_b8 v[80:81], v58 offset:16384\n\t"
    "ds_read_b64_tr_b8 v[82:83], v58 offset:17408\n\t"
    "v_min3_u32 v53, v53, v150, v151\n\t"
    "v_min3_u32 v54, v54, v152, v153\n\t"
    "v_mfma_f32_16x16x128_f8f6f4 v[248:251], v[112:119], v[136:143], v[248:251] cbsz:1\n\t"
    "ds_read_b64_tr_b8 v[84:85], v58 offset:24576\n\t"
    "ds_read_b64_tr_b8 v[86:87], v58 offset:25600\n\t"
    "v_min3_u32 v53, v53, v154, v155\n\t"
    "v_min3_u32 v54, v54, v156, v157\n\t"
    "s_waitcnt lgkmcnt(12)\n\t"
    "v_mfma_f32_16x16x128_f8f6f4 v[220:223], v[120:127], v[128:135], v[220:223] cbsz:1\n\t"
    "ds_read_b64_tr_b8 v[88:89], v59 offset:16384\n\t"
    "ds_read_b64_tr_b8 v[90:91], v59 offset:17408\n\t"
    "v_min3_u32 v53, v53, v158, v159\n\t"
    "v_min_u32 v53, v53, v54\n\t"
    "v_mfma_f32_16x16x128_f8f6f4 v[252:255], v[120:127], v[136:143], v[252:255] cbsz:1\n\t"
    "ds_read_b64_tr_b8 v[92:93], v59 offset:24576\n\t"
    "ds_read_b64_tr_b8 v[94:95], v59 offset:25600\n\t"
    "v_cmp_eq_u32 vcc, 0, v53\n\t"
    "s_cbranch_vccz L_ok25_%=\n\t"
    "v_readlane_b32 s77, %[tx], 25\n\t"
    "s_mov_b32 s82, s77\n\t"
    "s_getpc_b64 s[58:59]\n\t"
    "s_branch L_slow1_%=\n\t"
    "L_ok25_%=:\n\t"
    "s_waitcnt lgkmcnt(12)\n\t"
    "v_mfma_f32_16x16x128_f8f6f4 v[192:195], v[64:71], v[144:151], v[192:195] cbsz:1\n\t"
    "ds_read_b64_tr_b8 v[96:97], v60 offset:16384\n\t"
    "ds_read_b64_tr_b8 v[98:99], v60 offset:17408\n\t"
    "v_mfma_f32_16x16x128_f8f6f4 v[224:227], v[64:71], v[152:159], v[224:227] cbsz:1\n\t"
    "ds_read_b64_tr_b8 v[100:101], v60 offset:24576\n\t"
    "ds_read_b64_tr_b8 v[102:103], v60 offset:25600\n\t"
    "v_readlane_b32 s77, %[tx], 27\n\t"
    "s_mov_b32 s75, s77\n\t"
    "buffer_load_dwordx4 v[176:179], %[xvoff], s[60:63], s75 offen offset:0 sc0 sc1\n\t"
    "s_waitcnt lgkmcnt(12)\n\t"
    "v_mfma_f32_16x16x128_f8f6f4 v[196:199], v[72:79], v[144:151], v[196:199] cbsz:1\n\t"
    "ds_read_b64_tr_b8 v[104:105], v61 offset:16384\n\t"
    "ds_read_b64_tr_b8 v[106:107], v61 offset:17408\n\t"
    "buffer_load_dwordx4 v[180:183], %[xvoff], s[60:63], s75 offen offset:1024 sc0 sc1\n\t"
    "v_mfma_f32_16x16x128_f8f6f4 v[228:231], v[72:79], v[152:159], v[228:231] cbsz:1\n\t"
    "ds_read_b64_tr_b8 v[108:109], v61 offset:24576\n\t"
    "ds_read_b64_tr_b8 v[110:111], v61 offset:25600\n\t"
    "s_waitcnt lgkmcnt(12)\n\t"
    "v_mfma_f32_16x16x128_f8f6f4 v[200:203], v[80:87], v[144:151], v[200:203] cbsz:1\n\t"
    "ds_read_b64_tr_b8 v[112:113], v62 offset:16384\n\t"
    "ds_read_b64_tr_b8 v[114:115], v62 offset:17408\n\t"
    "buffer_load_dwordx4 v[184:187], %[xvoff], s[60:63], s75 offen offset:2048 sc0 sc1\n\t"
    "v_mfma_f32_16x16x128_f8f6f4 v[232:235], v[80:87], v[152:159], v[232:235] cbsz:1\n\t"
    "ds_read_b64_tr_b8 v[116:117], v62 offset:24576\n\t"
    "ds_read_b64_tr_b8 v[118:119], v62 offset:25600\n\t"
    "s_waitcnt lgkmcnt(12)\n\t"
    "v_mfma_f32_16x16x128_f8f6f4 v[204:207], v[88:95], v[144:151], v[204:207] cbsz:1\n\t"
    "ds_read_b64_tr_b8 v[120:121], v63 offset:16384\n\t"
    "ds_read_b64_tr_b8 v[122:123], v63 offset:17408\n\t"
    "buffer_load_dwordx4 v[188:191], %[xvoff], s[60:63], s75 offen offset:3072 sc0 sc1\n\t"
    "v_mfma_f32_16x16x128_f8f6f4 v[236:239], v[88:95], v[152:159], v[236:239] cbsz:1\n\t"
    "ds_read_b64_tr_b8 v[124:125], v63 offset:24576\n\t"
    "ds_read_b64_tr_b8 v[126:127], v63 offset:25600\n\t"
    "s_cmp_lg_u32 s74, 0\n\t"
    "s_cbranch_scc1 L_gba2_%=\n\t"
    "s_waitcnt vmcnt(12)\n\t"
    "s_barrier\n\t"
    "L_gba2_%=:\n\t"
    "s_waitcnt lgkmcnt(12)\n\t"
    "v_mfma_f32_16x16x128_f8f6f4 v[208:211], v[96:103], v[144:151], v[208:211] cbsz:1\n\t"
    "ds_read_b64_tr_b8 v[64:65], v56 offset:32768\n\t"
    "ds_read_b64_tr_b8 v[66:67], v56 offset:33792\n\t"
    "v_mfma_f32_16x16x128_f8f6f4 v[240:243], v[96:103], v[152:159], v[240:243] cbsz:1\n\t"
    "ds_read_b64_tr_b8 v[68:69], v56 offset:40960\n\t"
    "ds_read_b64_tr_b8 v[70:71], v56 offset:41984\n\t"
    "s_waitcnt lgkmcnt(12)\n\t"
    "v_mfma_f32_16x16x128_f8f6f4 v[212:215], v[104:111], v[144:151], v[212:215] cbsz:1\n\t"
    "ds_read_b64_tr_b8 v[72:73], v57 offset:32768\n\t"
    "ds_read_b64_tr_b8 v[74:75], v57 offset:33792\n\t"
    "v_mfma_f32_16x16x128_f8f6f4 v[244:247], v[104:111], v[152:159], v[244:247] cbsz:1\n\t"
    "ds_read_b64_tr_b8 v[76:77], v57 offset:40960\n\t"
    "ds_read_b64_tr_b8 v[78:79], v57 offset:41984\n\t"
    "s_waitcnt vmcnt(4)\n\t"
    "v_min3_u32 v53, v160, v161, v162\n\t"
    "v_min3_u32 v54, v163, v164, v165\n\t"
    "s_waitcnt lgkmcnt(12)\n\t"
    "v_mfma_f32_16x16x128_f8f6f4 v[216:219], v[112:119], v[144:151], v[216:219] cbsz:1\n\t"
    "ds_read_b64_tr_b8 v[80:81], v58 offset:32768\n\t"
    "ds_read_b64_tr_b8 v[82:83], v58 offset:33792\n\t"
    "v_min3_u32 v53, v53, v166, v167\n\t"
    "v_min3_u32 v54, v54, v168, v169\n\t"
    "v_mfma_f32_16x16x128_f8f6f4 v[248:251], v[112:119], v[152:159], v[248:251] cbsz:1\n\t"
    "ds_read_b64_tr_b8 v[84:85], v58 offset:40960\n\t"
    "ds_read_b64_tr_b8 v[86:87], v58 offset:41984\n\t"
    "v_min3_u32 v53, v53, v170, v171\n\t"
    "v_min3_u32 v54, v54, v172, v173\n\t"
    "s_waitcnt lgkmcnt(12)\n\t"
    "v_mfma_f32_16x16x128_f8f6f4 v[220:223], v[120:127], v[144:151], v[220:223] cbsz:1\n\t"
    "ds_read_b64_tr_b8 v[88:89], v59 offset:32768\n\t"
    "ds_read_b64_tr_b8 v[90:91], v59 offset:33792\n\t"
    "v_min3_u32 v53, v53, v174, v175\n\t"
    "v_min_u32 v53, v53, v54\n\t"
    "v_mfma_f32_16x16x128_f8f6f4 v[252:255], v[120:127], v[152:159], v[252:255] cbsz:1\n\t"
    "ds_read_b64_tr_b8 v[92:93], v59 offset:40960\n\t"
    "ds_read_b64_tr_b8 v[94:95], v59 offset:41984\n\t"
    "v_cmp_eq_u32 vcc, 0, v53\n\t"
    "s_cbranch_vccz L_ok26_%=\n\t"
    "v_readlane_b32 s77, %[tx], 26\n\t"
    "s_mov_b32 s82, s77\n\t"
    "s_getpc_b64 s[58:59]\n\t"
    "s_branch L_slow2_%=\n\t"
    "L_ok26_%=:\n\t"
    "s_waitcnt lgkmcnt(12)\n\t"
    "v_mfma_f32_16x16x128_f8f6f4 v[192:195], v[64:71], v[160:167], v[192:195] cbsz:1\n\t"
    "ds_read_b64_tr_b8 v[96:97], v60 offset:32768\n\t"
    "ds_read_b64_tr_b8 v[98:99], v60 offset:33792\n\t"
    "v_mfma_f32_16x16x128_f8f6f4 v[224:227], v[64:71], v[168:175], v[224:227] cbsz:1\n\t"
    "ds_read_b64_tr_b8 v[100:101], v60 offset:40960\n\t"
    "ds_read_b64_tr_b8 v[102:103], v60 offset:41984\n\t"
    "v_readlane_b32 s77, %[tx], 28\n\t"
    "s_mov_b32 s75, s77\n\t"
    "buffer_load_dwordx4 v[128:131], %[xvoff], s[60:63], s75 offen offset:0 sc0 sc1\n\t"
    "s_waitcnt lgkmcnt(12)\n\t"
    "v_mfma_f32_16x16x128_f8f6f4 v[196:199], v[72:79], v[160:167], v[196:199] cbsz:1\n\t"
    "ds_read_b64_tr_b8 v[104:105], v61 offset:32768\n\t"
    "ds_read_b64_tr_b8 v[106:107], v61 offset:33792\n\t"
    "buffer_load_dwordx4 v[132:135], %[xvoff], s[60:63], s75 offen offset:1024 sc0 sc1\n\t"
    "v_mfma_f32_16x16x128_f8f6f4 v[228:231], v[72:79], v[168:175], v[228:231] cbsz:1\n\t"
    "ds_read_b64_tr_b8 v[108:109], v61 offset:40960\n\t"
    "ds_read_b64_tr_b8 v[110:111], v61 offset:41984\n\t"
    "s_waitcnt lgkmcnt(12)\n\t"
    "v_mfma_f32_16x16x128_f8f6f4 v[200:203], v[80:87], v[160:167], v[200:203] cbsz:1\n\t"
    "ds_read_b64_tr_b8 v[112:113], v62 offset:32768\n\t"
    "ds_read_b64_tr_b8 v[114:115], v62 offset:33792\n\t"
    "buffer_load_dwordx4 v[136:139], %[xvoff], s[60:63], s75 offen offset:2048 sc0 sc1\n\t"
    "v_mfma_f32_16x16x128_f8f6f4 v[232:235], v[80:87], v[168:175], v[232:235] cbsz:1\n\t"
    "ds_read_b64_tr_b8 v[116:117], v62 offset:40960\n\t"
    "ds_read_b64_tr_b8 v[118:119], v62 offset:41984\n\t"
    "s_waitcnt lgkmcnt(12)\n\t"
    "v_mfma_f32_16x16x128_f8f6f4 v[204:207], v[88:95], v[160:167], v[204:207] cbsz:1\n\t"
    "ds_read_b64_tr_b8 v[120:121], v63 offset:32768\n\t"
    "ds_read_b64_tr_b8 v[122:123], v63 offset:33792\n\t"
    "buffer_load_dwordx4 v[140:143], %[xvoff], s[60:63], s75 offen offset:3072 sc0 sc1\n\t"
    "v_mfma_f32_16x16x128_f8f6f4 v[236:239], v[88:95], v[168:175], v[236:239] cbsz:1\n\t"
    "ds_read_b64_tr_b8 v[124:125], v63 offset:40960\n\t"
    "ds_read_b64_tr_b8 v[126:127], v63 offset:41984\n\t"
    "s_waitcnt lgkmcnt(12)\n\t"
    "v_mfma_f32_16x16x128_f8f6f4 v[208:211], v[96:103], v[160:167], v[208:211] cbsz:1\n\t"
    "ds_read_b64_tr_b8 v[64:65], v56 offset:49152\n\t"
    "ds_read_b64_tr_b8 v[66:67], v56 offset:50176\n\t"
    "v_mfma_f32_16x16x128_f8f6f4 v[240:243], v[96:103], v[168:175], v[240:243] cbsz:1\n\t"
    "ds_read_b64_tr_b8 v[68:69], v56 offset:57344\n\t"
    "ds_read_b64_tr_b8 v[70:71], v56 offset:58368\n\t"
    "s_waitcnt lgkmcnt(12)\n\t"
    "v_mfma_f32_16x16x128_f8f6f4 v[212:215], v[104:111], v[160:167], v[212:215] cbsz:1\n\t"
    "ds_read_b64_tr_b8 v[72:73], v57 offset:49152\n\t"
    "ds_read_b64_tr_b8 v[74:75], v57 offset:50176\n\t"
    "v_mfma_f32_16x16x128_f8f6f4 v[244:247], v[104:111], v[168:175], v[244:247] cbsz:1\n\t"
    "ds_read_b64_tr_b8 v[76:77], v57 offset:57344\n\t"
    "ds_read_b64_tr_b8 v[78:79], v57 offset:58368\n\t"
    "s_waitcnt vmcnt(4)\n\t"
    "v_min3_u32 v53, v176, v177, v178\n\t"
    "v_min3_u32 v54, v179, v180, v181\n\t"
    "s_waitcnt lgkmcnt(12)\n\t"
    "v_mfma_f32_16x16x128_f8f6f4 v[216:219], v[112:119], v[160:167], v[216:219] cbsz:1\n\t"
    "ds_read_b64_tr_b8 v[80:81], v58 offset:49152\n\t"
    "ds_read_b64_tr_b8 v[82:83], v58 offset:50176\n\t"
    "v_min3_u32 v53, v53, v182, v183\n\t"
    "v_min3_u32 v54, v54, v184, v185\n\t"
    "v_mfma_f32_16x16x128_f8f6f4 v[248:251], v[112:119], v[168:175], v[248:251] cbsz:1\n\t"
    "ds_read_b64_tr_b8 v[84:85], v58 offset:57344\n\t"
    "ds_read_b64_tr_b8 v[86:87], v58 offset:58368\n\t"
    "v_min3_u32 v53, v53, v186, v187\n\t"
    "v_min3_u32 v54, v54, v188, v189\n\t"
    "s_waitcnt lgkmcnt(12)\n\t"
    "v_mfma_f32_16x16x128_f8f6f4 v[220:223], v[120:127], v[160:167], v[220:223] cbsz:1\n\t"
    "ds_read_b64_tr_b8 v[88:89], v59 offset:49152\n\t"
    "ds_read_b64_tr_b8 v[90:91], v59 offset:50176\n\t"
    "v_min3_u32 v53, v53, v190, v191\n\t"
    "v_min_u32 v53, v53, v54\n\t"
    "v_mfma_f32_16x16x128_f8f6f4 v[252:255], v[120:127], v[168:175], v[252:255] cbsz:1\n\t"
    "ds_read_b64_tr_b8 v[92:93], v59 offset:57344\n\t"
    "ds_read_b64_tr_b8 v[94:95], v59 offset:58368\n\t"
    "v_cmp_eq_u32 vcc, 0, v53\n\t"
    "s_cbranch_vccz L_ok27_%=\n\t"
    "v_readlane_b32 s77, %[tx], 27\n\t"
    "s_mov_b32 s82, s77\n\t"
    "s_getpc_b64 s[58:59]\n\t"
    "s_branch L_slow3_%=\n\t"
    "L_ok27_%=:\n\t"
    "s_waitcnt lgkmcnt(12)\n\t"
    "v_mfma_f32_16x16x128_f8f6f4 v[192:195], v[64:71], v[176:183], v[192:195] cbsz:1\n\t"
    "ds_read_b64_tr_b8 v[96:97], v60 offset:49152\n\t"
    "ds_read_b64_tr_b8 v[98:99], v60 offset:50176\n\t"
    "v_xor_b32 v56, 0x10000, v56\n\t"
    "v_mfma_f32_16x16x128_f8f6f4 v[224:227], v[64:71], v[184:191], v[224:227] cbsz:1\n\t"
    "ds_read_b64_tr_b8 v[100:101], v60 offset:57344\n\t"
    "ds_read_b64_tr_b8 v[102:103], v60 offset:58368\n\t"
    "v_readlane_b32 s77, %[tx], 29\n\t"
    "s_mov_b32 s75, s77\n\t"
    "buffer_load_dwordx4 v[144:147], %[xvoff], s[60:63], s75 offen offset:0 sc0 sc1\n\t"
    "v_xor_b32 v57, 0x10000, v57\n\t"
    "s_waitcnt lgkmcnt(12)\n\t"
    "v_mfma_f32_16x16x128_f8f6f4 v[196:199], v[72:79], v[176:183], v[196:199] cbsz:1\n\t"
    "ds_read_b64_tr_b8 v[104:105], v61 offset:49152\n\t"
    "ds_read_b64_tr_b8 v[106:107], v61 offset:50176\n\t"
    "buffer_load_dwordx4 v[148:151], %[xvoff], s[60:63], s75 offen offset:1024 sc0 sc1\n\t"
    "v_xor_b32 v58, 0x10000, v58\n\t"
    "v_mfma_f32_16x16x128_f8f6f4 v[228:231], v[72:79], v[184:191], v[228:231] cbsz:1\n\t"
    "ds_read_b64_tr_b8 v[108:109], v61 offset:57344\n\t"
    "ds_read_b64_tr_b8 v[110:111], v61 offset:58368\n\t"
    "v_xor_b32 v59, 0x10000, v59\n\t"
    "s_waitcnt lgkmcnt(12)\n\t"
    "v_mfma_f32_16x16x128_f8f6f4 v[200:203], v[80:87], v[176:183], v[200:203] cbsz:1\n\t"
    "ds_read_b64_tr_b8 v[112:113], v62 offset:49152\n\t"
    "ds_read_b64_tr_b8 v[114:115], v62 offset:50176\n\t"
    "buffer_load_dwordx4 v[152:155], %[xvoff], s[60:63], s75 offen offset:2048 sc0 sc1\n\t"
    "v_mfma_f32_16x16x128_f8f6f4 v[232:235], v[80:87], v[184:191], v[232:235] cbsz:1\n\t"
    "ds_read_b64_tr_b8 v[116:117], v62 offset:57344\n\t"
    "ds_read_b64_tr_b8 v[118:119], v62 offset:58368\n\t"
    "s_waitcnt lgkmcnt(12)\n\t"
    "v_mfma_f32_16x16x128_f8f6f4 v[204:207], v[88:95], v[176:183], v[204:207] cbsz:1\n\t"
    "ds_read_b64_tr_b8 v[120:121], v63 offset:49152\n\t"
    "ds_read_b64_tr_b8 v[122:123], v63 offset:50176\n\t"
    "buffer_load_dwordx4 v[156:159], %[xvoff], s[60:63], s75 offen offset:3072 sc0 sc1\n\t"
    "v_mfma_f32_16x16x128_f8f6f4 v[236:239], v[88:95], v[184:191], v[236:239] cbsz:1\n\t"
    "ds_read_b64_tr_b8 v[124:125], v63 offset:57344\n\t"
    "ds_read_b64_tr_b8 v[126:127], v63 offset:58368\n\t"
    "s_waitcnt lgkmcnt(12)\n\t"
    "v_mfma_f32_16x16x128_f8f6f4 v[208:211], v[96:103], v[176:183], v[208:211] cbsz:1\n\t"
    "ds_read_b64_tr_b8 v[64:65], v56 offset:0\n\t"
    "ds_read_b64_tr_b8 v[66:67], v56 offset:1024\n\t"
    "v_xor_b32 v60, 0x10000, v60\n\t"
    "v_mfma_f32_16x16x128_f8f6f4 v[240:243], v[96:103], v[184:191], v[240:243] cbsz:1\n\t"
    "ds_read_b64_tr_b8 v[68:69], v56 offset:8192\n\t"
    "ds_read_b64_tr_b8 v[70:71], v56 offset:9216\n\t"
    "v_xor_b32 v61, 0x10000, v61\n\t"
    "s_waitcnt lgkmcnt(12)\n\t"
    "v_mfma_f32_16x16x128_f8f6f4 v[212:215], v[104:111], v[176:183], v[212:215] cbsz:1\n\t"
    "ds_read_b64_tr_b8 v[72:73], v57 offset:0\n\t"
    "ds_read_b64_tr_b8 v[74:75], v57 offset:1024\n\t"
    "v_xor_b32 v62, 0x10000, v62\n\t"
    "v_mfma_f32_16x16x128_f8f6f4 v[244:247], v[104:111], v[184:191], v[244:247] cbsz:1\n\t"
    "ds_read_b64_tr_b8 v[76:77], v57 offset:8192\n\t"
    "ds_read_b64_tr_b8 v[78:79], v57 offset:9216\n\t"
    "s_waitcnt vmcnt(4)\n\t"
    "v_min3_u32 v53, v128, v129, v130\n\t"
    "v_min3_u32 v54, v131, v132, v133\n\t"
    "v_xor_b32 v63, 0x10000, v63\n\t"
    "s_waitcnt lgkmcnt(12)\n\t"
    "v_mfma_f32_16x16x128_f8f6f4 v[216:219], v[112:119], v[176:183], v[216:219] cbsz:1\n\t"
    "ds_read_b64_tr_b8 v[80:81], v58 offset:0\n\t"
    "ds_read_b64_tr_b8 v[82:83], v58 offset:1024\n\t"
    "v_min3_u32 v53, v53, v134, v135\n\t"
    "v_min3_u32 v54, v54, v136, v137\n\t"
    "v_mfma_f32_16x16x128_f8f6f4 v[248:251], v[112:119], v[184:191], v[248:251] cbsz:1\n\t"
    "ds_read_b64_tr_b8 v[84:85], v58 offset:8192\n\t"
    "ds_read_b64_tr_b8 v[86:87], v58 offset:9216\n\t"
    "v_min3_u32 v53, v53, v138, v139\n\t"
    "v_min3_u32 v54, v54, v140, v141\n\t"
    "s_waitcnt lgkmcnt(12)\n\t"
    "v_mfma_f32_16x16x128_f8f6f4 v[220:223], v[120:127], v[176:183], v[220:223] cbsz:1\n\t"
    "ds_read_b64_tr_b8 v[88:89], v59 offset:0\n\t"
    "ds_read_b64_tr_b8 v[90:91], v59 offset:1024\n\t"
    "v_min3_u32 v53, v53, v142, v143\n\t"
    "v_min_u32 v53, v53, v54\n\t"
    "v_mfma_f32_16x16x128_f8f6f4 v[252:255], v[120:127], v[184:191], v[252:255] cbsz:1\n\t"
    "ds_read_b64_tr_b8 v[92:93], v59 offset:8192\n\t"
    "ds_read_b64_tr_b8 v[94:95], v59 offset:9216\n\t"
    "v_cmp_eq_u32 vcc, 0, v53\n\t"
    "s_cbranch_vccz L_ok28_%=\n\t"
    "v_readlane_b32 s77, %[tx], 28\n\t"
    "s_mov_b32 s82, s77\n\t"
    "s_getpc_b64 s[58:59]\n\t"
    "s_branch L_slow0_%=\n\t"
    "L_ok28_%=:\n\t"
    "s_waitcnt lgkmcnt(12)\n\t"
    "v_mfma_f32_16x16x128_f8f6f4 v[192:195], v[64:71], v[128:135], v[192:195] cbsz:1\n\t"
    "ds_read_b64_tr_b8 v[96:97], v60 offset:0\n\t"
    "ds_read_b64_tr_b8 v[98:99], v60 offset:1024\n\t"
    "v_mfma_f32_16x16x128_f8f6f4 v[224:227], v[64:71], v[136:143], v[224:227] cbsz:1\n\t"
    "ds_read_b64_tr_b8 v[100:101], v60 offset:8192\n\t"
    "ds_read_b64_tr_b8 v[102:103], v60 offset:9216\n\t"
    "v_readlane_b32 s77, %[tx], 30\n\t"
    "s_mov_b32 s75, s77\n\t"
    "buffer_load_dwordx4 v[160:163], %[xvoff], s[60:63], s75 offen offset:0 sc0 sc1\n\t"
    "s_waitcnt lgkmcnt(12)\n\t"
    "v_mfma_f32_16x16x128_f8f6f4 v[196:199], v[72:79], v[128:135], v[196:199] cbsz:1\n\t"
    "ds_read_b64_tr_b8 v[104:105], v61 offset:0\n\t"
    "ds_read_b64_tr_b8 v[106:107], v61 offset:1024\n\t"
    "buffer_load_dwordx4 v[164:167], %[xvoff], s[60:63], s75 offen offset:1024 sc0 sc1\n\t"
    "v_mfma_f32_16x16x128_f8f6f4 v[228:231], v[72:79], v[136:143], v[228:231] cbsz:1\n\t"
    "ds_read_b64_tr_b8 v[108:109], v61 offset:8192\n\t"
    "ds_read_b64_tr_b8 v[110:111], v61 offset:9216\n\t"
    "s_waitcnt lgkmcnt(12)\n\t"
    "v_mfma_f32_16x16x128_f8f6f4 v[200:203], v[80:87], v[128:135], v[200:203] cbsz:1\n\t"
    "ds_read_b64_tr_b8 v[112:113], v62 offset:0\n\t"
    "ds_read_b64_tr_b8 v[114:115], v62 offset:1024\n\t"
    "buffer_load_dwordx4 v[168:171], %[xvoff], s[60:63], s75 offen offset:2048 sc0 sc1\n\t"
    "v_mfma_f32_16x16x128_f8f6f4 v[232:235], v[80:87], v[136:143], v[232:235] cbsz:1\n\t"
    "ds_read_b64_tr_b8 v[116:117], v62 offset:8192\n\t"
    "ds_read_b64_tr_b8 v[118:119], v62 offset:9216\n\t"
    "s_waitcnt lgkmcnt(12)\n\t"
    "v_mfma_f32_16x16x128_f8f6f4 v[204:207], v[88:95], v[128:135], v[204:207] cbsz:1\n\t"
    "ds_read_b64_tr_b8 v[120:121], v63 offset:0\n\t"
    "ds_read_b64_tr_b8 v[122:123], v63 offset:1024\n\t"
    "buffer_load_dwordx4 v[172:175], %[xvoff], s[60:63], s75 offen offset:3072 sc0 sc1\n\t"
    "v_mfma_f32_16x16x128_f8f6f4 v[236:239], v[88:95], v[136:143], v[236:239] cbsz:1\n\t"
    "ds_read_b64_tr_b8 v[124:125], v63 offset:8192\n\t"
    "ds_read_b64_tr_b8 v[126:127], v63 offset:9216\n\t"
    "s_waitcnt lgkmcnt(12)\n\t"
    "v_mfma_f32_16x16x128_f8f6f4 v[208:211], v[96:103], v[128:135], v[208:211] cbsz:1\n\t"
    "ds_read_b64_tr_b8 v[64:65], v56 offset:16384\n\t"
    "ds_read_b64_tr_b8 v[66:67], v56 offset:17408\n\t"
    "v_mfma_f32_16x16x128_f8f6f4 v[240:243], v[96:103], v[136:143], v[240:243] cbsz:1\n\t"
    "ds_read_b64_tr_b8 v[68:69], v56 offset:24576\n\t"
    "ds_read_b64_tr_b8 v[70:71], v56 offset:25600\n\t"
    "s_waitcnt lgkmcnt(12)\n\t"
    "v_mfma_f32_16x16x128_f8f6f4 v[212:215], v[104:111], v[128:135], v[212:215] cbsz:1\n\t"
    "ds_read_b64_tr_b8 v[72:73], v57 offset:16384\n\t"
    "ds_read_b64_tr_b8 v[74:75], v57 offset:17408\n\t"
    "v_mfma_f32_16x16x128_f8f6f4 v[244:247], v[104:111], v[136:143], v[244:247] cbsz:1\n\t"
    "ds_read_b64_tr_b8 v[76:77], v57 offset:24576\n\t"
    "ds_read_b64_tr_b8 v[78:79], v57 offset:25600\n\t"
    "s_waitcnt vmcnt(4)\n\t"
    "v_min3_u32 v53, v144, v145, v146\n\t"
    "v_min3_u32 v54, v147, v148, v149\n\t"
    "s_waitcnt lgkmcnt(12)\n\t"
    "v_mfma_f32_16x16x128_f8f6f4 v[216:219], v[112:119], v[128:135], v[216:219] cbsz:1\n\t"
    "ds_read_b64_tr_b8 v[80:81], v58 offset:16384\n\t"
    "ds_read_b64_tr_b8 v[82:83], v58 offset:17408\n\t"
    "v_min3_u32 v53, v53, v150, v151\n\t"
    "v_min3_u32 v54, v54, v152, v153\n\t"
    "v_mfma_f32_16x16x128_f8f6f4 v[248:251], v[112:119], v[136:143], v[248:251] cbsz:1\n\t"
    "ds_read_b64_tr_b8 v[84:85], v58 offset:24576\n\t"
    "ds_read_b64_tr_b8 v[86:87], v58 offset:25600\n\t"
    "v_min3_u32 v53, v53, v154, v155\n\t"
    "v_min3_u32 v54, v54, v156, v157\n\t"
    "s_waitcnt lgkmcnt(12)\n\t"
    "v_mfma_f32_16x16x128_f8f6f4 v[220:223], v[120:127], v[128:135], v[220:223] cbsz:1\n\t"
    "ds_read_b64_tr_b8 v[88:89], v59 offset:16384\n\t"
    "ds_read_b64_tr_b8 v[90:91], v59 offset:17408\n\t"
    "v_min3_u32 v53, v53, v158, v159\n\t"
    "v_min_u32 v53, v53, v54\n\t"
    "v_mfma_f32_16x16x128_f8f6f4 v[252:255], v[120:127], v[136:143], v[252:255] cbsz:1\n\t"
    "ds_read_b64_tr_b8 v[92:93], v59 offset:24576\n\t"
    "ds_read_b64_tr_b8 v[94:95], v59 offset:25600\n\t"
    "v_cmp_eq_u32 vcc, 0, v53\n\t"
    "s_cbranch_vccz L_ok29_%=\n\t"
    "v_readlane_b32 s77, %[tx], 29\n\t"
    "s_mov_b32 s82, s77\n\t"
    "s_getpc_b64 s[58:59]\n\t"
    "s_branch L_slow1_%=\n\t"
    "L_ok29_%=:\n\t"
    "s_waitcnt lgkmcnt(12)\n\t"
    "v_mfma_f32_16x16x128_f8f6f4 v[192:195], v[64:71], v[144:151], v[192:195] cbsz:1\n\t"
    "ds_read_b64_tr_b8 v[96:97], v60 offset:16384\n\t"
    "ds_read_b64_tr_b8 v[98:99], v60 offset:17408\n\t"
    "v_xor_b32 v56, 0x30000, v56\n\t"
    "v_mfma_f32_16x16x128_f8f6f4 v[224:227], v[64:71], v[152:159], v[224:227] cbsz:1\n\t"
    "ds_read_b64_tr_b8 v[100:101], v60 offset:24576\n\t"
    "ds_read_b64_tr_b8 v[102:103], v60 offset:25600\n\t"
    "v_readlane_b32 s77, %[tx], 31\n\t"
    "s_mov_b32 s75, s77\n\t"
    "buffer_load_dwordx4 v[176:179], %[xvoff], s[60:63], s75 offen offset:0 sc0 sc1\n\t"
    "v_xor_b32 v57, 0x30000, v57\n\t"
    "s_waitcnt lgkmcnt(12)\n\t"
    "v_mfma_f32_16x16x128_f8f6f4 v[196:199], v[72:79], v[144:151], v[196:199] cbsz:1\n\t"
    "ds_read_b64_tr_b8 v[104:105], v61 offset:16384\n\t"
    "ds_read_b64_tr_b8 v[106:107], v61 offset:17408\n\t"
    "buffer_load_dwordx4 v[180:183], %[xvoff], s[60:63], s75 offen offset:1024 sc0 sc1\n\t"
    "v_xor_b32 v58, 0x30000, v58\n\t"
    "v_mfma_f32_16x16x128_f8f6f4 v[228:231], v[72:79], v[152:159], v[228:231] cbsz:1\n\t"
    "ds_read_b64_tr_b8 v[108:109], v61 offset:24576\n\t"
    "ds_read_b64_tr_b8 v[110:111], v61 offset:25600\n\t"
    "v_xor_b32 v59, 0x30000, v59\n\t"
    "s_waitcnt lgkmcnt(12)\n\t"
    "v_mfma_f32_16x16x128_f8f6f4 v[200:203], v[80:87], v[144:151], v[200:203] cbsz:1\n\t"
    "ds_read_b64_tr_b8 v[112:113], v62 offset:16384\n\t"
    "ds_read_b64_tr_b8 v[114:115], v62 offset:17408\n\t"
    "buffer_load_dwordx4 v[184:187], %[xvoff], s[60:63], s75 offen offset:2048 sc0 sc1\n\t"
    "v_mfma_f32_16x16x128_f8f6f4 v[232:235], v[80:87], v[152:159], v[232:235] cbsz:1\n\t"
    "ds_read_b64_tr_b8 v[116:117], v62 offset:24576\n\t"
    "ds_read_b64_tr_b8 v[118:119], v62 offset:25600\n\t"
    "s_waitcnt lgkmcnt(12)\n\t"
    "v_mfma_f32_16x16x128_f8f6f4 v[204:207], v[88:95], v[144:151], v[204:207] cbsz:1\n\t"
    "ds_read_b64_tr_b8 v[120:121], v63 offset:16384\n\t"
    "ds_read_b64_tr_b8 v[122:123], v63 offset:17408\n\t"
    "buffer_load_dwordx4 v[188:191], %[xvoff], s[60:63], s75 offen offset:3072 sc0 sc1\n\t"
    "v_mfma_f32_16x16x128_f8f6f4 v[236:239], v[88:95], v[152:159], v[236:239] cbsz:1\n\t"
    "ds_read_b64_tr_b8 v[124:125], v63 offset:24576\n\t"
    "ds_read_b64_tr_b8 v[126:127], v63 offset:25600\n\t"
    "s_waitcnt lgkmcnt(12)\n\t"
    "v_mfma_f32_16x16x128_f8f6f4 v[208:211], v[96:103], v[144:151], v[208:211] cbsz:1\n\t"
    "ds_read_b64_tr_b8 v[64:65], v56 offset:0\n\t"
    "ds_read_b64_tr_b8 v[66:67], v56 offset:1024\n\t"
    "v_xor_b32 v60, 0x30000, v60\n\t"
    "v_mfma_f32_16x16x128_f8f6f4 v[240:243], v[96:103], v[152:159], v[240:243] cbsz:1\n\t"
    "ds_read_b64_tr_b8 v[68:69], v56 offset:8192\n\t"
    "ds_read_b64_tr_b8 v[70:71], v56 offset:9216\n\t"
    "v_xor_b32 v61, 0x30000, v61\n\t"
    "s_waitcnt lgkmcnt(12)\n\t"
    "v_mfma_f32_16x16x128_f8f6f4 v[212:215], v[104:111], v[144:151], v[212:215] cbsz:1\n\t"
    "ds_read_b64_tr_b8 v[72:73], v57 offset:0\n\t"
    "ds_read_b64_tr_b8 v[74:75], v57 offset:1024\n\t"
    "v_xor_b32 v62, 0x30000, v62\n\t"
    "v_mfma_f32_16x16x128_f8f6f4 v[244:247], v[104:111], v[152:159], v[244:247] cbsz:1\n\t"
    "ds_read_b64_tr_b8 v[76:77], v57 offset:8192\n\t"
    "ds_read_b64_tr_b8 v[78:79], v57 offset:9216\n\t"
    "s_waitcnt vmcnt(4)\n\t"
    "v_min3_u32 v53, v160, v161, v162\n\t"
    "v_min3_u32 v54, v163, v164, v165\n\t"
    "v_xor_b32 v63, 0x30000, v63\n\t"
    "s_waitcnt lgkmcnt(12)\n\t"
    "v_mfma_f32_16x16x128_f8f6f4 v[216:219], v[112:119], v[144:151], v[216:219] cbsz:1\n\t"
    "ds_read_b64_tr_b8 v[80:81], v58 offset:0\n\t"
    "ds_read_b64_tr_b8 v[82:83], v58 offset:1024\n\t"
    "v_min3_u32 v53, v53, v166, v167\n\t"
    "v_min3_u32 v54, v54, v168, v169\n\t"
    "v_mfma_f32_16x16x128_f8f6f4 v[248:251], v[112:119], v[152:159], v[248:251] cbsz:1\n\t"
    "ds_read_b64_tr_b8 v[84:85], v58 offset:8192\n\t"
    "ds_read_b64_tr_b8 v[86:87], v58 offset:9216\n\t"
    "v_min3_u32 v53, v53, v170, v171\n\t"
    "v_min3_u32 v54, v54, v172, v173\n\t"
    "s_waitcnt lgkmcnt(12)\n\t"
    "v_mfma_f32_16x16x128_f8f6f4 v[220:223], v[120:127], v[144:151], v[220:223] cbsz:1\n\t"
    "ds_read_b64_tr_b8 v[88:89], v59 offset:0\n\t"
    "ds_read_b64_tr_b8 v[90:91], v59 offset:1024\n\t"
    "v_min3_u32 v53, v53, v174, v175\n\t"
    "v_min_u32 v53, v53, v54\n\t"
    "v_mfma_f32_16x16x128_f8f6f4 v[252:255], v[120:127], v[152:159], v[252:255] cbsz:1\n\t"
    "ds_read_b64_tr_b8 v[92:93], v59 offset:8192\n\t"
    "ds_read_b64_tr_b8 v[94:95], v59 offset:9216\n\t"
    "v_cmp_eq_u32 vcc, 0, v53\n\t"
    "s_cbranch_vccz L_ok30_%=\n\t"
    "v_readlane_b32 s77, %[tx], 30\n\t"
    "s_mov_b32 s82, s77\n\t"
    "s_getpc_b64 s[58:59]\n\t"
    "s_branch L_slow2_%=\n\t"
    "L_ok30_%=:\n\t"
    "s_waitcnt lgkmcnt(12)\n\t"
    "v_mfma_f32_16x16x128_f8f6f4 v[192:195], v[64:71], v[160:167], v[192:195] cbsz:1\n\t"
    "ds_read_b64_tr_b8 v[96:97], v60 offset:0\n\t"
    "ds_read_b64_tr_b8 v[98:99], v60 offset:1024\n\t"
    "v_mfma_f32_16x16x128_f8f6f4 v[224:227], v[64:71], v[168:175], v[224:227] cbsz:1\n\t"
    "ds_read_b64_tr_b8 v[100:101], v60 offset:8192\n\t"
    "ds_read_b64_tr_b8 v[102:103], v60 offset:9216\n\t"
    "s_waitcnt lgkmcnt(12)\n\t"
    "v_mfma_f32_16x16x128_f8f6f4 v[196:199], v[72:79], v[160:167], v[196:199] cbsz:1\n\t"
    "ds_read_b64_tr_b8 v[104:105], v61 offset:0\n\t"
    "ds_read_b64_tr_b8 v[106:107], v61 offset:1024\n\t"
    "v_mfma_f32_16x16x128_f8f6f4 v[228:231], v[72:79], v[168:175], v[228:231] cbsz:1\n\t"
    "ds_read_b64_tr_b8 v[108:109], v61 offset:8192\n\t"
    "ds_read_b64_tr_b8 v[110:111], v61 offset:9216\n\t"
    "s_waitcnt lgkmcnt(12)\n\t"
    "v_mfma_f32_16x16x128_f8f6f4 v[200:203], v[80:87], v[160:167], v[200:203] cbsz:1\n\t"
    "ds_read_b64_tr_b8 v[112:113], v62 offset:0\n\t"
    "ds_read_b64_tr_b8 v[114:115], v62 offset:1024\n\t"
    "v_mfma_f32_16x16x128_f8f6f4 v[232:235], v[80:87], v[168:175], v[232:235] cbsz:1\n\t"
    "ds_read_b64_tr_b8 v[116:117], v62 offset:8192\n\t"
    "ds_read_b64_tr_b8 v[118:119], v62 offset:9216\n\t"
    "s_waitcnt lgkmcnt(12)\n\t"
    "v_mfma_f32_16x16x128_f8f6f4 v[204:207], v[88:95], v[160:167], v[204:207] cbsz:1\n\t"
    "ds_read_b64_tr_b8 v[120:121], v63 offset:0\n\t"
    "ds_read_b64_tr_b8 v[122:123], v63 offset:1024\n\t"
    "v_mfma_f32_16x16x128_f8f6f4 v[236:239], v[88:95], v[168:175], v[236:239] cbsz:1\n\t"
    "ds_read_b64_tr_b8 v[124:125], v63 offset:8192\n\t"
    "ds_read_b64_tr_b8 v[126:127], v63 offset:9216\n\t"
    "s_waitcnt lgkmcnt(12)\n\t"
    "v_mfma_f32_16x16x128_f8f6f4 v[208:211], v[96:103], v[160:167], v[208:211] cbsz:1\n\t"
    "ds_read_b64_tr_b8 v[64:65], v56 offset:16384\n\t"
    "ds_read_b64_tr_b8 v[66:67], v56 offset:17408\n\t"
    "v_mfma_f32_16x16x128_f8f6f4 v[240:243], v[96:103], v[168:175], v[240:243] cbsz:1\n\t"
    "ds_read_b64_tr_b8 v[68:69], v56 offset:24576\n\t"
    "ds_read_b64_tr_b8 v[70:71], v56 offset:25600\n\t"
    "s_waitcnt lgkmcnt(12)\n\t"
    "v_mfma_f32_16x16x128_f8f6f4 v[212:215], v[104:111], v[160:167], v[212:215] cbsz:1\n\t"
    "ds_read_b64_tr_b8 v[72:73], v57 offset:16384\n\t"
    "ds_read_b64_tr_b8 v[74:75], v57 offset:17408\n\t"
    "v_mfma_f32_16x16x128_f8f6f4 v[244:247], v[104:111], v[168:175], v[244:247] cbsz:1\n\t"
    "ds_read_b64_tr_b8 v[76:77], v57 offset:24576\n\t"
    "ds_read_b64_tr_b8 v[78:79], v57 offset:25600\n\t"
    "s_waitcnt vmcnt(0)\n\t"
    "v_min3_u32 v53, v176, v177, v178\n\t"
    "v_min3_u32 v54, v179, v180, v181\n\t"
    "s_waitcnt lgkmcnt(12)\n\t"
    "v_mfma_f32_16x16x128_f8f6f4 v[216:219], v[112:119], v[160:167], v[216:219] cbsz:1\n\t"
    "ds_read_b64_tr_b8 v[80:81], v58 offset:16384\n\t"
    "ds_read_b64_tr_b8 v[82:83], v58 offset:17408\n\t"
    "v_min3_u32 v53, v53, v182, v183\n\t"
    "v_min3_u32 v54, v54, v184, v185\n\t"
    "v_mfma_f32_16x16x128_f8f6f4 v[248:251], v[112:119], v[168:175], v[248:251] cbsz:1\n\t"
    "ds_read_b64_tr_b8 v[84:85], v58 offset:24576\n\t"
    "ds_read_b64_tr_b8 v[86:87], v58 offset:25600\n\t"
    "v_min3_u32 v53, v53, v186, v187\n\t"
    "v_min3_u32 v54, v54, v188, v189\n\t"
    "s_waitcnt lgkmcnt(12)\n\t"
    "v_mfma_f32_16x16x128_f8f6f4 v[220:223], v[120:127], v[160:167], v[220:223] cbsz:1\n\t"
    "ds_read_b64_tr_b8 v[88:89], v59 offset:16384\n\t"
    "ds_read_b64_tr_b8 v[90:91], v59 offset:17408\n\t"
    "v_min3_u32 v53, v53, v190, v191\n\t"
    "v_min_u32 v53, v53, v54\n\t"
    "v_mfma_f32_16x16x128_f8f6f4 v[252:255], v[120:127], v[168:175], v[252:255] cbsz:1\n\t"
    "ds_read_b64_tr_b8 v[92:93], v59 offset:24576\n\t"
    "ds_read_b64_tr_b8 v[94:95], v59 offset:25600\n\t"
    "v_cmp_eq_u32 vcc, 0, v53\n\t"
    "s_cbranch_vccz L_ok31_%=\n\t"
    "v_readlane_b32 s77, %[tx], 31\n\t"
    "s_mov_b32 s82, s77\n\t"
    "s_getpc_b64 s[58:59]\n\t"
    "s_branch L_slow3_%=\n\t"
    "L_ok31_%=:\n\t"
    "s_waitcnt lgkmcnt(12)\n\t"
    "v_mfma_f32_16x16x128_f8f6f4 v[192:195], v[64:71], v[176:183], v[192:195] cbsz:1\n\t"
    "ds_read_b64_tr_b8 v[96:97], v60 offset:16384\n\t"
    "ds_read_b64_tr_b8 v[98:99], v60 offset:17408\n\t"
    "v_mfma_f32_16x16x128_f8f6f4 v[224:227], v[64:71], v[184:191], v[224:227] cbsz:1\n\t"
    "ds_read_b64_tr_b8 v[100:101], v60 offset:24576\n\t"
    "ds_read_b64_tr_b8 v[102:103], v60 offset:25600\n\t"
    "s_waitcnt lgkmcnt(12)\n\t"
    "v_mfma_f32_16x16x128_f8f6f4 v[196:199], v[72:79], v[176:183], v[196:199] cbsz:1\n\t"
    "ds_read_b64_tr_b8 v[104:105], v61 offset:16384\n\t"
    "ds_read_b64_tr_b8 v[106:107], v61 offset:17408\n\t"
    "v_mfma_f32_16x16x128_f8f6f4 v[228:231], v[72:79], v[184:191], v[228:231] cbsz:1\n\t"
    "ds_read_b64_tr_b8 v[108:109], v61 offset:24576\n\t"
    "ds_read_b64_tr_b8 v[110:111], v61 offset:25600\n\t"
    "s_waitcnt lgkmcnt(12)\n\t"
    "v_mfma_f32_16x16x128_f8f6f4 v[200:203], v[80:87], v[176:183], v[200:203] cbsz:1\n\t"
    "ds_read_b64_tr_b8 v[112:113], v62 offset:16384\n\t"
    "ds_read_b64_tr_b8 v[114:115], v62 offset:17408\n\t"
    "v_mfma_f32_16x16x128_f8f6f4 v[232:235], v[80:87], v[184:191], v[232:235] cbsz:1\n\t"
    "ds_read_b64_tr_b8 v[116:117], v62 offset:24576\n\t"
    "ds_read_b64_tr_b8 v[118:119], v62 offset:25600\n\t"
    "s_waitcnt lgkmcnt(12)\n\t"
    "v_mfma_f32_16x16x128_f8f6f4 v[204:207], v[88:95], v[176:183], v[204:207] cbsz:1\n\t"
    "ds_read_b64_tr_b8 v[120:121], v63 offset:16384\n\t"
    "ds_read_b64_tr_b8 v[122:123], v63 offset:17408\n\t"
    "v_mfma_f32_16x16x128_f8f6f4 v[236:239], v[88:95], v[184:191], v[236:239] cbsz:1\n\t"
    "ds_read_b64_tr_b8 v[124:125], v63 offset:24576\n\t"
    "ds_read_b64_tr_b8 v[126:127], v63 offset:25600\n\t"
    "s_waitcnt lgkmcnt(12)\n\t"
    "v_mfma_f32_16x16x128_f8f6f4 v[208:211], v[96:103], v[176:183], v[208:211] cbsz:1\n\t"
    "v_mfma_f32_16x16x128_f8f6f4 v[240:243], v[96:103], v[184:191], v[240:243] cbsz:1\n\t"
    "s_waitcnt lgkmcnt(8)\n\t"
    "v_mfma_f32_16x16x128_f8f6f4 v[212:215], v[104:111], v[176:183], v[212:215] cbsz:1\n\t"
    "v_mfma_f32_16x16x128_f8f6f4 v[244:247], v[104:111], v[184:191], v[244:247] cbsz:1\n\t"
    "s_waitcnt lgkmcnt(4)\n\t"
    "v_mfma_f32_16x16x128_f8f6f4 v[216:219], v[112:119], v[176:183], v[216:219] cbsz:1\n\t"
    "v_mfma_f32_16x16x128_f8f6f4 v[248:251], v[112:119], v[184:191], v[248:251] cbsz:1\n\t"
    "s_waitcnt lgkmcnt(0)\n\t"
    "v_mfma_f32_16x16x128_f8f6f4 v[220:223], v[120:127], v[176:183], v[220:223] cbsz:1\n\t"
    "v_mfma_f32_16x16x128_f8f6f4 v[252:255], v[120:127], v[184:191], v[252:255] cbsz:1\n\t"
    "s_cmp_lg_u32 s70, 0\n\t"
    "s_cselect_b32 s98, 0x7fc00000, s98\n\t"
    "s_mov_b32 s99, s98\n\t"
    "s_mov_b32 s101, s100\n\t"
    "s_mov_b32 s77, 0xc2ad1f97\n\t"
    "v_mov_b32 v53, 0x42ad1f97\n\t"
    "s_mov_b32 s96, 1.0\n\t"
    "s_mov_b32 s97, 1.0\n\t"
    "s_mov_b32 s60, 0\n\t"
    "s_mov_b32 s61, 524288\n\t"
    "s_mov_b32 s62, 1048576\n\t"
    "s_mov_b32 s63, 1572864\n\t"
    "v_pk_mul_f32 v[64:65], v[192:193], s[100:101]\n\t"
    "v_pk_mul_f32 v[66:67], v[194:195], s[100:101]\n\t"
    "v_pk_mul_f32 v[68:69], v[196:197], s[100:101]\n\t"
    "v_pk_mul_f32 v[70:71], v[198:199], s[100:101]\n\t"
    "v_med3_f32 v64, v64, s77, v53\n\t"
    "v_med3_f32 v65, v65, s77, v53\n\t"
    "v_med3_f32 v66, v66, s77, v53\n\t"
    "v_med3_f32 v67, v67, s77, v53\n\t"
    "v_med3_f32 v68, v68, s77, v53\n\t"
    "v_med3_f32 v69, v69, s77, v53\n\t"
    "v_med3_f32 v70, v70, s77, v53\n\t"
    "v_med3_f32 v71, v71, s77, v53\n\t"
    "v_exp_f32 v64, v64\n\t"
    "v_exp_f32 v65, v65\n\t"
    "v_exp_f32 v66, v66\n\t"
    "v_exp_f32 v67, v67\n\t"
    "v_exp_f32 v68, v68\n\t"
    "v_exp_f32 v69, v69\n\t"
    "v_exp_f32 v70, v70\n\t"
    "v_exp_f32 v71, v71\n\t"
    "v_pk_mul_f32 v[80:81], v[192:193], s[98:99]\n\t"
    "v_pk_mul_f32 v[82:83], v[194:195], s[98:99]\n\t"
    "v_pk_mul_f32 v[84:85], v[196:197], s[98:99]\n\t"
    "v_pk_mul_f32 v[86:87], v[198:199], s[98:99]\n\t"
    "v_pk_add_f32 v[64:65], v[64:65], s[96:97]\n\t"
    "v_pk_add_f32 v[66:67], v[66:67], s[96:97]\n\t"
    "v_pk_add_f32 v[68:69], v[68:69], s[96:97]\n\t"
    "v_pk_add_f32 v[70:71], v[70:71], s[96:97]\n\t"
    "v_rcp_f32 v64, v64\n\t"
    "v_rcp_f32 v65, v65\n\t"
    "v_rcp_f32 v66, v66\n\t"
    "v_rcp_f32 v67, v67\n\t"
    "v_rcp_f32 v68, v68\n\t"
    "v_rcp_f32 v69, v69\n\t"
    "v_rcp_f32 v70, v70\n\t"
    "v_rcp_f32 v71, v71\n\t"
    "s_nop 0\n\t"
    "v_pk_mul_f32 v[80:81], v[80:81], v[64:65]\n\t"
    "v_pk_mul_f32 v[82:83], v[82:83], v[66:67]\n\t"
    "v_pk_mul_f32 v[84:85], v[84:85], v[68:69]\n\t"
    "v_pk_mul_f32 v[86:87], v[86:87], v[70:71]\n\t"
    "ds_write_b128 %[ew0], v[80:83]\n\t"
    "ds_write_b128 %[ew1], v[84:87]\n\t"
    "ds_read_b128 v[96:99], %[erd]\n\t"
    "ds_read_b128 v[100:103], %[erd] offset:1024\n\t"
    "v_pk_mul_f32 v[72:73], v[224:225], s[100:101]\n\t"
    "v_pk_mul_f32 v[74:75], v[226:227], s[100:101]\n\t"
    "v_pk_mul_f32 v[76:77], v[228:229], s[100:101]\n\t"
    "v_pk_mul_f32 v[78:79], v[230:231], s[100:101]\n\t"
    "v_med3_f32 v72, v72, s77, v53\n\t"
    "v_med3_f32 v73, v73, s77, v53\n\t"
    "v_med3_f32 v74, v74, s77, v53\n\t"
    "v_med3_f32 v75, v75, s77, v53\n\t"
    "v_med3_f32 v76, v76, s77, v53\n\t"
    "v_med3_f32 v77, v77, s77, v53\n\t"
    "v_med3_f32 v78, v78, s77, v53\n\t"
    "v_med3_f32 v79, v79, s77, v53\n\t"
    "v_exp_f32 v72, v72\n\t"
    "v_exp_f32 v73, v73\n\t"
    "v_exp_f32 v74, v74\n\t"
    "v_exp_f32 v75, v75\n\t"
    "v_exp_f32 v76, v76\n\t"
    "v_exp_f32 v77, v77\n\t"
    "v_exp_f32 v78, v78\n\t"
    "v_exp_f32 v79, v79\n\t"
    "v_pk_mul_f32 v[88:89], v[224:225], s[98:99]\n\t"
    "v_pk_mul_f32 v[90:91], v[226:227], s[98:99]\n\t"
    "v_pk_mul_f32 v[92:93], v[228:229], s[98:99]\n\t"
    "v_pk_mul_f32 v[94:95], v[230:231], s[98:99]\n\t"
    "v_pk_add_f32 v[72:73], v[72:73], s[96:97]\n\t"
    "v_pk_add_f32 v[74:75], v[74:75], s[96:97]\n\t"
    "v_pk_add_f32 v[76:77], v[76:77], s[96:97]\n\t"
    "v_pk_add_f32 v[78:79], v[78:79], s[96:97]\n\t"
    "v_rcp_f32 v72, v72\n\t"
    "v_rcp_f32 v73, v73\n\t"
    "v_rcp_f32 v74, v74\n\t"
    "v_rcp_f32 v75, v75\n\t"
    "v_rcp_f32 v76, v76\n\t"
    "v_rcp_f32 v77, v77\n\t"
    "v_rcp_f32 v78, v78\n\t"
    "v_rcp_f32 v79, v79\n\t"
    "s_waitcnt lgkmcnt(0)\n\t"
    "buffer_store_dwordx4 v[96:99], %[ovo], %[srdO], s60 offen offset:0 sc1\n\t"
    "buffer_store_dwordx4 v[100:103], %[ovo], %[srdO], s61 offen offset:0 sc1\n\t"
    "v_pk_mul_f32 v[88:89], v[88:89], v[72:73]\n\t"
    "v_pk_mul_f32 v[90:91], v[90:91], v[74:75]\n\t"
    "v_pk_mul_f32 v[92:93], v[92:93], v[76:77]\n\t"
    "v_pk_mul_f32 v[94:95], v[94:95], v[78:79]\n\t"
    "ds_write_b128 %[ew0], v[88:91]\n\t"
    "ds_write_b128 %[ew1], v[92:95]\n\t"
    "ds_read_b128 v[104:107], %[erd]\n\t"
    "ds_read_b128 v[108:111], %[erd] offset:1024\n\t"
    "v_pk_mul_f32 v[64:65], v[200:201], s[100:101]\n\t"
    "v_pk_mul_f32 v[66:67], v[202:203], s[100:101]\n\t"
    "v_pk_mul_f32 v[68:69], v[204:205], s[100:101]\n\t"
    "v_pk_mul_f32 v[70:71], v[206:207], s[100:101]\n\t"
    "v_med3_f32 v64, v64, s77, v53\n\t"
    "v_med3_f32 v65, v65, s77, v53\n\t"
    "v_med3_f32 v66, v66, s77, v53\n\t"
    "v_med3_f32 v67, v67, s77, v53\n\t"
    "v_med3_f32 v68, v68, s77, v53\n\t"
    "v_med3_f32 v69, v69, s77, v53\n\t"
    "v_med3_f32 v70, v70, s77, v53\n\t"
    "v_med3_f32 v71, v71, s77, v53\n\t"
    "v_exp_f32 v64, v64\n\t"
    "v_exp_f32 v65, v65\n\t"
    "v_exp_f32 v66, v66\n\t"
    "v_exp_f32 v67, v67\n\t"
    "v_exp_f32 v68, v68\n\t"
    "v_exp_f32 v69, v69\n\t"
    "v_exp_f32 v70, v70\n\t"
    "v_exp_f32 v71, v71\n\t"
    "v_pk_mul_f32 v[80:81], v[200:201], s[98:99]\n\t"
    "v_pk_mul_f32 v[82:83], v[202:203], s[98:99]\n\t"
    "v_pk_mul_f32 v[84:85], v[204:205], s[98:99]\n\t"
    "v_pk_mul_f32 v[86:87], v[206:207], s[98:99]\n\t"
    "v_pk_add_f32 v[64:65], v[64:65], s[96:97]\n\t"
    "v_pk_add_f32 v[66:67], v[66:67], s[96:97]\n\t"
    "v_pk_add_f32 v[68:69], v[68:69], s[96:97]\n\t"
    "v_pk_add_f32 v[70:71], v[70:71], s[96:97]\n\t"
    "v_rcp_f32 v64, v64\n\t"
    "v_rcp_f32 v65, v65\n\t"
    "v_rcp_f32 v66, v66\n\t"
    "v_rcp_f32 v67, v67\n\t"
    "v_rcp_f32 v68, v68\n\t"
    "v_rcp_f32 v69, v69\n\t"
    "v_rcp_f32 v70, v70\n\t"
    "v_rcp_f32 v71, v71\n\t"
    "s_waitcnt lgkmcnt(0)\n\t"
    "buffer_store_dwordx4 v[104:107], %[ovo], %[srdO], s62 offen offset:0 sc1\n\t"
    "buffer_store_dwordx4 v[108:111], %[ovo], %[srdO], s63 offen offset:0 sc1\n\t"
    "v_pk_mul_f32 v[80:81], v[80:81], v[64:65]\n\t"
    "v_pk_mul_f32 v[82:83], v[82:83], v[66:67]\n\t"
    "v_pk_mul_f32 v[84:85], v[84:85], v[68:69]\n\t"
    "v_pk_mul_f32 v[86:87], v[86:87], v[70:71]\n\t"
    "ds_write_b128 %[ew0], v[80:83]\n\t"
    "ds_write_b128 %[ew1], v[84:87]\n\t"
    "ds_read_b128 v[112:115], %[erd]\n\t"
    "ds_read_b128 v[116:119], %[erd] offset:1024\n\t"
    "v_pk_mul_f32 v[72:73], v[232:233], s[100:101]\n\t"
    "v_pk_mul_f32 v[74:75], v[234:235], s[100:101]\n\t"
    "v_pk_mul_f32 v[76:77], v[236:237], s[100:101]\n\t"
    "v_pk_mul_f32 v[78:79], v[238:239], s[100:101]\n\t"
    "v_med3_f32 v72, v72, s77, v53\n\t"
    "v_med3_f32 v73, v73, s77, v53\n\t"
    "v_med3_f32 v74, v74, s77, v53\n\t"
    "v_med3_f32 v75, v75, s77, v53\n\t"
    "v_med3_f32 v76, v76, s77, v53\n\t"
    "v_med3_f32 v77, v77, s77, v53\n\t"
    "v_med3_f32 v78, v78, s77, v53\n\t"
    "v_med3_f32 v79, v79, s77, v53\n\t"
    "v_exp_f32 v72, v72\n\t"
    "v_exp_f32 v73, v73\n\t"
    "v_exp_f32 v74, v74\n\t"
    "v_exp_f32 v75, v75\n\t"
    "v_exp_f32 v76, v76\n\t"
    "v_exp_f32 v77, v77\n\t"
    "v_exp_f32 v78, v78\n\t"
    "v_exp_f32 v79, v79\n\t"
    "v_pk_mul_f32 v[88:89], v[232:233], s[98:99]\n\t"
    "v_pk_mul_f32 v[90:91], v[234:235], s[98:99]\n\t"
    "v_pk_mul_f32 v[92:93], v[236:237], s[98:99]\n\t"
    "v_pk_mul_f32 v[94:95], v[238:239], s[98:99]\n\t"
    "v_pk_add_f32 v[72:73], v[72:73], s[96:97]\n\t"
    "v_pk_add_f32 v[74:75], v[74:75], s[96:97]\n\t"
    "v_pk_add_f32 v[76:77], v[76:77], s[96:97]\n\t"
    "v_pk_add_f32 v[78:79], v[78:79], s[96:97]\n\t"
    "v_rcp_f32 v72, v72\n\t"
    "v_rcp_f32 v73, v73\n\t"
    "v_rcp_f32 v74, v74\n\t"
    "v_rcp_f32 v75, v75\n\t"
    "v_rcp_f32 v76, v76\n\t"
    "v_rcp_f32 v77, v77\n\t"
    "v_rcp_f32 v78, v78\n\t"
    "v_rcp_f32 v79, v79\n\t"
    "s_waitcnt lgkmcnt(0)\n\t"
    "buffer_store_dwordx4 v[112:115], %[ovo], %[srdO], s60 offen offset:128 sc1\n\t"
    "buffer_store_dwordx4 v[116:119], %[ovo], %[srdO], s61 offen offset:128 sc1\n\t"
    "v_pk_mul_f32 v[88:89], v[88:89], v[72:73]\n\t"
    "v_pk_mul_f32 v[90:91], v[90:91], v[74:75]\n\t"
    "v_pk_mul_f32 v[92:93], v[92:93], v[76:77]\n\t"
    "v_pk_mul_f32 v[94:95], v[94:95], v[78:79]\n\t"
    "ds_write_b128 %[ew0], v[88:91]\n\t"
    "ds_write_b128 %[ew1], v[92:95]\n\t"
    "ds_read_b128 v[120:123], %[erd]\n\t"
    "ds_read_b128 v[124:127], %[erd] offset:1024\n\t"
    "v_pk_mul_f32 v[64:65], v[208:209], s[100:101]\n\t"
    "v_pk_mul_f32 v[66:67], v[210:211], s[100:101]\n\t"
    "v_pk_mul_f32 v[68:69], v[212:213], s[100:101]\n\t"
    "v_pk_mul_f32 v[70:71], v[214:215], s[100:101]\n\t"
    "v_med3_f32 v64, v64, s77, v53\n\t"
    "v_med3_f32 v65, v65, s77, v53\n\t"
    "v_med3_f32 v66, v66, s77, v53\n\t"
    "v_med3_f32 v67, v67, s77, v53\n\t"
    "v_med3_f32 v68, v68, s77, v53\n\t"
    "v_med3_f32 v69, v69, s77, v53\n\t"
    "v_med3_f32 v70, v70, s77, v53\n\t"
    "v_med3_f32 v71, v71, s77, v53\n\t"
    "v_exp_f32 v64, v64\n\t"
    "v_exp_f32 v65, v65\n\t"
    "v_exp_f32 v66, v66\n\t"
    "v_exp_f32 v67, v67\n\t"
    "v_exp_f32 v68, v68\n\t"
    "v_exp_f32 v69, v69\n\t"
    "v_exp_f32 v70, v70\n\t"
    "v_exp_f32 v71, v71\n\t"
    "v_pk_mul_f32 v[80:81], v[208:209], s[98:99]\n\t"
    "v_pk_mul_f32 v[82:83], v[210:211], s[98:99]\n\t"
    "v_pk_mul_f32 v[84:85], v[212:213], s[98:99]\n\t"
    "v_pk_mul_f32 v[86:87], v[214:215], s[98:99]\n\t"
    "v_pk_add_f32 v[64:65], v[64:65], s[96:97]\n\t"
    "v_pk_add_f32 v[66:67], v[66:67], s[96:97]\n\t"
    "v_pk_add_f32 v[68:69], v[68:69], s[96:97]\n\t"
    "v_pk_add_f32 v[70:71], v[70:71], s[96:97]\n\t"
    "v_rcp_f32 v64, v64\n\t"
    "v_rcp_f32 v65, v65\n\t"
    "v_rcp_f32 v66, v66\n\t"
    "v_rcp_f32 v67, v67\n\t"
    "v_rcp_f32 v68, v68\n\t"
    "v_rcp_f32 v69, v69\n\t"
    "v_rcp_f32 v70, v70\n\t"
    "v_rcp_f32 v71, v71\n\t"
    "s_waitcnt lgkmcnt(0)\n\t"
    "buffer_store_dwordx4 v[120:123], %[ovo], %[srdO], s62 offen offset:128 sc1\n\t"
    "buffer_store_dwordx4 v[124:127], %[ovo], %[srdO], s63 offen offset:128 sc1\n\t"
    "v_pk_mul_f32 v[80:81], v[80:81], v[64:65]\n\t"
    "v_pk_mul_f32 v[82:83], v[82:83], v[66:67]\n\t"
    "v_pk_mul_f32 v[84:85], v[84:85], v[68:69]\n\t"
    "v_pk_mul_f32 v[86:87], v[86:87], v[70:71]\n\t"
    "ds_write_b128 %[ew0], v[80:83]\n\t"
    "ds_write_b128 %[ew1], v[84:87]\n\t"
    "ds_read_b128 v[96:99], %[erd]\n\t"
    "ds_read_b128 v[100:103], %[erd] offset:1024\n\t"
    "v_pk_mul_f32 v[72:73], v[240:241], s[100:101]\n\t"
    "v_pk_mul_f32 v[74:75], v[242:243], s[100:101]\n\t"
    "v_pk_mul_f32 v[76:77], v[244:245], s[100:101]\n\t"
    "v_pk_mul_f32 v[78:79], v[246:247], s[100:101]\n\t"
    "v_med3_f32 v72, v72, s77, v53\n\t"
    "v_med3_f32 v73, v73, s77, v53\n\t"
    "v_med3_f32 v74, v74, s77, v53\n\t"
    "v_med3_f32 v75, v75, s77, v53\n\t"
    "v_med3_f32 v76, v76, s77, v53\n\t"
    "v_med3_f32 v77, v77, s77, v53\n\t"
    "v_med3_f32 v78, v78, s77, v53\n\t"
    "v_med3_f32 v79, v79, s77, v53\n\t"
    "v_exp_f32 v72, v72\n\t"
    "v_exp_f32 v73, v73\n\t"
    "v_exp_f32 v74, v74\n\t"
    "v_exp_f32 v75, v75\n\t"
    "v_exp_f32 v76, v76\n\t"
    "v_exp_f32 v77, v77\n\t"
    "v_exp_f32 v78, v78\n\t"
    "v_exp_f32 v79, v79\n\t"
    "v_pk_mul_f32 v[88:89], v[240:241], s[98:99]\n\t"
    "v_pk_mul_f32 v[90:91], v[242:243], s[98:99]\n\t"
    "v_pk_mul_f32 v[92:93], v[244:245], s[98:99]\n\t"
    "v_pk_mul_f32 v[94:95], v[246:247], s[98:99]\n\t"
    "v_pk_add_f32 v[72:73], v[72:73], s[96:97]\n\t"
    "v_pk_add_f32 v[74:75], v[74:75], s[96:97]\n\t"
    "v_pk_add_f32 v[76:77], v[76:77], s[96:97]\n\t"
    "v_pk_add_f32 v[78:79], v[78:79], s[96:97]\n\t"
    "v_rcp_f32 v72, v72\n\t"
    "v_rcp_f32 v73, v73\n\t"
    "v_rcp_f32 v74, v74\n\t"
    "v_rcp_f32 v75, v75\n\t"
    "v_rcp_f32 v76, v76\n\t"
    "v_rcp_f32 v77, v77\n\t"
    "v_rcp_f32 v78, v78\n\t"
    "v_rcp_f32 v79, v79\n\t"
    "s_waitcnt lgkmcnt(0)\n\t"
    "buffer_store_dwordx4 v[96:99], %[ovo], %[srdO], s60 offen offset:256 sc1\n\t"
    "buffer_store_dwordx4 v[100:103], %[ovo], %[srdO], s61 offen offset:256 sc1\n\t"
    "v_pk_mul_f32 v[88:89], v[88:89], v[72:73]\n\t"
    "v_pk_mul_f32 v[90:91], v[90:91], v[74:75]\n\t"
    "v_pk_mul_f32 v[92:93], v[92:93], v[76:77]\n\t"
    "v_pk_mul_f32 v[94:95], v[94:95], v[78:79]\n\t"
    "ds_write_b128 %[ew0], v[88:91]\n\t"
    "ds_write_b128 %[ew1], v[92:95]\n\t"
    "ds_read_b128 v[104:107], %[erd]\n\t"
    "ds_read_b128 v[108:111], %[erd] offset:1024\n\t"
    "v_pk_mul_f32 v[64:65], v[216:217], s[100:101]\n\t"
    "v_pk_mul_f32 v[66:67], v[218:219], s[100:101]\n\t"
    "v_pk_mul_f32 v[68:69], v[220:221], s[100:101]\n\t"
    "v_pk_mul_f32 v[70:71], v[222:223], s[100:101]\n\t"
    "v_med3_f32 v64, v64, s77, v53\n\t"
    "v_med3_f32 v65, v65, s77, v53\n\t"
    "v_med3_f32 v66, v66, s77, v53\n\t"
    "v_med3_f32 v67, v67, s77, v53\n\t"
    "v_med3_f32 v68, v68, s77, v53\n\t"
    "v_med3_f32 v69, v69, s77, v53\n\t"
    "v_med3_f32 v70, v70, s77, v53\n\t"
    "v_med3_f32 v71, v71, s77, v53\n\t"
    "v_exp_f32 v64, v64\n\t"
    "v_exp_f32 v65, v65\n\t"
    "v_exp_f32 v66, v66\n\t"
    "v_exp_f32 v67, v67\n\t"
    "v_exp_f32 v68, v68\n\t"
    "v_exp_f32 v69, v69\n\t"
    "v_exp_f32 v70, v70\n\t"
    "v_exp_f32 v71, v71\n\t"
    "v_pk_mul_f32 v[80:81], v[216:217], s[98:99]\n\t"
    "v_pk_mul_f32 v[82:83], v[218:219], s[98:99]\n\t"
    "v_pk_mul_f32 v[84:85], v[220:221], s[98:99]\n\t"
    "v_pk_mul_f32 v[86:87], v[222:223], s[98:99]\n\t"
    "v_pk_add_f32 v[64:65], v[64:65], s[96:97]\n\t"
    "v_pk_add_f32 v[66:67], v[66:67], s[96:97]\n\t"
    "v_pk_add_f32 v[68:69], v[68:69], s[96:97]\n\t"
    "v_pk_add_f32 v[70:71], v[70:71], s[96:97]\n\t"
    "v_rcp_f32 v64, v64\n\t"
    "v_rcp_f32 v65, v65\n\t"
    "v_rcp_f32 v66, v66\n\t"
    "v_rcp_f32 v67, v67\n\t"
    "v_rcp_f32 v68, v68\n\t"
    "v_rcp_f32 v69, v69\n\t"
    "v_rcp_f32 v70, v70\n\t"
    "v_rcp_f32 v71, v71\n\t"
    "s_waitcnt lgkmcnt(0)\n\t"
    "buffer_store_dwordx4 v[104:107], %[ovo], %[srdO], s62 offen offset:256 sc1\n\t"
    "buffer_store_dwordx4 v[108:111], %[ovo], %[srdO], s63 offen offset:256 sc1\n\t"
    "v_pk_mul_f32 v[80:81], v[80:81], v[64:65]\n\t"
    "v_pk_mul_f32 v[82:83], v[82:83], v[66:67]\n\t"
    "v_pk_mul_f32 v[84:85], v[84:85], v[68:69]\n\t"
    "v_pk_mul_f32 v[86:87], v[86:87], v[70:71]\n\t"
    "ds_write_b128 %[ew0], v[80:83]\n\t"
    "ds_write_b128 %[ew1], v[84:87]\n\t"
    "ds_read_b128 v[112:115], %[erd]\n\t"
    "ds_read_b128 v[116:119], %[erd] offset:1024\n\t"
    "v_pk_mul_f32 v[72:73], v[248:249], s[100:101]\n\t"
    "v_pk_mul_f32 v[74:75], v[250:251], s[100:101]\n\t"
    "v_pk_mul_f32 v[76:77], v[252:253], s[100:101]\n\t"
    "v_pk_mul_f32 v[78:79], v[254:255], s[100:101]\n\t"
    "v_med3_f32 v72, v72, s77, v53\n\t"
    "v_med3_f32 v73, v73, s77, v53\n\t"
    "v_med3_f32 v74, v74, s77, v53\n\t"
    "v_med3_f32 v75, v75, s77, v53\n\t"
    "v_med3_f32 v76, v76, s77, v53\n\t"
    "v_med3_f32 v77, v77, s77, v53\n\t"
    "v_med3_f32 v78, v78, s77, v53\n\t"
    "v_med3_f32 v79, v79, s77, v53\n\t"
    "v_exp_f32 v72, v72\n\t"
    "v_exp_f32 v73, v73\n\t"
    "v_exp_f32 v74, v74\n\t"
    "v_exp_f32 v75, v75\n\t"
    "v_exp_f32 v76, v76\n\t"
    "v_exp_f32 v77, v77\n\t"
    "v_exp_f32 v78, v78\n\t"
    "v_exp_f32 v79, v79\n\t"
    "v_pk_mul_f32 v[88:89], v[248:249], s[98:99]\n\t"
    "v_pk_mul_f32 v[90:91], v[250:251], s[98:99]\n\t"
    "v_pk_mul_f32 v[92:93], v[252:253], s[98:99]\n\t"
    "v_pk_mul_f32 v[94:95], v[254:255], s[98:99]\n\t"
    "v_pk_add_f32 v[72:73], v[72:73], s[96:97]\n\t"
    "v_pk_add_f32 v[74:75], v[74:75], s[96:97]\n\t"
    "v_pk_add_f32 v[76:77], v[76:77], s[96:97]\n\t"
    "v_pk_add_f32 v[78:79], v[78:79], s[96:97]\n\t"
    "v_rcp_f32 v72, v72\n\t"
    "v_rcp_f32 v73, v73\n\t"
    "v_rcp_f32 v74, v74\n\t"
    "v_rcp_f32 v75, v75\n\t"
    "v_rcp_f32 v76, v76\n\t"
    "v_rcp_f32 v77, v77\n\t"
    "v_rcp_f32 v78, v78\n\t"
    "v_rcp_f32 v79, v79\n\t"
    "s_waitcnt lgkmcnt(0)\n\t"
    "buffer_store_dwordx4 v[112:115], %[ovo], %[srdO], s60 offen offset:384 sc1\n\t"
    "buffer_store_dwordx4 v[116:119], %[ovo], %[srdO], s61 offen offset:384 sc1\n\t"
    "v_pk_mul_f32 v[88:89], v[88:89], v[72:73]\n\t"
    "v_pk_mul_f32 v[90:91], v[90:91], v[74:75]\n\t"
    "v_pk_mul_f32 v[92:93], v[92:93], v[76:77]\n\t"
    "v_pk_mul_f32 v[94:95], v[94:95], v[78:79]\n\t"
    "ds_write_b128 %[ew0], v[88:91]\n\t"
    "ds_write_b128 %[ew1], v[92:95]\n\t"
    "ds_read_b128 v[120:123], %[erd]\n\t"
    "ds_read_b128 v[124:127], %[erd] offset:1024\n\t"
    "s_waitcnt lgkmcnt(0)\n\t"
    "buffer_store_dwordx4 v[120:123], %[ovo], %[srdO], s62 offen offset:384 sc1\n\t"
    "buffer_store_dwordx4 v[124:127], %[ovo], %[srdO], s63 offen offset:384 sc1\n\t"
    "s_branch L_end%=\n\t"
    "L_slow0_%=:\n\t"
    "s_add_u32 s58, s58, 4\n\t"
    "s_addc_u32 s59, s59, 0\n\t"
    "L_slowl0_%=:\n\t"
    "s_cmp_lg_u32 s70, 0\n\t"
    "s_cbranch_scc1 L_ret0_%=\n\t"
    "s_memrealtime s[80:81]\n\t"
    "s_mov_b32 s77, 0x608000\n\t"
    "buffer_load_dword v52, v55, %[srdM], s77 offen sc1\n\t"
    "buffer_load_dwordx4 v[128:131], %[xvoff], s[60:63], s82 offen offset:0 sc0 sc1\n\t"
    "buffer_load_dwordx4 v[132:135], %[xvoff], s[60:63], s82 offen offset:1024 sc0 sc1\n\t"
    "buffer_load_dwordx4 v[136:139], %[xvoff], s[60:63], s82 offen offset:2048 sc0 sc1\n\t"
    "buffer_load_dwordx4 v[140:143], %[xvoff], s[60:63], s82 offen offset:3072 sc0 sc1\n\t"
    "s_waitcnt vmcnt(0) lgkmcnt(0)\n\t"
    "v_readfirstlane_b32 s78, v52\n\t"
    "s_sub_u32 s80, s80, s69\n\t"
    "s_cmp_gt_u32 s80, 50000000\n\t"
    "s_cbranch_scc1 L_fail0_%=\n\t"
    "s_cmp_eq_u32 s78, s68\n\t"
    "s_cbranch_scc1 L_fail0_%=\n\t"
    "v_min3_u32 v53, v128, v129, v130\n\t"
    "v_min3_u32 v53, v53, v131, v132\n\t"
    "v_min3_u32 v53, v53, v133, v134\n\t"
    "v_min3_u32 v53, v53, v135, v136\n\t"
    "v_min3_u32 v53, v53, v137, v138\n\t"
    "v_min3_u32 v53, v53, v139, v140\n\t"
    "v_min3_u32 v53, v53, v141, v142\n\t"
    "v_min_u32 v53, v53, v143\n\t"
    "v_cmp_eq_u32 vcc, 0, v53\n\t"
    "s_cbranch_vccz L_ret0_%=\n\t"
    "s_sleep 1\n\t"
    "s_branch L_slowl0_%=\n\t"
    "L_fail0_%=:\n\t"
    "s_mov_b32 s70, 1\n\t"
    "v_mov_b32 v52, s68\n\t"
    "buffer_store_dword v52, v55, %[srdM], s77 offen sc1\n\t"
    "L_ret0_%=:\n\t"
    "s_setpc_b64 s[58:59]\n\t"
    "L_slow1_%=:\n\t"
    "s_add_u32 s58, s58, 4\n\t"
    "s_addc_u32 s59, s59, 0\n\t"
    "L_slowl1_%=:\n\t"
    "s_cmp_lg_u32 s70, 0\n\t"
    "s_cbranch_scc1 L_ret1_%=\n\t"
    "s_memrealtime s[80:81]\n\t"
    "s_mov_b32 s77, 0x608000\n\t"
    "buffer_load_dword v52, v55, %[srdM], s77 offen sc1\n\t"
    "buffer_load_dwordx4 v[144:147], %[xvoff], s[60:63], s82 offen offset:0 sc0 sc1\n\t"
    "buffer_load_dwordx4 v[148:151], %[xvoff], s[60:63], s82 offen offset:1024 sc0 sc1\n\t"
    "buffer_load_dwordx4 v[152:155], %[xvoff], s[60:63], s82 offen offset:2048 sc0 sc1\n\t"
    "buffer_load_dwordx4 v[156:159], %[xvoff], s[60:63], s82 offen offset:3072 sc0 sc1\n\t"
    "s_waitcnt vmcnt(0) lgkmcnt(0)\n\t"
    "v_readfirstlane_b32 s78, v52\n\t"
    "s_sub_u32 s80, s80, s69\n\t"
    "s_cmp_gt_u32 s80, 50000000\n\t"
    "s_cbranch_scc1 L_fail1_%=\n\t"
    "s_cmp_eq_u32 s78, s68\n\t"
    "s_cbranch_scc1 L_fail1_%=\n\t"
    "v_min3_u32 v53, v144, v145, v146\n\t"
    "v_min3_u32 v53, v53, v147, v148\n\t"
    "v_min3_u32 v53, v53, v149, v150\n\t"
    "v_min3_u32 v53, v53, v151, v152\n\t"
    "v_min3_u32 v53, v53, v153, v154\n\t"
    "v_min3_u32 v53, v53, v155, v156\n\t"
    "v_min3_u32 v53, v53, v157, v158\n\t"
    "v_min_u32 v53, v53, v159\n\t"
    "v_cmp_eq_u32 vcc, 0, v53\n\t"
    "s_cbranch_vccz L_ret1_%=\n\t"
    "s_sleep 1\n\t"
    "s_branch L_slowl1_%=\n\t"
    "L_fail1_%=:\n\t"
    "s_mov_b32 s70, 1\n\t"
    "v_mov_b32 v52, s68\n\t"
    "buffer_store_dword v52, v55, %[srdM], s77 offen sc1\n\t"
    "L_ret1_%=:\n\t"
    "s_setpc_b64 s[58:59]\n\t"
    "L_slow2_%=:\n\t"
    "s_add_u32 s58, s58, 4\n\t"
    "s_addc_u32 s59, s59, 0\n\t"
    "L_slowl2_%=:\n\t"
    "s_cmp_lg_u32 s70, 0\n\t"
    "s_cbranch_scc1 L_ret2_%=\n\t"
    "s_memrealtime s[80:81]\n\t"
    "s_mov_b32 s77, 0x608000\n\t"
    "buffer_load_dword v52, v55, %[srdM], s77 offen sc1\n\t"
    "buffer_load_dwordx4 v[160:163], %[xvoff], s[60:63], s82 offen offset:0 sc0 sc1\n\t"
    "buffer_load_dwordx4 v[164:167], %[xvoff], s[60:63], s82 offen offset:1024 sc0 sc1\n\t"
    "buffer_load_dwordx4 v[168:171], %[xvoff], s[60:63], s82 offen offset:2048 sc0 sc1\n\t"
    "buffer_load_dwordx4 v[172:175], %[xvoff], s[60:63], s82 offen offset:3072 sc0 sc1\n\t"
    "s_waitcnt vmcnt(0) lgkmcnt(0)\n\t"
    "v_readfirstlane_b32 s78, v52\n\t"
    "s_sub_u32 s80, s80, s69\n\t"
    "s_cmp_gt_u32 s80, 50000000\n\t"
    "s_cbranch_scc1 L_fail2_%=\n\t"
    "s_cmp_eq_u32 s78, s68\n\t"
    "s_cbranch_scc1 L_fail2_%=\n\t"
    "v_min3_u32 v53, v160, v161, v162\n\t"
    "v_min3_u32 v53, v53, v163, v164\n\t"
    "v_min3_u32 v53, v53, v165, v166\n\t"
    "v_min3_u32 v53, v53, v167, v168\n\t"
    "v_min3_u32 v53, v53, v169, v170\n\t"
    "v_min3_u32 v53, v53, v171, v172\n\t"
    "v_min3_u32 v53, v53, v173, v174\n\t"
    "v_min_u32 v53, v53, v175\n\t"
    "v_cmp_eq_u32 vcc, 0, v53\n\t"
    "s_cbranch_vccz L_ret2_%=\n\t"
    "s_sleep 1\n\t"
    "s_branch L_slowl2_%=\n\t"
    "L_fail2_%=:\n\t"
    "s_mov_b32 s70, 1\n\t"
    "v_mov_b32 v52, s68\n\t"
    "buffer_store_dword v52, v55, %[srdM], s77 offen sc1\n\t"
    "L_ret2_%=:\n\t"
    "s_setpc_b64 s[58:59]\n\t"
    "L_slow3_%=:\n\t"
    "s_add_u32 s58, s58, 4\n\t"
    "s_addc_u32 s59, s59, 0\n\t"
    "L_slowl3_%=:\n\t"
    "s_cmp_lg_u32 s70, 0\n\t"
    "s_cbranch_scc1 L_ret3_%=\n\t"
    "s_memrealtime s[80:81]\n\t"
    "s_mov_b32 s77, 0x608000\n\t"
    "buffer_load_dword v52, v55, %[srdM], s77 offen sc1\n\t"
    "buffer_load_dwordx4 v[176:179], %[xvoff], s[60:63], s82 offen offset:0 sc0 sc1\n\t"
    "buffer_load_dwordx4 v[180:183], %[xvoff], s[60:63], s82 offen offset:1024 sc0 sc1\n\t"
    "buffer_load_dwordx4 v[184:187], %[xvoff], s[60:63], s82 offen offset:2048 sc0 sc1\n\t"
    "buffer_load_dwordx4 v[188:191], %[xvoff], s[60:63], s82 offen offset:3072 sc0 sc1\n\t"
    "s_waitcnt vmcnt(0) lgkmcnt(0)\n\t"
    "v_readfirstlane_b32 s78, v52\n\t"
    "s_sub_u32 s80, s80, s69\n\t"
    "s_cmp_gt_u32 s80, 50000000\n\t"
    "s_cbranch_scc1 L_fail3_%=\n\t"
    "s_cmp_eq_u32 s78, s68\n\t"
    "s_cbranch_scc1 L_fail3_%=\n\t"
    "v_min3_u32 v53, v176, v177, v178\n\t"
    "v_min3_u32 v53, v53, v179, v180\n\t"
    "v_min3_u32 v53, v53, v181, v182\n\t"
    "v_min3_u32 v53, v53, v183, v184\n\t"
    "v_min3_u32 v53, v53, v185, v186\n\t"
    "v_min3_u32 v53, v53, v187, v188\n\t"
    "v_min3_u32 v53, v53, v189, v190\n\t"
    "v_min_u32 v53, v53, v191\n\t"
    "v_cmp_eq_u32 vcc, 0, v53\n\t"
    "s_cbranch_vccz L_ret3_%=\n\t"
    "s_sleep 1\n\t"
    "s_branch L_slowl3_%=\n\t"
    "L_fail3_%=:\n\t"
    "s_mov_b32 s70, 1\n\t"
    "v_mov_b32 v52, s68\n\t"
    "buffer_store_dword v52, v55, %[srdM], s77 offen sc1\n\t"
    "L_ret3_%=:\n\t"
    "s_setpc_b64 s[58:59]\n\t"
    "L_end%=:\n\t"
    "s_nop 0\n\t"
    : "+{v[48:51]}"(xp), "+{v52}"(ctr)
    : [psx] "s"(a.sx), [psw] "s"(a.sw), [srdW] "s"(srdW), [srdXin] "s"(srdXin), [srdM] "s"(srdM), [srdO] "s"(srdO), [srdPF] "s"(srdPF),
      [wsMyLo] "s"((unsigned)myws), [wsMyHi] "s"((unsigned)(myws >> 32)),
      [wsDstLo] "s"((unsigned)dws), [wsDstHi] "s"((unsigned)(dws >> 32)),
      [ctroff] "s"(ctroff), [xpoff] "s"(xpoff), [pushstep] "s"(pushstep),
      [zoff] "s"(b * 7168u + w * 1024u), [wid] "s"(w), [m0w] "s"(m0w),
      [pfso] "s"(pfso), [pfw] "s"(pfw), [cpfw] "s"(tile * 8u + w),
      [tw] "v"(tw), [tx] "v"(tx), [wvoff] "v"(wvoff), [tabase] "v"(tabase), [frs] "v"(frs), [xvoff] "v"(lane * 16u), [cvoff] "v"(lane * 128u),
      [ovoff] "v"(ovoff), [pvst] "v"(pvst), [pfvoff] "v"(((lane >> 3) * 16u + (lane & 7)) * NTOT), [xovoff] "v"(xovoff),
      [xrd] "v"(etb + r16 * 128u + ((g4 ^ ((lane >> 1) & 3)) << 4)),
      [ew0] "v"(etb + r16 * 128u + ((g4 ^ (r16 & 7)) << 4)), [ew1] "v"(etb + r16 * 128u + (((g4 + 4) ^ (r16 & 7)) << 4)),
      [erd] "v"(etb + (lane >> 3) * 128u + (((lane & 7) ^ (lane >> 3)) << 4)),
      [ovo] "v"(((rg0 * 16 + (lane >> 3)) * NTOT + ntile * 128 + (lane & 7) * 4) * 4u)
    : "memory", "vcc", "scc", "v46", "v47", "v53", "v54", "v55", "v56", "v57", "v58", "v59", "v60", "v61", "v62", "v63", "v64", "v65", "v66", "v67", "v68", "v69", "v70", "v71", "v72", "v73", "v74", "v75", "v76", "v77", "v78", "v79", "v80", "v81", "v82", "v83", "v84", "v85", "v86", "v87", "v88", "v89", "v90", "v91", "v92", "v93", "v94", "v95", "v96", "v97", "v98", "v99", "v100", "v101", "v102", "v103", "v104", "v105", "v106", "v107", "v108", "v109", "v110", "v111", "v112", "v113", "v114", "v115", "v116", "v117", "v118", "v119", "v120", "v121", "v122", "v123", "v124", "v125", "v126", "v127", "v128", "v129", "v130", "v131", "v132", "v133", "v134", "v135", "v136", "v137", "v138", "v139", "v140", "v141", "v142", "v143", "v144", "v145", "v146", "v147", "v148", "v149", "v150", "v151", "v152", "v153", "v154", "v155", "v156", "v157", "v158", "v159", "v160", "v161", "v162", "v163", "v164", "v165", "v166", "v167", "v168", "v169", "v170", "v171", "v172", "v173", "v174", "v175", "v176", "v177", "v178", "v179", "v180", "v181", "v182", "v183", "v184", "v185", "v186", "v187", "v188", "v189", "v190", "v191", "v192", "v193", "v194", "v195", "v196", "v197", "v198", "v199", "v200", "v201", "v202", "v203", "v204", "v205", "v206", "v207", "v208", "v209", "v210", "v211", "v212", "v213", "v214", "v215", "v216", "v217", "v218", "v219", "v220", "v221", "v222", "v223", "v224", "v225", "v226", "v227", "v228", "v229", "v230", "v231", "v232", "v233", "v234", "v235", "v236", "v237", "v238", "v239", "v240", "v241", "v242", "v243", "v244", "v245", "v246", "v247", "v248", "v249", "v250", "v251", "v252", "v253", "v254", "v255", "s58", "s59", "s60", "s61", "s62", "s63", "s64", "s65", "s66", "s67", "s68", "s69", "s70", "s71", "s72", "s73", "s74", "s75", "s76", "s77", "s78", "s79", "s80", "s81", "s82", "s83", "s84", "s85", "s86", "s87", "s96", "s97", "s98", "s99", "s100", "s101"
      , "s88", "s89", "s90", "s91", "s92", "s93", "s94", "s95"
  );
}


extern "C" void kernel_launch(int rank, int nranks, ncclComm_t comm, hipStream_t stream,
                              void* const* d_in, const int* in_sizes, int n_in,
                              void* d_out, int out_size, const int* in_dtypes, int out_dtype,
                              void* const* d_out_all, void* const* d_ws_all, size_t ws_bytes) {
  if (nranks != NR || (unsigned)rank >= (unsigned)NR || n_in < 4 || ws_bytes < WS_NEED || out_size != MLOC * NTOT ||
      in_sizes[0] != KTOT * KLOC || in_sizes[1] != KTOT * NTOT) {
    fprintf(stderr, "unexpected problem configuration\n");
    return;
  }
  HIP_CHECK(hipFuncSetAttribute((const void*)a2a_gemm_kernel, hipFuncAttributeMaxDynamicSharedMemorySize, LDS_BYTES));
  Args a = {};
  a.x = (const uint8_t*)d_in[0]; a.w = (const uint8_t*)d_in[1];
  a.sx = (const float*)d_in[2]; a.sw = (const float*)d_in[3];
  a.out = (float*)d_out;
  a.myws = (uint8_t*)d_ws_all[rank];
  for (int p = 0; p < NR; ++p) a.dws[p] = (uint8_t*)d_ws_all[(rank + 7 - p) & 7];
  a.rank = (unsigned)rank;
  hipLaunchKernelGGL(a2a_gemm_kernel, dim3(256), dim3(512), LDS_BYTES, stream, a);
  HIP_CHECK(hipGetLastError());
}
```
